```python
import numpy as np

kernel_name = "gpu_gemm_ar_m4096_k4096_n16384_bf16_gelu"


ABSMAX_RTOL = 0.008

COMPUTE_DTYPE = "bfloat16"
EPILOGUE = "gelu"
IN_QUANT = None
OUT_QUANT = None
IN_DTYPES = {"x": "bfloat16", "w_mat": "bfloat16"}
WORLD_SIZE = 8

def _to_bf16(x):
    u = np.ascontiguousarray(x, dtype=np.float32).view(np.uint32)
    return ((u + 0x7FFF + ((u >> 16) & 1)) & 0xFFFF0000).view(np.float32)

def setup_inputs():

    x = _to_bf16(np.random.randn(4096, 4096).astype(np.float32))
    w_mat = _to_bf16((np.random.randn(4096, 16384) / np.sqrt(4096)).astype(np.float32))
    return {"x": x, "w_mat": w_mat}

def _epilogue(y):
    c = 0.7978845608028654
    return (0.5 * y * (1.0 + np.tanh(c * (y + 0.044715 * y ** 3)))).astype(np.float32)

def reference(x, w_mat):
    y = (x @ w_mat).astype(np.float32)
    return _epilogue(y)
```

```cpp
#include <hip/hip_runtime.h>
#include <rccl/rccl.h>
#include <cstdio>

#define HIP_CHECK(expr) do { hipError_t _e = (expr); if (_e != hipSuccess) { \
    fprintf(stderr, "HIP error %s at %s:%d\n", hipGetErrorString(_e), __FILE__, __LINE__); \
    return; } } while (0)

namespace {
#define LAS __attribute__((address_space(3)))
typedef unsigned short u16;
typedef short bf16x8 __attribute__((ext_vector_type(8)));
typedef float f32x4 __attribute__((ext_vector_type(4)));
typedef unsigned u32x4 __attribute__((ext_vector_type(4)));
typedef __amdgpu_buffer_rsrc_t rsrc_t;

#define CFG_OWNH 8
#define CFG_CPR 12
#define CFG_NPULL 28
#define CFG_PU 4

constexpr int NRANK = 8, M = 4096, N = 16384, KS = 512;
constexpr int NWG = 256;
constexpr int OWNH = CFG_OWNH;
constexpr int RH = 128 - NRANK * OWNH;
constexpr int NH = OWNH + RH;
constexpr int NCT = (NH + 1) / 2;
constexpr int CPR = CFG_CPR;
constexpr int NROUND = (NCT + CPR - 1) / CPR;
constexpr int GPX = 2 * CPR;
constexpr int NPULL = CFG_NPULL;
constexpr int NPRE = NWG - NPULL;
constexpr int PU = CFG_PU;
constexpr unsigned BLK = 16384;
constexpr unsigned PPB = 16 / PU;
constexpr unsigned XKK = 32 * BLK, WKK = 128 * BLK;
constexpr unsigned XBYTES = 8 * XKK, SLOT = XBYTES + 8 * WKK;
constexpr size_t STG = 1u << 20;
constexpr size_t WS_NEED = STG + 2 * (size_t)NRANK * SLOT;
constexpr unsigned OUT_BYTES = (unsigned)M * N * 2;
constexpr unsigned ROWB = N * 2;
constexpr unsigned C_CTR = 0, C_TMO = 128, C_SRC = 1024, C_ARR = 4096, C_YD = 16384, C_END = 32768, C_STAMP = 65536;
constexpr int R0 = (2 * CPR - OWNH) < RH ? (2 * CPR - OWNH) : RH;
constexpr int C0 = 32 + R0 + NRANK * OWNH;
constexpr int U0 = (C0 + 3) / 4;
constexpr int C1 = (RH - R0) * 8, U1 = (C1 + 3) / 4;
constexpr int NUNIT = 8 * U0 + U1;
constexpr int WPL = (NPULL / 7) * 8;
constexpr unsigned long long DEADLINE = 50000000ull;
constexpr int LDS_BYTES = 8 * BLK;
static_assert(CPR % 2 == 0 && 2 * CPR >= OWNH && GPX * 8 + NPULL <= NWG && NPULL % 7 == 0 && NROUND <= 4, "config");
static_assert(C_ARR + (NROUND * 8 + 1) * 128 <= C_YD, "ctrl");

__host__ __device__ constexpr int nh_of(int r) { return ((r + 1) * 2 * CPR < NH ? (r + 1) * 2 * CPR : NH) - r * 2 * CPR; }
__host__ __device__ constexpr unsigned gsz_of(int r) { return ((r == 0 ? 32 : 0) + nh_of(r)) * PPB; }
__host__ __device__ constexpr unsigned rstart_of(int r) { return r == 0 ? 0u : rstart_of(r - 1) + 8 * gsz_of(r - 1); }
static_assert(gsz_of(0) >= WPL, "every puller wave starts inside group 0");
constexpr unsigned NPI = rstart_of(NROUND);
constexpr unsigned YCH = OWNH * 16;
constexpr unsigned YPT = 256 * YCH / (64 * PU);
constexpr unsigned NPT = NPI + 16 * YPT;

struct Params { const u16* x; const u16* w; char* ws[NRANK]; char* out[NRANK]; int rank; };

#define RLX_AGENT __ATOMIC_RELAXED, __HIP_MEMORY_SCOPE_AGENT
#define RLX_SYS __ATOMIC_RELAXED, __HIP_MEMORY_SCOPE_SYSTEM
#define AUX_SYS 17
#define AUX_DEV 16

__device__ __forceinline__ unsigned rfl(unsigned v) { return (unsigned)__builtin_amdgcn_readfirstlane((int)v); }
__device__ __forceinline__ rsrc_t mk_rsrc(const void* p, unsigned bytes) { return __builtin_amdgcn_make_buffer_rsrc((void*)p, 0, (int)bytes, 0x00020000); }
__device__ __forceinline__ int panel_of(int h, int rank) { return h < OWNH ? RH + rank * OWNH + h : h - OWNH; }

__device__ __forceinline__ bool expired(unsigned* tmo, unsigned call, unsigned long long t0) {
    if (rfl(__hip_atomic_load(tmo, RLX_AGENT)) == call || __builtin_amdgcn_s_memrealtime() - t0 > DEADLINE) {
        __hip_atomic_store(tmo, call, RLX_AGENT);
        return true;
    }
    return false;
}
__device__ __forceinline__ bool wait_local(const unsigned* p, unsigned target, unsigned* tmo, unsigned call) {
    const unsigned long long t0 = __builtin_amdgcn_s_memrealtime();
    while ((int)(rfl(__hip_atomic_load(p, RLX_AGENT)) - target) < 0) {
        if (expired(tmo, call, t0)) return false;
        __builtin_amdgcn_s_sleep(8);
    }
    asm volatile("" ::: "memory");
    return true;
}

#define STAMP(i) do {} while (0)
#define STAMPW(i) do {} while (0)

__device__ __forceinline__ void prepass(const Params& P, char* myws, unsigned parity, int u) {
    const int tid = threadIdx.x, lane = tid & 63, wave = (int)rfl(tid >> 6), half = wave & 1;
    const rsrc_t rs = mk_rsrc(myws + STG + (size_t)(parity * NRANK + P.rank) * SLOT, SLOT);
    const rsrc_t rx = mk_rsrc(P.x, M * KS * 2), rw = mk_rsrc(P.w, KS * N * 2);
    for (int v = u; v < NUNIT; v += NPRE) {
        int kk, img, cls;
        if (v < 8 * U0) { kk = v / U0; cls = kk; const int bi = (v % U0) * 4 + (wave >> 1);
            img = bi >= C0 ? -1 : (bi < 32 + R0 ? bi : bi + (RH - R0)); }
        else { const int idx = (v - 8 * U0) * 4 + (wave >> 1); kk = idx & 7; cls = 8; img = idx >= C1 ? -1 : 32 + R0 + (idx >> 3); }
        if (img >= 32) {
            const int g = img - 32, kgl = lane & 3, ng = lane >> 2;
            u32x4 in[8];
            const unsigned so = (unsigned)((kk * 64 + (half * 4 + kgl) * 8) * N + g * 128 + ng * 8) * 2u;
#pragma unroll
            for (int j = 0; j < 8; ++j) in[j] = __builtin_amdgcn_raw_buffer_load_b128(rw, so + j * ROWB, 0, 0);
            const unsigned dbase = XBYTES + (unsigned)kk * WKK + (unsigned)g * BLK + (unsigned)((ng >> 2) * 4096 + half * 1024 + (ng & 3) * 256 + ((kgl * 16) ^ (((ng >> 1) & 1) << 5)));
#pragma unroll
            for (int v8 = 0; v8 < 8; ++v8) {
                u32x4 o;
#pragma unroll
                for (int e = 0; e < 4; ++e) o[e] = __builtin_amdgcn_perm(in[2 * e + 1][v8 >> 1], in[2 * e][v8 >> 1], (v8 & 1) ? 0x07060302u : 0x05040100u);
                __builtin_amdgcn_raw_buffer_store_b128(o, rs, dbase + (v8 >> 2) * 2048 + (v8 & 3) * 64, 0, AUX_SYS);
            }
        } else if (img >= 0) {
            const int sb = lane * 16, swz = sb ^ (((sb >> 9) & 1) << 5);
            u32x4 in[8];
            const unsigned so = (unsigned)((img * 128 + half * 64 + (swz >> 6)) * KS + kk * 64) * 2u + (unsigned)(swz & 63);
#pragma unroll
            for (int j = 0; j < 8; ++j) in[j] = __builtin_amdgcn_raw_buffer_load_b128(rx, so + (j >> 1) * (16 * KS * 2) + (j & 1) * 64, 0, 0);
            const unsigned dbase = (unsigned)kk * XKK + (unsigned)img * BLK + (unsigned)(half * 8192 + sb);
#pragma unroll
            for (int j = 0; j < 8; ++j) __builtin_amdgcn_raw_buffer_store_b128(in[j], rs, dbase + j * 1024, 0, AUX_SYS);
        }
        asm volatile("s_waitcnt vmcnt(0)" ::: "memory");
        __syncthreads();
        if (tid == 0) __hip_atomic_fetch_add((unsigned*)(myws + C_SRC + cls * 128), 1u, RLX_SYS);
    }
}

struct Piece { unsigned off; int need; };
__device__ __forceinline__ Piece in_piece(unsigned n, int rank) {
    Piece pc; int r = 0;
#pragma unroll
    for (int i = 1; i < NROUND; ++i) if (n >= rstart_of(i)) r = i;
    unsigned kk = 0, pi = 0;
#pragma unroll
    for (int i = 0; i < NROUND; ++i) if (r == i) { const unsigned q = n - rstart_of(i); kk = q / gsz_of(i); pi = q % gsz_of(i); }
    if (r == 0 && pi < 32 * PPB) pc.off = kk * XKK + pi * (PU * 1024);
    else { if (r == 0) pi -= 32 * PPB;
        pc.off = XBYTES + kk * WKK + (unsigned)panel_of(r * 2 * CPR + (int)(pi / PPB), rank) * BLK + (pi % PPB) * (PU * 1024); }
    pc.need = r == 0 ? (int)kk : 8;
    return pc;
}
__device__ __forceinline__ unsigned group_end(int G) {
    unsigned e = 0;
#pragma unroll
    for (int i = 0; i < NROUND; ++i) if ((G >> 3) == i) e = rstart_of(i) + (unsigned)((G & 7) + 1) * gsz_of(i);
    return e;
}

__device__ __forceinline__ void puller(const Params& P, char* myws, LAS unsigned* lctr, unsigned call, int pi) {
    const int tid = threadIdx.x, lane = tid & 63, wave = (int)rfl(tid >> 6);
    const unsigned parity = call & 1u;
    const unsigned p = (unsigned)(P.rank + 1 + pi % 7) & 7u;
    const unsigned w0 = (unsigned)(pi / 7) * 8u + (unsigned)wave;
    char* pws = P.ws[p];
    const size_t soff = STG + (size_t)(parity * NRANK + p) * SLOT;
    const rsrc_t rsi = mk_rsrc(pws + soff, SLOT), rdi = mk_rsrc(myws + soff, SLOT);
    const rsrc_t rso = mk_rsrc(P.out[p], OUT_BYTES), rdo = mk_rsrc(P.out[P.rank], OUT_BYTES);
    unsigned* tmo = (unsigned*)(myws + C_TMO);
    if (tid < NROUND * 8) lctr[tid] = 0;
    __syncthreads();
    const unsigned* fw = (const unsigned*)(pws + (lane < 16 ? C_SRC + (lane < 9 ? lane : 0) * 128 : C_YD + (lane & 15) * 128));
    const unsigned ft = call * (lane < 8 ? U0 : lane == 8 ? U1 : lane < 16 ? U0 : 8 * OWNH);
    unsigned long long rdy = 0;
    const unsigned ycol = (unsigned)(RH + p * OWNH) * 256u;
    bool ok = true;
    u32x4 bufA[PU], bufB[PU];
    unsigned offA[PU], offB[PU];
    int gsig = 0;
    auto issue = [&](unsigned n, u32x4 (&buf)[PU], unsigned (&off)[PU]) {
        int need;
        if (n < NPI) { const Piece pc = in_piece(n, P.rank); need = pc.need;
#pragma unroll
            for (int u = 0; u < PU; ++u) off[u] = pc.off + u * 1024 + lane * 16; }
        else { const unsigned q = n - NPI; need = 16 + (int)(q / YPT);
#pragma unroll
            for (int u = 0; u < PU; ++u) { const unsigned ci = (q * PU + u) * 64 + lane; off[u] = (ci / YCH) * ROWB + ycol + (ci % YCH) * 16; } }
        if (!((rdy >> need) & 1)) {
            const unsigned long long t0 = __builtin_amdgcn_s_memrealtime();
            for (;;) {
                rdy = __ballot((int)(__hip_atomic_load(fw, RLX_SYS) - ft) >= 0);
                if ((rdy >> need) & 1) break;
                if (expired(tmo, call, t0)) { ok = false; break; }
                __builtin_amdgcn_s_sleep(2);
            }
            asm volatile("" ::: "memory");
        }
        if (n < NPI) {
#pragma unroll
            for (int u = 0; u < PU; ++u) buf[u] = __builtin_amdgcn_raw_buffer_load_b128(rsi, off[u], 0, AUX_SYS); }
        else {
#pragma unroll
            for (int u = 0; u < PU; ++u) buf[u] = __builtin_amdgcn_raw_buffer_load_b128(rso, off[u], 0, AUX_SYS); }
    };
    auto drain = [&](unsigned n, u32x4 (&buf)[PU], unsigned (&off)[PU]) {
        if (n < NPI) {
#pragma unroll
            for (int u = 0; u < PU; ++u) __builtin_amdgcn_raw_buffer_store_b128(buf[u], rdi, off[u], 0, AUX_DEV);
            if (group_end(gsig) <= n + WPL) {
                asm volatile("s_waitcnt vmcnt(0)" ::: "memory");
                do {
                    if (lane == 0) { const unsigned old = __hip_atomic_fetch_add(lctr + gsig, 1u, __ATOMIC_RELAXED, __HIP_MEMORY_SCOPE_WORKGROUP);
                        if (old == 7u) __hip_atomic_fetch_add((unsigned*)(myws + C_ARR + gsig * 128), 1u, RLX_AGENT); }
                    STAMP(2 + gsig);
                    ++gsig;
                } while (gsig < NROUND * 8 && group_end(gsig) <= n + WPL);
            }
        } else {
#pragma unroll
            for (int u = 0; u < PU; ++u) __builtin_amdgcn_raw_buffer_store_b128(buf[u], rdo, off[u], 0, AUX_DEV); }
    };
    unsigned n = w0;
    issue(n, bufA, offA);
    STAMP(1);
    while (ok) {
        bool more = n + WPL < NPT;
        if (more) issue(n + WPL, bufB, offB);
        if (!ok) break;
        drain(n, bufA, offA);
        if (!more) break;
        n += WPL;
        more = n + WPL < NPT;
        if (more) issue(n + WPL, bufA, offA);
        if (!ok) break;
        drain(n, bufB, offB);
        if (!more) break;
        n += WPL;
    }
    if (!ok) {
        const unsigned q = (NPT - 1 - w0) / WPL * WPL + w0 - NPI, ci = q * PU * 64 + lane;
        const u32x4 nanv = {0x7fc07fc0u, 0x7fc07fc0u, 0x7fc07fc0u, 0x7fc07fc0u};
        __builtin_amdgcn_raw_buffer_store_b128(nanv, rdo, (ci / YCH) * ROWB + ycol + (ci % YCH) * 16, 0, AUX_DEV);
    }
}

__device__ __forceinline__ unsigned cvt_pk_bf16(float lo, float hi) { unsigned r; asm volatile("v_cvt_pk_bf16_f32 %0, %1, %2" : "=v"(r) : "v"(lo), "v"(hi)); return r; }
__device__ __forceinline__ float gelu(float y) {
    const float k1 = -2.0f * 0.7978845608028654f * 1.4426950408889634f, k3 = k1 * 0.044715f;
    const float e = __builtin_amdgcn_exp2f(y * __builtin_fmaf(y * y, k3, k1));
    return y * __builtin_amdgcn_rcpf(1.0f + e);
}

__device__ __forceinline__ void gemm(const Params& P, char* myws, LAS unsigned char* lds, unsigned call, int xcd, int j) {
    const int tid = threadIdx.x, wid = (int)rfl(tid >> 6), lane = tid & 63, wr = wid >> 2, wc = wid & 3, fr = lane & 15, fq = lane >> 4;
    const int pm = (xcd & 3) * 4 + (j & 3), cc = (xcd >> 2) * (CPR / 2) + (j >> 2);
    if (cc >= NCT) return;
    unsigned* tmo = (unsigned*)(myws + C_TMO);
    const char* stg = myws + STG + (size_t)(call & 1u) * NRANK * SLOT;
    const char* baseA = stg + (size_t)pm * 2 * BLK;
    const rsrc_t rctl = mk_rsrc(myws, C_END);
    const unsigned voff = (unsigned)tid * 16u, ldsw = (unsigned)wid * 1024u;
    const unsigned atarget = call * NPULL;
    int aoff, boff;
    { const int ob = fr * 64 + fq * 16, sw = ob ^ (((ob >> 9) & 1) << 5); aoff = wr * 8192 + sw; boff = wc * 4096 + sw; }
    bool ok = true;
#define SA(b, h) (((b) * 2 + (h)) * BLK)
#define SB(b, h) ((4 + (b) * 2 + (h)) * BLK)
#define STAGE(bufoff, gbase) do { _Pragma("unroll") for (int _i = 0; _i < 2; ++_i) \
        __builtin_amdgcn_global_load_lds((const unsigned*)((const char*)(gbase) + (voff + _i * 8192u)), (LAS unsigned*)(lds + (bufoff) + ldsw + _i * 8192), 16, 0, AUX_DEV); } while (0)
#define LDA(dst, b, h) do { _Pragma("unroll") for (int m = 0; m < 4; ++m) _Pragma("unroll") for (int k = 0; k < 2; ++k) dst[m][k] = *(const LAS bf16x8*)(lds + SA(b, h) + aoff + m * 2048 + k * 1024); } while (0)
#define LDB(dst, b, h) do { _Pragma("unroll") for (int n = 0; n < 2; ++n) _Pragma("unroll") for (int k = 0; k < 2; ++k) dst[n][k] = *(const LAS bf16x8*)(lds + SB(b, h) + boff + n * 2048 + k * 1024); } while (0)
#define MMA(ai, bj, At, Bt) do { __builtin_amdgcn_s_setprio(1); _Pragma("unroll") for (int m = 0; m < 4; ++m) _Pragma("unroll") for (int n = 0; n < 2; ++n) _Pragma("unroll") for (int k = 0; k < 2; ++k) \
        acc[ai][bj][m][n] = __builtin_amdgcn_mfma_f32_16x16x32_bf16(Bt[n][k], At[m][k], acc[ai][bj][m][n], 0, 0, 0); __builtin_amdgcn_s_setprio(0); } while (0)
#define WAIT_V(n) asm volatile("s_waitcnt vmcnt(" #n ")" ::: "memory")
#define WAIT_L(n) asm volatile("s_waitcnt lgkmcnt(" #n ")" ::: "memory")
#define BAR __builtin_amdgcn_s_barrier()
#define SCHED __builtin_amdgcn_sched_barrier(0)
    f32x4 acc[2][2][4][2];
#pragma unroll
    for (int a = 0; a < 2; ++a)
#pragma unroll
        for (int b = 0; b < 2; ++b)
#pragma unroll
            for (int m = 0; m < 4; ++m)
#pragma unroll
                for (int n = 0; n < 2; ++n) acc[a][b][m][n] = (f32x4){0.f, 0.f, 0.f, 0.f};
    bf16x8 At[4][2], B0[2][2], B1[2][2];
    {
        const unsigned* fw = (const unsigned*)(myws + C_SRC + (lane < 9 ? lane : 0) * 128);
        const unsigned ft = call * (lane == 8 ? U1 : U0);
        const unsigned long long t0 = __builtin_amdgcn_s_memrealtime();
        while (!__all((int)(__hip_atomic_load(fw, RLX_AGENT) - ft) >= 0)) {
            if (expired(tmo, call, t0)) { ok = false; break; }
            __builtin_amdgcn_s_sleep(8);
        }
        ok = wait_local((const unsigned*)(myws + C_ARR), atarget, tmo, call) && ok;
    }
    STAMP(2);
    int c = cc;
    int h1 = 2 * c + 1 < NH ? 2 * c + 1 : 2 * c;
    const char* cB0 = stg + XBYTES + (size_t)panel_of(2 * c, P.rank) * BLK;
    const char* cB1 = stg + XBYTES + (size_t)panel_of(h1, P.rank) * BLK;
    STAGE(SB(0, 0), cB0); STAGE(SA(0, 0), baseA); STAGE(SB(0, 1), cB1); STAGE(SA(0, 1), baseA + BLK);
    if (wr == 1) BAR;
    WAIT_V(4); BAR;
    STAGE(SB(1, 0), cB0 + SLOT); STAGE(SA(1, 0), baseA + SLOT); STAGE(SB(1, 1), cB1 + SLOT);
    WAIT_V(6); BAR;
    unsigned pollv = 0;
    for (int round = 0;; ++round) {
        const int cn = c + CPR;
        const bool has_next = cn < NCT;
        const int h1n = 2 * cn + 1 < NH ? 2 * cn + 1 : 2 * cn;
        const char* nB0 = has_next ? stg + XBYTES + (size_t)panel_of(2 * cn, P.rank) * BLK : cB0;
        const char* nB1 = has_next ? stg + XBYTES + (size_t)panel_of(h1n, P.rank) * BLK : cB1;
        for (int t = 0; t < 64; t += 2) {
            const bool last = t == 62;
            const int t2 = (t + 2) & 63;
            const char* a1 = baseA + (size_t)((unsigned)((t & 7) + 1) * SLOT + (unsigned)(t >> 3) * XKK);
            const char* a2 = baseA + (size_t)((unsigned)(t2 & 7) * SLOT + (unsigned)(t2 >> 3) * XKK);
            const unsigned kb = (unsigned)(t2 & 7) * SLOT + (unsigned)(t2 >> 3) * WKK;
            const char* b20 = (last ? nB0 : cB0) + kb; const char* b21 = (last ? nB1 : cB1) + kb;
            const unsigned gnext = (unsigned)(round * 8 + (t >> 3) + 1);
            if ((t & 7) == 6 && (has_next || !last)) {
                if ((int)(rfl(pollv) - atarget) < 0) ok = wait_local((const unsigned*)(myws + C_ARR + gnext * 128), atarget, tmo, call) && ok;
            }
            LDB(B0, 0, 0); SCHED; LDA(At, 0, 0); STAGE(SA(1, 1), a1 + BLK);
            WAIT_L(8); BAR; WAIT_L(0); MMA(0, 0, At, B0); BAR; SCHED;
            LDB(B1, 0, 1); STAGE(SB(0, 0), b20);
            BAR; WAIT_L(0); MMA(0, 1, At, B1); BAR;
            LDA(At, 0, 1); STAGE(SA(0, 0), a2);
            BAR; WAIT_L(0); MMA(1, 0, At, B0); BAR; SCHED;
            STAGE(SB(0, 1), b21);
            WAIT_V(6); BAR; MMA(1, 1, At, B1); BAR;
            asm volatile("buffer_load_dword %0, off, %1, %2 sc1" : "=v"(pollv) : "s"(rctl), "s"(C_ARR + gnext * 128) : "memory");
            LDB(B0, 1, 0); SCHED; LDA(At, 1, 0); STAGE(SA(0, 1), a2 + BLK);
            WAIT_L(8); BAR; WAIT_L(0); MMA(0, 0, At, B0); BAR; SCHED;
            LDB(B1, 1, 1); STAGE(SB(1, 0), b20 + SLOT);
            BAR; WAIT_L(0); MMA(0, 1, At, B1); BAR;
            LDA(At, 1, 1); STAGE(SA(1, 0), a2 + SLOT);
            BAR; WAIT_L(0); MMA(1, 0, At, B0); BAR; SCHED;
            STAGE(SB(1, 1), b21 + SLOT);
            asm volatile("s_waitcnt vmcnt(6)" : "+v"(pollv) :: "memory"); BAR; MMA(1, 1, At, B1); BAR;
        }
        STAMP(3 + 2 * round);
        {
            const rsrc_t ro = mk_rsrc(P.out[P.rank], OUT_BYTES);
            const unsigned rowoff = (unsigned)(pm * 256 + wr * 64 + fr) * ROWB + (unsigned)(wc * 32 + fq * 8) * 2u;
#pragma unroll
            for (int bj = 0; bj < 2; ++bj) {
                const unsigned colb = (2 * c + bj < NH) ? (unsigned)panel_of(2 * c + bj, P.rank) * 256u : OUT_BYTES;
#pragma unroll
                for (int ai = 0; ai < 2; ++ai)
#pragma unroll
                    for (int m = 0; m < 4; ++m) {
                        const f32x4 v0 = acc[ai][bj][m][0], v1 = acc[ai][bj][m][1];
                        u32x4 o;
                        o.x = cvt_pk_bf16(gelu(v0[0]), gelu(v0[1])); o.y = cvt_pk_bf16(gelu(v0[2]), gelu(v0[3]));
                        o.z = cvt_pk_bf16(gelu(v1[0]), gelu(v1[1])); o.w = cvt_pk_bf16(gelu(v1[2]), gelu(v1[3]));
                        if (!ok) o = (u32x4){0x7fc07fc0u, 0x7fc07fc0u, 0x7fc07fc0u, 0x7fc07fc0u};
                        __builtin_amdgcn_raw_buffer_store_b128(o, ro, rowoff + colb + (unsigned)(ai * 128 + m * 16) * ROWB, 0, AUX_SYS);
                    }
            }
            if (2 * c < OWNH) {
                WAIT_V(0);
                if (lane == 0) __hip_atomic_fetch_add((unsigned*)(myws + C_YD + pm * 128), 2 * c + 1 < OWNH ? 2u : 1u, RLX_SYS);
            }
        }
        STAMP(4 + 2 * round);
        if (!has_next) break;
#pragma unroll
        for (int a = 0; a < 2; ++a)
#pragma unroll
            for (int b = 0; b < 2; ++b)
#pragma unroll
                for (int m = 0; m < 4; ++m)
#pragma unroll
                    for (int n = 0; n < 2; ++n) acc[a][b][m][n] = (f32x4){0.f, 0.f, 0.f, 0.f};
        c = cn; cB0 = nB0; cB1 = nB1;
    }
    WAIT_V(0);
    if (wr == 0) BAR;
#undef SA
#undef SB
#undef STAGE
#undef LDA
#undef LDB
#undef MMA
}

__global__ __launch_bounds__(512, 2) void fused_gemm_allreduce(Params P) {
    extern __shared__ __attribute__((aligned(16))) unsigned char shm[];
    char* myws = P.ws[P.rank];
    const int b = blockIdx.x, xcd = b & 7, j = b >> 3;
    STAMP(0);
    const unsigned call = rfl(__hip_atomic_load((unsigned*)(myws + C_CTR), RLX_AGENT)) / NWG + 1u;
    const int pi = (j - GPX) * 8 + xcd;
    if (j >= GPX && pi < NPULL) puller(P, myws, (LAS unsigned*)shm, call, pi);
    else {
        prepass(P, myws, call & 1u, j < GPX ? b : GPX * 8 + pi - NPULL);
        STAMP(1);
        if (j < GPX) gemm(P, myws, (LAS unsigned char*)shm, call, xcd, j);
    }
    __syncthreads();
    STAMP(63);
    if (threadIdx.x == 0) __hip_atomic_fetch_add((unsigned*)(myws + C_CTR), 1u, RLX_AGENT);
}
}

extern "C" void kernel_launch(int rank, int nranks, ncclComm_t comm, hipStream_t stream,
                              void* const* d_in, const int* in_sizes, int n_in,
                              void* d_out, int out_size, const int* in_dtypes, int out_dtype,
                              void* const* d_out_all, void* const* d_ws_all, size_t ws_bytes) {
    if (nranks != NRANK || (unsigned)rank >= (unsigned)NRANK || n_in < 2 || in_sizes[0] != M * KS || in_sizes[1] != KS * N || out_size != M * N || ws_bytes < WS_NEED) {
        fprintf(stderr, "kernel_launch: unexpected problem geometry\n");
        return;
    }
    static bool attr_set = false;
    if (!attr_set) { HIP_CHECK(hipFuncSetAttribute((const void*)fused_gemm_allreduce, hipFuncAttributeMaxDynamicSharedMemorySize, LDS_BYTES)); attr_set = true; }
    Params P;
    P.x = (const u16*)d_in[0]; P.w = (const u16*)d_in[1]; P.rank = rank;
    for (int p = 0; p < NRANK; ++p) { P.ws[p] = (char*)d_ws_all[p]; P.out[p] = (char*)d_out_all[p]; }
    P.out[rank] = (char*)d_out;
    hipLaunchKernelGGL(fused_gemm_allreduce, dim3(NWG), dim3(512), LDS_BYTES, stream, P);
    HIP_CHECK(hipGetLastError());
}
```

```cpp
#include <hip/hip_runtime.h>
#include <rccl/rccl.h>
#include <cstdio>

#define HIP_CHECK(expr) do { hipError_t _e = (expr); if (_e != hipSuccess) { \
    fprintf(stderr, "HIP error %s at %s:%d\n", hipGetErrorString(_e), __FILE__, __LINE__); \
    return; } } while (0)

namespace {
#define LAS __attribute__((address_space(3)))
typedef unsigned short u16;
typedef short bf16x8 __attribute__((ext_vector_type(8)));
typedef float f32x4 __attribute__((ext_vector_type(4)));
typedef unsigned u32x4 __attribute__((ext_vector_type(4)));
typedef __amdgpu_buffer_rsrc_t rsrc_t;

#define CFG_OWNH 7
#define CFG_CPR 14
#define CFG_NPULL 28
#define CFG_PW 4
#define CFG_SIN 16
#define CFG_SO 17
#define CFG_NPUSH 8
#define CFG_DELAY 350
#define CFG_SPAN 1200
#define CFG_PWIN 4
#define CFG_HU 4

constexpr int NRANK = 8, M = 4096, N = 16384, KS = 512;
constexpr int NWG = 256;
constexpr int OWNH = CFG_OWNH;
constexpr int RH = 128 - NRANK * OWNH;
constexpr int NH = OWNH + RH;
constexpr int NCT = (NH + 1) / 2;
constexpr int CPR = CFG_CPR;
constexpr int NROUND = (NCT + CPR - 1) / CPR;
constexpr int GPX = 2 * CPR;
constexpr int NPULL = CFG_NPULL;
constexpr int NPRE = NWG - NPULL;
constexpr int HU = CFG_HU, PU = 2 * HU;
constexpr unsigned BLK = 16384;
constexpr unsigned PPB = 16 / PU;
constexpr unsigned XKK = 32 * BLK, WKK = 128 * BLK;
constexpr unsigned XBYTES = 8 * XKK, SLOT = XBYTES + 8 * WKK;
constexpr size_t STG = 1u << 20;
constexpr size_t WS_NEED = STG + 2 * (size_t)NRANK * SLOT;
constexpr unsigned OUT_BYTES = (unsigned)M * N * 2;
constexpr unsigned ROWB = N * 2;
constexpr unsigned C_CTR = 0, C_TMO = 128, C_PSH = 256, C_S1 = 384, C_G = 1024, C_TK = 3072, C_ARR = 4096, C_YD = 16384, C_END = 32768, C_STAMP = 65536;
constexpr int R0 = (2 * CPR - OWNH) < RH ? (2 * CPR - OWNH) : RH;
constexpr int C0 = 32 + R0 + NRANK * OWNH;
constexpr int FK = 2;
constexpr int N0 = FK * C0, NIMG = 8 * (32 + 128);
constexpr int NPUSH = CFG_NPUSH;
static_assert(N0 <= NPRE && NPRE + NIMG - N0 <= 8 * NPRE, "pre-pass list");
constexpr int PW = CFG_PW;
constexpr int WPL = (NPULL / 7) * PW;
constexpr unsigned long long DEADLINE = 150000000ull;
constexpr int LDS_BYTES = 8 * BLK + 256;
static_assert(CPR % 2 == 0 && 2 * CPR >= OWNH && GPX * 8 + NPULL <= NWG && NPULL % 14 == 0 && NROUND <= 4, "config");
static_assert(C_ARR + (NROUND * 8 + 1) * 128 <= C_YD, "ctrl");

__host__ __device__ constexpr int nh_of(int r) { return ((r + 1) * 2 * CPR < NH ? (r + 1) * 2 * CPR : NH) - r * 2 * CPR; }
__host__ __device__ constexpr unsigned gsz_of(int r) { return ((r == 0 ? 32 : 0) + nh_of(r)) * PPB; }
__host__ __device__ constexpr unsigned rstart_of(int r) { return r == 0 ? 0u : rstart_of(r - 1) + 8 * gsz_of(r - 1); }
constexpr unsigned NP0 = NPUSH * PPB;
static_assert(gsz_of(0) >= NP0 + WPL, "every puller wave starts inside group 0");
constexpr unsigned NPI = rstart_of(NROUND);
constexpr unsigned YCH = OWNH * 16;
constexpr unsigned YPT = 256 * YCH / (64 * PU);
constexpr unsigned NPT = NPI + 16 * YPT;

struct Params { const u16* x; const u16* w; char* ws[NRANK]; char* out[NRANK]; int rank; };

#define RLX_AGENT __ATOMIC_RELAXED, __HIP_MEMORY_SCOPE_AGENT
#define RLX_SYS __ATOMIC_RELAXED, __HIP_MEMORY_SCOPE_SYSTEM
#define AUX_SYS 17
#define AUX_DEV 16

__device__ __forceinline__ unsigned rfl(unsigned v) { return (unsigned)__builtin_amdgcn_readfirstlane((int)v); }
__device__ __forceinline__ rsrc_t mk_rsrc(const void* p, unsigned bytes) { return __builtin_amdgcn_make_buffer_rsrc((void*)p, 0, (int)bytes, 0x00020000); }
__device__ __forceinline__ int panel_of(int h, int rank) { return h < OWNH ? RH + rank * OWNH + h : h - OWNH; }

__device__ __forceinline__ bool expired(unsigned* tmo, unsigned call, unsigned long long t0) {
    if (rfl(__hip_atomic_load(tmo, RLX_AGENT)) == call || __builtin_amdgcn_s_memrealtime() - t0 > DEADLINE) {
        __hip_atomic_store(tmo, call, RLX_AGENT);
        return true;
    }
    return false;
}
__device__ __forceinline__ bool wait_local(const unsigned* p, unsigned target, unsigned* tmo, unsigned call) {
    const unsigned long long t0 = __builtin_amdgcn_s_memrealtime();
    unsigned polls = 0;
    while ((int)(rfl(__hip_atomic_load(p, RLX_AGENT)) - target) < 0) {
        if ((++polls & 15u) == 0u && expired(tmo, call, t0)) return false;
        __builtin_amdgcn_s_sleep(8);
    }
    asm volatile("" ::: "memory");
    return true;
}

#define STAMP(i) do {} while (0)
#define STAMPW(i) do {} while (0)

__device__ __forceinline__ void prepass(const Params& P, char* myws, LAS unsigned char* lds, unsigned ctrv, int u) {
    const int tid = threadIdx.x, lane = tid & 63, wave = (int)rfl(tid >> 6), half = wave & 1, pair = wave >> 1;
    const int kgl = lane >> 4, ng = lane & 15, sb = lane * 16, swz = sb ^ (((sb >> 9) & 1) << 5);
    const unsigned wlo = (unsigned)((half * 4 + kgl) * 8 * N + ng * 8) * 2u, xlo = (unsigned)((swz >> 6) * KS + half * 32) * 2u + (unsigned)(swz & 63);
    LAS unsigned char* lb = lds + pair * BLK + half * 1024;
    const unsigned ioff = (unsigned)((ng >> 2) * 4096 + (ng & 3) * 256 + ((kgl * 16) ^ (((ng >> 1) & 1) << 5)));
    LAS unsigned* cnt = (LAS unsigned*)(lds + 8 * BLK);
    char* const wsl = P.ws[lane & 7];
    const unsigned long long tgo = __builtin_amdgcn_s_memrealtime() + CFG_DELAY;
    for (int i = pair * NPRE + u; i < NPRE + NIMG - N0; i += 4 * NPRE) {
        int kk, img;
        if (i < NPRE + (8 - FK) * C0) { const int v = i < NPRE ? i : i - NPRE + N0; if (i >= N0 && i < NPRE) continue;
            kk = v / C0; const int bi = v % C0; img = bi < 32 + R0 ? bi : bi + (RH - R0); }
        else { const int v = i - (NPRE + (8 - FK) * C0); kk = v / (RH - R0 > 0 ? RH - R0 : 1); img = 32 + R0 + v % (RH - R0 > 0 ? RH - R0 : 1); }
        if (i >= NPRE) while (__builtin_amdgcn_s_memrealtime() < tgo + (unsigned)((i - NPRE) * CFG_SPAN / (NIMG - N0))) __builtin_amdgcn_s_sleep(4);
        const bool isw = img >= 32;
        const unsigned so = isw ? wlo + (unsigned)(kk * 64 * N + (img - 32) * 128) * 2u : xlo + (unsigned)(img * 128 * KS + kk * 64) * 2u;
        const unsigned sstep = isw ? ROWB : 16 * KS * 2;
        const unsigned dbase = isw ? XBYTES + (unsigned)kk * WKK + (unsigned)(img - 32) * BLK : (unsigned)kk * XKK + (unsigned)img * BLK;
        const rsrc_t ri = mk_rsrc(isw ? P.w : P.x, isw ? KS * N * 2 : M * KS * 2);
        u32x4 in[8];
#pragma unroll
        for (int j = 0; j < 8; ++j) in[j] = __builtin_amdgcn_raw_buffer_load_b128(ri, so + j * sstep, 0, 0);
        if (isw) {
#pragma unroll
            for (int v8 = 0; v8 < 8; ++v8) {
                u32x4 o;
#pragma unroll
                for (int e = 0; e < 4; ++e) o[e] = __builtin_amdgcn_perm(in[2 * e + 1][v8 >> 1], in[2 * e][v8 >> 1], (v8 & 1) ? 0x07060302u : 0x05040100u);
                *(LAS u32x4*)(lb + ioff + (v8 >> 2) * 2048 + (v8 & 3) * 64) = o;
            }
            asm volatile("s_waitcnt lgkmcnt(0)" ::: "memory");
#pragma unroll
            for (int j = 0; j < 8; ++j) in[j] = *(LAS u32x4*)(lb + j * 2048 + lane * 16);
        }
        const unsigned parity = (rfl(ctrv) / NWG + 1u) & 1u;
        const rsrc_t rs = mk_rsrc(myws + STG + (size_t)(parity * NRANK + P.rank) * SLOT, SLOT);
#pragma unroll
        for (int j = 0; j < 8; ++j) __builtin_amdgcn_raw_buffer_store_b128(in[j], rs, dbase + (unsigned)(j * 2048 + half * 1024 + lane * 16), 0, AUX_SYS);
        if (i < NPRE) {
            asm volatile("s_waitcnt vmcnt(0)" ::: "memory");
            unsigned second = 0;
            if (lane == 0) second = __hip_atomic_fetch_add(cnt, 1u, __ATOMIC_RELAXED, __HIP_MEMORY_SCOPE_WORKGROUP);
            if (rfl(second) == 1u && i >= NPUSH && lane < NRANK && lane != P.rank && (img < 32 + RH || (img - 32 - RH) / OWNH == lane))
                __hip_atomic_fetch_add((unsigned*)(wsl + C_G + (P.rank * 2 + kk) * 128), 1u, RLX_SYS);
            STAMPW(40);
        }
    }
    asm volatile("s_waitcnt vmcnt(0)" ::: "memory");
    __syncthreads();
    if (tid == 0) __hip_atomic_fetch_add((unsigned*)(myws + C_S1), 1u, RLX_SYS);
}

__device__ __forceinline__ unsigned in_piece(unsigned n, int rank) {
    int r = 0;
#pragma unroll
    for (int i = 1; i < NROUND; ++i) if (n >= rstart_of(i)) r = i;
    unsigned kk = 0, pi = 0;
#pragma unroll
    for (int i = 0; i < NROUND; ++i) if (r == i) { const unsigned q = n - rstart_of(i); kk = q / gsz_of(i); pi = q % gsz_of(i); }
    if (r == 0 && pi < 32 * PPB) return kk * XKK + pi * (PU * 1024);
    if (r == 0) pi -= 32 * PPB;
    return XBYTES + kk * WKK + (unsigned)panel_of(r * 2 * CPR + (int)(pi / PPB), rank) * BLK + (pi % PPB) * (PU * 1024);
}
__device__ __forceinline__ unsigned group_end(int G) {
    unsigned e = 0;
#pragma unroll
    for (int i = 0; i < NROUND; ++i) if ((G >> 3) == i) e = rstart_of(i) + (unsigned)((G & 7) + 1) * gsz_of(i);
    return e;
}

#define WAIT_V(n) asm volatile("s_waitcnt vmcnt(" #n ")" ::: "memory")
#define WAIT_L(n) asm volatile("s_waitcnt lgkmcnt(" #n ")" ::: "memory")
#define RLX_WG __ATOMIC_RELAXED, __HIP_MEMORY_SCOPE_WORKGROUP
__device__ __forceinline__ void puller(const Params& P, char* myws, LAS unsigned* lctr, unsigned ctrv, int link, int idx) {
    const int tid = threadIdx.x, lane = tid & 63, wave = (int)rfl(tid >> 6);
    const unsigned p = (unsigned)(P.rank + 1 + link) & 7u;
    char* pws = P.ws[p];
    unsigned* tmo = (unsigned*)(myws + C_TMO);
    if (NPUSH > 0 && wave >= 4) {
        constexpr int PJ = NPUSH * 16 / ((NPULL / 7) * 4);
        static_assert(PJ * (NPULL / 7) * 4 == NPUSH * 16 && PJ % 2 == 0 && NPUSH <= 32, "push split");
        const rsrc_t rx = mk_rsrc(P.x, M * KS * 2);
        const int sb = lane * 16, swz = sb ^ (((sb >> 9) & 1) << 5), c0 = (idx * 4 + wave - 4) * PJ;
        u32x4 in[PJ + 1];
        const unsigned so = (unsigned)((c0 * 8 + (swz >> 6)) * KS) * 2u + (unsigned)(swz & 63);
#pragma unroll
        for (int j = 0; j < PJ; ++j) in[j] = __builtin_amdgcn_raw_buffer_load_b128(rx, so + (j >> 1) * (16 * KS * 2) + (j & 1) * 64, 0, 0);
        const rsrc_t rp = mk_rsrc(pws + STG + (size_t)(((rfl(ctrv) / NWG + 1u) & 1u) * NRANK + P.rank) * SLOT, XKK);
#pragma unroll
        for (int j = 0; j < PJ; ++j) {
            if (j >= CFG_PWIN) asm volatile("s_waitcnt vmcnt(%0)" :: "n"(CFG_PWIN - 1) : "memory");
            __builtin_amdgcn_raw_buffer_store_b128(in[j], rp, (unsigned)((c0 + j) * 1024 + sb), 0, AUX_SYS);
        }
        WAIT_V(0);
        if (lane == 0 && __hip_atomic_fetch_add(lctr + 61, 1u, RLX_WG) == 3u) { __hip_atomic_fetch_add((unsigned*)(pws + C_PSH), 1u, RLX_SYS); STAMPW(40); }
    }
    const unsigned call = rfl(ctrv) / NWG + 1u, parity = call & 1u;
    if (wave == 7) {
        for (unsigned st = 1; st <= 3; ++st) {
            const unsigned* fw = (const unsigned*)(st == 1 ? myws + C_G + (p * 2 + 1) * 128 : pws + (st == 2 ? C_S1 : C_YD));
            const unsigned ft = call * (unsigned)(st == 1 ? 32 + nh_of(0) : st == 2 ? NPRE : 16 * ((OWNH + 1) / 2));
            const unsigned long long t0 = __builtin_amdgcn_s_memrealtime();
            while ((int)(rfl(__hip_atomic_load(fw, RLX_SYS)) - ft) < 0) {
                if (expired(tmo, call, t0)) { st = 4; break; }
                __builtin_amdgcn_s_sleep(64);
            }
            if (lane == 0) __hip_atomic_store(lctr + 63, st, RLX_WG);
            STAMPW(40 + st);
        }
    }
    if (wave >= PW) return;
    static_assert(PW <= 7, "wave 7 is taken");
    const size_t soff = STG + (size_t)(parity * NRANK + p) * SLOT;
    char* const srcI = pws + soff; char* const dstI = myws + soff;
    char* const srcO = P.out[p]; char* const dstO = P.out[P.rank];
    const unsigned ycol = (unsigned)(RH + p * OWNH) * 256u;
    const rsrc_t rctl = mk_rsrc(myws, C_END);
    const unsigned tkoff = lane == 0 ? C_TK + (unsigned)link * 128u : C_END;
    const unsigned tbase = (call - 1u) * (NPT - NP0) - WPL - NP0;
    int gsig = 0;
    auto signal = [&](unsigned n) {
        while (gsig < NROUND * 8 && group_end(gsig) <= n) {
            if (lane == 0 && __hip_atomic_fetch_add(lctr + gsig, 1u, RLX_WG) == PW - 1u) { __hip_atomic_fetch_add((unsigned*)(myws + C_ARR + gsig * 128), 1u, RLX_AGENT); STAMPW(2 + gsig); }
            ++gsig;
        }
    };
    auto decode = [&](unsigned n, unsigned (&off)[PU]) {
        if (n < NPI) { const unsigned o = in_piece(n, P.rank);
#pragma unroll
            for (int u = 0; u < PU; ++u) off[u] = o + u * 1024 + lane * 16; }
        else { const unsigned q = n - NPI;
#pragma unroll
            for (int u = 0; u < PU; ++u) { const unsigned ci = (q * PU + u) * 64 + lane; off[u] = (ci / YCH) * ROWB + ycol + (ci % YCH) * 16; } }
    };
    unsigned n = NP0 + (unsigned)idx * PW + (unsigned)wave;
    unsigned off[PU];
    decode(n, off);
    rsrc_t rs = mk_rsrc(srcI, SLOT), rd = mk_rsrc(dstI, SLOT);
    bool ok = true;
    {
        const unsigned long long t0 = __builtin_amdgcn_s_memrealtime();
        unsigned polls = 0;
        if (wave == 0) {
            while ((int)(rfl(__hip_atomic_load((const unsigned*)(myws + C_G + p * 2 * 128), RLX_SYS)) - call * (unsigned)(32 - NPUSH + nh_of(0))) < 0) {
                if ((++polls & 15u) == 0u && expired(tmo, call, t0)) { ok = false; break; }
                __builtin_amdgcn_s_sleep(1);
            }
            if (lane == 0) __hip_atomic_store(lctr + 62, ok ? 1u : 2u, RLX_WG);
        } else {
            unsigned g;
            while ((g = rfl(__hip_atomic_load(lctr + 62, RLX_WG))) == 0u) {
                if ((++polls & 1023u) == 0u && expired(tmo, call, t0)) break;
                __builtin_amdgcn_s_sleep(1);
            }
            ok = g == 1u;
        }
        asm volatile("" ::: "memory");
    }
    u32x4 A[HU], B[HU];
    unsigned tk, stage = 0;
#define LD16(dst, off, rs) asm volatile("buffer_load_dwordx4 %0, %1, %2, 0 offen sc0 sc1" : "=v"(dst) : "v"(off), "s"(rs) : "memory")
#define HEAD(off, rs) do { tk = 1u; asm volatile("buffer_atomic_add %0, %1, %2, 0 offen sc0" : "+v"(tk) : "v"(tkoff), "s"(rctl) : "memory"); \
        _Pragma("unroll") for (int u = 0; u < HU; ++u) LD16(A[u], off[u], rs); } while (0)
#define LANDED(X) do { asm volatile("s_waitcnt vmcnt(%0)" :: "n"(HU) : "memory"); \
        _Pragma("unroll") for (int u = 0; u < HU; ++u) { asm volatile("" : "+v"(X[u]) :: "memory"); if (!ok) X[u] = (u32x4){0x7fc07fc0u, 0x7fc07fc0u, 0x7fc07fc0u, 0x7fc07fc0u}; } } while (0)
    STAMP(1);
    HEAD(off, rs);
    WAIT_V(0);
    for (;;) {
        LANDED(A);
        asm volatile("" : "+v"(tk) :: "memory");
#pragma unroll
        for (int u = 0; u < HU; ++u) LD16(B[u], off[HU + u], rs);
#pragma unroll
        for (int u = 0; u < HU; ++u) __builtin_amdgcn_raw_buffer_store_b128(A[u], rd, off[u], 0, CFG_SIN);
        const unsigned nn = rfl(tk) - tbase;
        unsigned offn[PU];
        decode(nn, offn);
        const unsigned need = nn < gsz_of(0) ? 0u : nn < 2 * gsz_of(0) ? 1u : nn < NPI ? 2u : 3u;
        if (stage < need) stage = rfl(__hip_atomic_load(lctr + 63, RLX_WG));
        rs = mk_rsrc(nn < NPI ? srcI : srcO, nn < NPI ? SLOT : OUT_BYTES);
        LANDED(B);
        if (nn >= NPT) break;
        if (stage < need) {
#pragma unroll
            for (int u = 0; u < HU; ++u) __builtin_amdgcn_raw_buffer_store_b128(B[u], rd, off[HU + u], 0, CFG_SIN);
            WAIT_V(0);
            signal(nn);
            const unsigned long long t0 = __builtin_amdgcn_s_memrealtime();
            unsigned polls = 0;
            while ((stage = rfl(__hip_atomic_load(lctr + 63, RLX_WG))) < need) {
                if ((++polls & 1023u) == 0u && expired(tmo, call, t0)) { stage = 4; break; }
                __builtin_amdgcn_s_sleep(2);
            }
        }
        HEAD(offn, rs);
#pragma unroll
        for (int u = 0; u < HU; ++u) __builtin_amdgcn_raw_buffer_store_b128(B[u], rd, off[HU + u], 0, CFG_SIN);
        if (stage == 4u) ok = false;
        signal(n);
        n = nn;
        rd = mk_rsrc(n < NPI ? dstI : dstO, n < NPI ? SLOT : OUT_BYTES);
#pragma unroll
        for (int u = 0; u < PU; ++u) off[u] = offn[u];
    }
#pragma unroll
    for (int u = 0; u < HU; ++u) __builtin_amdgcn_raw_buffer_store_b128(B[u], rd, off[HU + u], 0, CFG_SIN);
    WAIT_V(0);
    signal(NPT);
#undef LD16
#undef HEAD
#undef LANDED
}

__device__ __forceinline__ unsigned cvt_pk_bf16(float lo, float hi) { unsigned r; asm volatile("v_cvt_pk_bf16_f32 %0, %1, %2" : "=v"(r) : "v"(lo), "v"(hi)); return r; }
__device__ __forceinline__ float gelu(float y) {
    const float k1 = -2.0f * 0.7978845608028654f * 1.4426950408889634f, k3 = k1 * 0.044715f;
    const float e = __builtin_amdgcn_exp2f(y * __builtin_fmaf(y * y, k3, k1));
    return y * __builtin_amdgcn_rcpf(1.0f + e);
}

__device__ __forceinline__ void gemm(const Params& P, char* myws, LAS unsigned char* lds, unsigned call, int xcd, int j) {
    const int tid = threadIdx.x, wid = (int)rfl(tid >> 6), lane = tid & 63, wr = wid >> 2, wc = wid & 3, fr = lane & 15, fq = lane >> 4;
    const int pm = (xcd & 3) * 4 + (j & 3), cc = (xcd >> 2) * (CPR / 2) + (j >> 2);
    if (cc >= NCT) return;
    unsigned* tmo = (unsigned*)(myws + C_TMO);
    const char* stg = myws + STG + (size_t)(call & 1u) * NRANK * SLOT;
    const char* baseA = stg + (size_t)pm * 2 * BLK;
    const rsrc_t rctl = mk_rsrc(myws, C_END);
    const unsigned voff = (unsigned)tid * 16u, ldsw = (unsigned)wid * 1024u;
    const unsigned atarget = call * NPULL;
    int aoff, boff;
    { const int ob = fr * 64 + fq * 16, sw = ob ^ (((ob >> 9) & 1) << 5); aoff = wr * 8192 + sw; boff = wc * 4096 + sw; }
    bool ok = true;
#define SA(b, h) (((b) * 2 + (h)) * BLK)
#define SB(b, h) ((4 + (b) * 2 + (h)) * BLK)
#define STAGE(bufoff, gbase) do { _Pragma("unroll") for (int _i = 0; _i < 2; ++_i) \
        __builtin_amdgcn_global_load_lds((const unsigned*)((const char*)(gbase) + (voff + _i * 8192u)), (LAS unsigned*)(lds + (bufoff) + ldsw + _i * 8192), 16, 0, AUX_DEV); } while (0)
#define LDA(dst, b, h) do { _Pragma("unroll") for (int m = 0; m < 4; ++m) _Pragma("unroll") for (int k = 0; k < 2; ++k) dst[m][k] = *(const LAS bf16x8*)(lds + SA(b, h) + aoff + m * 2048 + k * 1024); } while (0)
#define LDB(dst, b, h) do { _Pragma("unroll") for (int n = 0; n < 2; ++n) _Pragma("unroll") for (int k = 0; k < 2; ++k) dst[n][k] = *(const LAS bf16x8*)(lds + SB(b, h) + boff + n * 2048 + k * 1024); } while (0)
#define MMA(ai, bj, At, Bt) do { __builtin_amdgcn_s_setprio(1); _Pragma("unroll") for (int m = 0; m < 4; ++m) _Pragma("unroll") for (int n = 0; n < 2; ++n) _Pragma("unroll") for (int k = 0; k < 2; ++k) \
        acc[ai][bj][m][n] = __builtin_amdgcn_mfma_f32_16x16x32_bf16(Bt[n][k], At[m][k], acc[ai][bj][m][n], 0, 0, 0); __builtin_amdgcn_s_setprio(0); } while (0)
#define BAR __builtin_amdgcn_s_barrier()
#define SCHED __builtin_amdgcn_sched_barrier(0)
    f32x4 acc[2][2][4][2];
#pragma unroll
    for (int a = 0; a < 2; ++a)
#pragma unroll
        for (int b = 0; b < 2; ++b)
#pragma unroll
            for (int m = 0; m < 4; ++m)
#pragma unroll
                for (int n = 0; n < 2; ++n) acc[a][b][m][n] = (f32x4){0.f, 0.f, 0.f, 0.f};
    bf16x8 At[4][2], B0[2][2], B1[2][2];
    if (wid == 0) {
        ok = wait_local((const unsigned*)(myws + C_ARR), atarget, tmo, call);
        const unsigned* fw = (const unsigned*)(myws + (lane == 0 || NPUSH == 0 ? C_S1 : C_PSH));
        const unsigned ft = call * (lane == 0 || NPUSH == 0 ? NPRE : NPULL);
        const unsigned long long t0 = __builtin_amdgcn_s_memrealtime();
        while (!__all((int)(__hip_atomic_load(fw, RLX_SYS) - ft) >= 0)) {
            if (expired(tmo, call, t0)) { ok = false; break; }
            __builtin_amdgcn_s_sleep(16);
        }
    }
    asm volatile("" ::: "memory"); BAR; asm volatile("" ::: "memory");
    STAMP(2);
    int c = cc;
    int h1 = 2 * c + 1 < NH ? 2 * c + 1 : 2 * c;
    const char* cB0 = stg + XBYTES + (size_t)panel_of(2 * c, P.rank) * BLK;
    const char* cB1 = stg + XBYTES + (size_t)panel_of(h1, P.rank) * BLK;
    STAGE(SB(0, 0), cB0); STAGE(SA(0, 0), baseA); STAGE(SB(0, 1), cB1); STAGE(SA(0, 1), baseA + BLK);
    if (wr == 1) BAR;
    WAIT_V(4); BAR;
    STAGE(SB(1, 0), cB0 + SLOT); STAGE(SA(1, 0), baseA + SLOT); STAGE(SB(1, 1), cB1 + SLOT);
    WAIT_V(6); BAR;
    unsigned pollv = 0;
    for (int round = 0;; ++round) {
        const int cn = c + CPR;
        const bool has_next = cn < NCT;
        const int h1n = 2 * cn + 1 < NH ? 2 * cn + 1 : 2 * cn;
        const char* nB0 = has_next ? stg + XBYTES + (size_t)panel_of(2 * cn, P.rank) * BLK : cB0;
        const char* nB1 = has_next ? stg + XBYTES + (size_t)panel_of(h1n, P.rank) * BLK : cB1;
        for (int t = 0; t < 64; t += 2) {
            const bool last = t == 62;
            const int t2 = (t + 2) & 63;
            const char* a1 = baseA + (size_t)((unsigned)((t & 7) + 1) * SLOT + (unsigned)(t >> 3) * XKK);
            const char* a2 = baseA + (size_t)((unsigned)(t2 & 7) * SLOT + (unsigned)(t2 >> 3) * XKK);
            const unsigned kb = (unsigned)(t2 & 7) * SLOT + (unsigned)(t2 >> 3) * WKK;
            const char* b20 = (last ? nB0 : cB0) + kb; const char* b21 = (last ? nB1 : cB1) + kb;
            const unsigned gnext = (unsigned)(round * 8 + (t >> 3) + 1);
            if (wid == 0 && (t & 7) == 6 && (has_next || !last)) {
                if ((int)(rfl(pollv) - atarget) < 0) ok = wait_local((const unsigned*)(myws + C_ARR + gnext * 128), atarget, tmo, call) && ok;
            }
            LDB(B0, 0, 0); SCHED; LDA(At, 0, 0); STAGE(SA(1, 1), a1 + BLK);
            WAIT_L(8); BAR; WAIT_L(0); MMA(0, 0, At, B0); BAR; SCHED;
            LDB(B1, 0, 1); STAGE(SB(0, 0), b20);
            BAR; WAIT_L(0); MMA(0, 1, At, B1); BAR;
            LDA(At, 0, 1); STAGE(SA(0, 0), a2);
            BAR; WAIT_L(0); MMA(1, 0, At, B0); BAR; SCHED;
            STAGE(SB(0, 1), b21);
            WAIT_V(6); BAR; MMA(1, 1, At, B1); BAR;
            if (wid == 0 && (t & 7) == 4) asm volatile("buffer_load_dword %0, off, %1, %2 sc1" : "+v"(pollv) : "s"(rctl), "s"(C_ARR + gnext * 128) : "memory");
            LDB(B0, 1, 0); SCHED; LDA(At, 1, 0); STAGE(SA(0, 1), a2 + BLK);
            WAIT_L(8); BAR; WAIT_L(0); MMA(0, 0, At, B0); BAR; SCHED;
            LDB(B1, 1, 1); STAGE(SB(1, 0), b20 + SLOT);
            BAR; WAIT_L(0); MMA(0, 1, At, B1); BAR;
            LDA(At, 1, 1); STAGE(SA(1, 0), a2 + SLOT);
            BAR; WAIT_L(0); MMA(1, 0, At, B0); BAR; SCHED;
            STAGE(SB(1, 1), b21 + SLOT);
            asm volatile("s_waitcnt vmcnt(6)" : "+v"(pollv) :: "memory"); BAR; MMA(1, 1, At, B1); BAR;
        }
        STAMP(3 + 2 * round);
        {
            const rsrc_t ro = mk_rsrc(P.out[P.rank], OUT_BYTES);
            const unsigned rowoff = (unsigned)(pm * 256 + wr * 64 + fr) * ROWB + (unsigned)(wc * 32 + fq * 8) * 2u;
#pragma unroll
            for (int bj = 0; bj < 2; ++bj) {
                const unsigned colb = (2 * c + bj < NH) ? (unsigned)panel_of(2 * c + bj, P.rank) * 256u : OUT_BYTES;
#pragma unroll
                for (int ai = 0; ai < 2; ++ai)
#pragma unroll
                    for (int m = 0; m < 4; ++m) {
                        const f32x4 v0 = acc[ai][bj][m][0], v1 = acc[ai][bj][m][1];
                        u32x4 o;
                        o.x = cvt_pk_bf16(gelu(v0[0]), gelu(v0[1])); o.y = cvt_pk_bf16(gelu(v0[2]), gelu(v0[3]));
                        o.z = cvt_pk_bf16(gelu(v1[0]), gelu(v1[1])); o.w = cvt_pk_bf16(gelu(v1[2]), gelu(v1[3]));
                        if (!ok) o = (u32x4){0x7fc07fc0u, 0x7fc07fc0u, 0x7fc07fc0u, 0x7fc07fc0u};
                        if (2 * c < OWNH) __builtin_amdgcn_raw_buffer_store_b128(o, ro, rowoff + colb + (unsigned)(ai * 128 + m * 16) * ROWB, 0, AUX_SYS);
                        else __builtin_amdgcn_raw_buffer_store_b128(o, ro, rowoff + colb + (unsigned)(ai * 128 + m * 16) * ROWB, 0, CFG_SO);
                    }
            }
            if (2 * c < OWNH) {
                WAIT_V(0);
                if (lane == 0 && __hip_atomic_fetch_add((LAS unsigned*)(lds + 8 * BLK) + 1, 1u, RLX_WG) == 7u) __hip_atomic_fetch_add((unsigned*)(myws + C_YD), 1u, RLX_SYS);
            }
        }
        STAMP(4 + 2 * round);
        if (!has_next) break;
#pragma unroll
        for (int a = 0; a < 2; ++a)
#pragma unroll
            for (int b = 0; b < 2; ++b)
#pragma unroll
                for (int m = 0; m < 4; ++m)
#pragma unroll
                    for (int n = 0; n < 2; ++n) acc[a][b][m][n] = (f32x4){0.f, 0.f, 0.f, 0.f};
        c = cn; cB0 = nB0; cB1 = nB1;
    }
    WAIT_V(0);
    if (wr == 0) BAR;
#undef SA
#undef SB
#undef STAGE
#undef LDA
#undef LDB
#undef MMA
}

__global__ __launch_bounds__(512, 2) void fused_gemm_allreduce(Params P) {
    extern __shared__ __attribute__((aligned(16))) unsigned char shm[];
    char* myws = P.ws[P.rank];
    const int b = blockIdx.x, xcd = b & 7, j = b >> 3;
    STAMP(0);
    const unsigned ctrv = __hip_atomic_load((unsigned*)(myws + C_CTR), RLX_AGENT);
    if (threadIdx.x < 64) ((LAS unsigned*)shm)[8 * BLK / 4 + threadIdx.x] = 0;
    WAIT_L(0); __builtin_amdgcn_s_barrier(); asm volatile("" ::: "memory");
    const int pi = ((j - GPX) * 4 + (xcd & 3)) * 2 + (xcd >> 2);
    if (j >= GPX && pi < NPULL) puller(P, myws, (LAS unsigned*)shm + 8 * BLK / 4, ctrv, (pi >> 1) % 7, (pi >> 1) / 7 * 2 + (pi & 1));
    else {
        prepass(P, myws, (LAS unsigned char*)shm, ctrv, j < GPX ? b : GPX * 8 + pi - NPULL);
        STAMP(1);
        if (j < GPX) gemm(P, myws, (LAS unsigned char*)shm, rfl(ctrv) / NWG + 1u, xcd, j);
    }
    __syncthreads();
    STAMP(63);
    if (threadIdx.x == 0) __hip_atomic_fetch_add((unsigned*)(myws + C_CTR), 1u, RLX_AGENT);
}
}

extern "C" void kernel_launch(int rank, int nranks, ncclComm_t comm, hipStream_t stream,
                              void* const* d_in, const int* in_sizes, int n_in,
                              void* d_out, int out_size, const int* in_dtypes, int out_dtype,
                              void* const* d_out_all, void* const* d_ws_all, size_t ws_bytes) {
    if (nranks != NRANK || (unsigned)rank >= (unsigned)NRANK || n_in < 2 || in_sizes[0] != M * KS || in_sizes[1] != KS * N || out_size != M * N || ws_bytes < WS_NEED) {
        fprintf(stderr, "kernel_launch: unexpected problem geometry\n");
        return;
    }
    static bool attr_set = false;
    if (!attr_set) { HIP_CHECK(hipFuncSetAttribute((const void*)fused_gemm_allreduce, hipFuncAttributeMaxDynamicSharedMemorySize, LDS_BYTES)); attr_set = true; }
    Params P;
    P.x = (const u16*)d_in[0]; P.w = (const u16*)d_in[1]; P.rank = rank;
    for (int p = 0; p < NRANK; ++p) { P.ws[p] = (char*)d_ws_all[p]; P.out[p] = (char*)d_out_all[p]; }
    P.out[rank] = (char*)d_out;
    hipLaunchKernelGGL(fused_gemm_allreduce, dim3(NWG), dim3(512), LDS_BYTES, stream, P);
    HIP_CHECK(hipGetLastError());
}
```

```cpp
#include <hip/hip_runtime.h>
#include <rccl/rccl.h>
#include <cstdio>

#define HIP_CHECK(expr) do { hipError_t _e = (expr); if (_e != hipSuccess) { \
    fprintf(stderr, "HIP error %s at %s:%d\n", hipGetErrorString(_e), __FILE__, __LINE__); \
    return; } } while (0)

namespace {
#define LAS __attribute__((address_space(3)))
typedef unsigned short u16;
typedef short bf16x8 __attribute__((ext_vector_type(8)));
typedef float f32x4 __attribute__((ext_vector_type(4)));
typedef unsigned u32x4 __attribute__((ext_vector_type(4)));
typedef __amdgpu_buffer_rsrc_t rsrc_t;

#define CFG_OWNH 6
#define CFG_GPX 29
#define CFG_PPX 3
#define CFG_SIN 16
#define CFG_SO 17
#define CFG_PJ 4
#define CFG_DELAY 350
#define CFG_SPAN 1200
#define CFG_HSLEEP 64
#define CFG_PWIN 3
#define CFG_W3 2
#define CFG_W3M 0
#define CFG_EW 0
#define CFG_DED 1

constexpr int NRANK = 8, M = 4096, N = 16384, KS = 512;
constexpr int NWG = 256;
constexpr int OWNH = CFG_OWNH;
constexpr int RH = 128 - NRANK * OWNH;
constexpr int NH = OWNH + RH;
constexpr int NCT = (NH + 1) / 2;
constexpr int GPX = CFG_GPX;
constexpr int TPX = 2 * NCT;
constexpr int NROUND = (TPX + GPX - 1) / GPX;
constexpr int NPULL = 8 * CFG_PPX;
constexpr int NPRE = NWG - NPULL;
constexpr int PU = 4;
constexpr unsigned BLK = 16384;
constexpr unsigned PPB = 16 / PU;
constexpr unsigned XKK = 32 * BLK, WKK = 128 * BLK;
constexpr unsigned XBYTES = 8 * XKK, SLOT = XBYTES + 8 * WKK;
constexpr size_t STG = 1u << 20;
constexpr size_t WS_NEED = STG + 2 * (size_t)NRANK * SLOT;
constexpr unsigned OUT_BYTES = (unsigned)M * N * 2;
constexpr unsigned ROWB = N * 2;
constexpr unsigned C_CTR = 0, C_TMO = 128, C_YD = 256, C_S1 = 384, C_F = 512, C_G = 1024, C_TK = 3072, C_ARR = 4096, C_END = 65536, C_STAMP = 65536;
__host__ __device__ constexpr int pend_of(int r) { return r < 0 ? 0 : (((r + 1) * GPX + 3) / 4 * 4 < NH ? ((r + 1) * GPX + 3) / 4 * 4 : NH); }
__host__ __device__ constexpr int nh_of(int r) { return pend_of(r) - pend_of(r - 1); }
__host__ __device__ constexpr unsigned gsz_of(int r) { return ((r == 0 ? 32 : 0) + nh_of(r)) * PPB; }
__host__ __device__ constexpr unsigned rstart_of(int r) { return r == 0 ? 0u : rstart_of(r - 1) + 8 * gsz_of(r - 1); }
constexpr int R0 = nh_of(0) - OWNH;
constexpr int C0 = 32 + R0 + NRANK * OWNH;
constexpr int FK = 2;
constexpr int N0 = FK * C0, NIMG = 8 * (32 + 128);
constexpr int PJ = CFG_PJ;
constexpr int NPUSH = PJ * NPULL / 16;
constexpr unsigned long long DEADLINE = 150000000ull;
constexpr int LDS_BYTES = 8 * BLK + 256;
constexpr unsigned NP0 = NPUSH * PPB;
constexpr unsigned NPI = rstart_of(NROUND);
constexpr unsigned YCH = OWNH * 16;
constexpr unsigned NPT = NPI + M * YCH / (64 * PU);
static_assert(N0 <= NPRE && NPRE + NIMG - N0 <= 8 * NPRE, "pre-pass list");
static_assert(NPUSH * 16 == PJ * NPULL && PJ % 2 == 0 && PJ > 0 && NPUSH <= 32, "push split");
static_assert(GPX / 4 * 4 >= OWNH + 1 && R0 > 0 && R0 < RH && GPX * 8 + NPULL <= NWG && NROUND <= 4 && nh_of(NROUND - 1) > 0, "config");
static_assert(C_ARR + (NROUND * 8 + 1) * 1024 <= C_END, "ctrl");
static_assert(32 * PPB >= NP0 + NPULL, "every puller wave starts on x of K tile 0");

template <int V> struct Int { static constexpr int value = V; };
struct Params { const u16* x; const u16* w; char* myws; char* myout; int rank; char* ws[NRANK]; char* out[NRANK]; };

#define RLX_AGENT __ATOMIC_RELAXED, __HIP_MEMORY_SCOPE_AGENT
#define RLX_SYS __ATOMIC_RELAXED, __HIP_MEMORY_SCOPE_SYSTEM
#define AUX_SYS 17
#define AUX_DEV 16

__device__ __forceinline__ unsigned rfl(unsigned v) { return (unsigned)__builtin_amdgcn_readfirstlane((int)v); }
__device__ __forceinline__ rsrc_t mk_rsrc(const void* p, unsigned bytes) { return __builtin_amdgcn_make_buffer_rsrc((void*)p, 0, (int)bytes, 0x00020000); }
__device__ __forceinline__ int panel_of(int h, int rank) { return h < OWNH ? RH + rank * OWNH + h : h - OWNH; }

__device__ __forceinline__ bool expired(unsigned* tmo, unsigned call, unsigned long long t0) {
    if (rfl(__hip_atomic_load(tmo, RLX_AGENT)) == call || __builtin_amdgcn_s_memrealtime() - t0 > DEADLINE) {
        __hip_atomic_store(tmo, call, RLX_AGENT);
        return true;
    }
    return false;
}
__device__ __forceinline__ bool wait_local(const unsigned* p, unsigned target, unsigned* tmo, unsigned call) {
    const unsigned long long t0 = __builtin_amdgcn_s_memrealtime();
    unsigned polls = 0;
    while ((int)(rfl(__hip_atomic_load(p, RLX_AGENT)) - target) < 0) {
        if ((++polls & 15u) == 0u && expired(tmo, call, t0)) return false;
        __builtin_amdgcn_s_sleep(8);
    }
    asm volatile("" ::: "memory");
    return true;
}

__device__ __forceinline__ void count_and_flag(const Params& P, char* myws, unsigned cnt, unsigned total, unsigned word, unsigned call, int lane) {
    unsigned old = 0;
    if (lane == 0) old = __hip_atomic_fetch_add((unsigned*)(myws + cnt), 1u, RLX_AGENT);
    if (rfl(old) + 1u == call * total && lane < NRANK) __hip_atomic_store((unsigned*)(P.ws[lane] + C_F) + word, call, RLX_SYS);
}

#define STAMPC(i) do {} while (0)
#define STAMP(i) do {} while (0)
#define STAMPW(i) do {} while (0)

__device__ __forceinline__ void prepass(const Params& P, char* myws, LAS unsigned char* lds, unsigned ctrv, int u) {
    const int tid = threadIdx.x, lane = tid & 63, wave = (int)rfl(tid >> 6), half = wave & 1, pair = wave >> 1;
    const int kgl = lane >> 4, ng = lane & 15, sb = lane * 16, swz = sb ^ (((sb >> 9) & 1) << 5);
    const unsigned wlo = (unsigned)((half * 4 + kgl) * 8 * N + ng * 8) * 2u, xlo = (unsigned)((swz >> 6) * KS + half * 32) * 2u + (unsigned)(swz & 63);
    LAS unsigned char* lb = lds + pair * BLK + half * 1024;
    const unsigned ioff = (unsigned)((ng >> 2) * 4096 + (ng & 3) * 256 + ((kgl * 16) ^ (((ng >> 1) & 1) << 5)));
    LAS unsigned* cnt = (LAS unsigned*)(lds + 8 * BLK);
    char* const wsl = P.ws[lane & 7];
    const unsigned long long tgo = __builtin_amdgcn_s_memrealtime() + CFG_DELAY;
    for (int i = pair * NPRE + u; i < NPRE + NIMG - N0; i += 4 * NPRE) {
        int kk, img;
        if (i < NPRE + (8 - FK) * C0) { const int v = i < NPRE ? i : i - NPRE + N0; if (i >= N0 && i < NPRE) continue;
            kk = v / C0; const int bi = v % C0; img = bi < 32 + R0 ? bi : bi + (RH - R0); }
        else { const int v = i - (NPRE + (8 - FK) * C0); kk = v / (RH - R0); img = 32 + R0 + v % (RH - R0); }
        if (i >= NPRE) while (__builtin_amdgcn_s_memrealtime() < tgo + (unsigned)((i - NPRE) * CFG_SPAN / (NIMG - N0))) __builtin_amdgcn_s_sleep(4);
        const bool isw = img >= 32;
        const unsigned so = isw ? wlo + (unsigned)(kk * 64 * N + (img - 32) * 128) * 2u : xlo + (unsigned)(img * 128 * KS + kk * 64) * 2u;
        const unsigned sstep = isw ? ROWB : 16 * KS * 2;
        const unsigned dbase = isw ? XBYTES + (unsigned)kk * WKK + (unsigned)(img - 32) * BLK : (unsigned)kk * XKK + (unsigned)img * BLK;
        const rsrc_t ri = mk_rsrc(isw ? P.w : P.x, isw ? KS * N * 2 : M * KS * 2);
        u32x4 in[8];
#pragma unroll
        for (int j = 0; j < 8; ++j) in[j] = __builtin_amdgcn_raw_buffer_load_b128(ri, so + j * sstep, 0, 0);
        if (isw) {
#pragma unroll
            for (int v8 = 0; v8 < 8; ++v8) {
                u32x4 o;
#pragma unroll
                for (int e = 0; e < 4; ++e) o[e] = __builtin_amdgcn_perm(in[2 * e + 1][v8 >> 1], in[2 * e][v8 >> 1], (v8 & 1) ? 0x07060302u : 0x05040100u);
                *(LAS u32x4*)(lb + ioff + (v8 >> 2) * 2048 + (v8 & 3) * 64) = o;
            }
            asm volatile("s_waitcnt lgkmcnt(0)" ::: "memory");
#pragma unroll
            for (int j = 0; j < 8; ++j) in[j] = *(LAS u32x4*)(lb + j * 2048 + lane * 16);
        }
        const unsigned parity = (rfl(ctrv) / NWG + 1u) & 1u;
        const rsrc_t rs = mk_rsrc(myws + STG + (size_t)(parity * NRANK + P.rank) * SLOT, SLOT);
#pragma unroll
        for (int j = 0; j < 8; ++j) __builtin_amdgcn_raw_buffer_store_b128(in[j], rs, dbase + (unsigned)(j * 2048 + half * 1024 + lane * 16), 0, AUX_SYS);
        if (i < NPRE) {
            asm volatile("s_waitcnt vmcnt(0)" ::: "memory");
            unsigned second = 0;
            if (lane == 0) second = __hip_atomic_fetch_add(cnt, 1u, __ATOMIC_RELAXED, __HIP_MEMORY_SCOPE_WORKGROUP);
            if (rfl(second) == 1u && i >= NPUSH && lane < NRANK && lane != P.rank && (img < 32 + RH || (img - 32 - RH) / OWNH == lane))
                __hip_atomic_fetch_add((unsigned*)(wsl + C_G + (P.rank * 2 + (i < 32 ? 0 : 1)) * 128), 1u, RLX_SYS);
            STAMPW(40);
        }
    }
    asm volatile("s_waitcnt vmcnt(0)" ::: "memory");
    __syncthreads();
    if (wave == 0) count_and_flag(P, myws, C_S1, NPRE, P.rank, rfl(ctrv) / NWG + 1u, lane);
}

__device__ __forceinline__ unsigned in_piece(unsigned n, int rank) {
    int r = 0;
#pragma unroll
    for (int i = 1; i < NROUND; ++i) if (n >= rstart_of(i)) r = i;
    unsigned kk = 0, pi = 0; int h0 = 0;
#pragma unroll
    for (int i = 0; i < NROUND; ++i) if (r == i) { const unsigned q = n - rstart_of(i); kk = q / gsz_of(i); pi = q % gsz_of(i); h0 = pend_of(i - 1); }
    if (r == 0 && pi < 32 * PPB) return kk * XKK + pi * (PU * 1024);
    if (r == 0) pi -= 32 * PPB;
    return XBYTES + kk * WKK + (unsigned)panel_of(h0 + (int)(pi / PPB), rank) * BLK + (pi % PPB) * (PU * 1024);
}
__device__ __forceinline__ unsigned group_end(int G) {
    unsigned e = 0;
#pragma unroll
    for (int i = 0; i < NROUND; ++i) if ((G >> 3) == i) e = rstart_of(i) + (unsigned)((G & 7) + 1) * gsz_of(i);
    return e;
}

#define WAIT_V(n) asm volatile("s_waitcnt vmcnt(" #n ")" ::: "memory")
#define WAIT_L(n) asm volatile("s_waitcnt lgkmcnt(" #n ")" ::: "memory")
#define RLX_WG __ATOMIC_RELAXED, __HIP_MEMORY_SCOPE_WORKGROUP
__device__ __forceinline__ void arrive(char* myws, LAS unsigned* lctr, int g, int lane) {
    unsigned old = 0;
    if (lane == 0) old = __hip_atomic_fetch_add(lctr + g, 1u, RLX_WG);
    if (rfl(old) == (g == 0 ? 7u : 6u)) {
        if (lane < 8) __hip_atomic_fetch_add((unsigned*)(myws + C_ARR + g * 1024 + lane * 128), 1u, RLX_AGENT);
        STAMPW(2 + g);
    }
}
__device__ __forceinline__ void puller(const Params& P, char* myws, LAS unsigned* lctr, unsigned ctrv, int idx) {
    const int tid = threadIdx.x, lane = tid & 63, wave = (int)rfl(tid >> 6);
    unsigned* tmo = (unsigned*)(myws + C_TMO);
    if (wave == 7) {
        const unsigned call = rfl(ctrv) / NWG + 1u, p = (unsigned)(P.rank + 1 + lane) & 7u;
        unsigned st = lane < 7 ? 0u : lane == 7 ? 2u : 4u;
        bool counted = false;
        const unsigned long long t0 = __builtin_amdgcn_s_memrealtime();
        for (unsigned polls = 1;; ++polls) {
            if (st < 4u) {
                const unsigned* fw = (const unsigned*)(myws + (st < 2u ? C_G + (p * 2 + st) * 128 : C_F + ((st - 2u) * 8 + p) * 4));
                const unsigned ft = call * (unsigned)(st == 0u ? 32 - NPUSH : st == 1u ? 32 + 2 * nh_of(0) + NPULL : 1);
                if ((int)(__hip_atomic_load(fw, RLX_SYS) - ft) >= 0) { st = lane == 7 ? 4u : st + 1u; __hip_atomic_store(lctr + 32 + lane, st, RLX_WG); }
            }
            if ((polls & 15u) == 0u && expired(tmo, call, t0)) { st = 5u; if (lane < 7) __hip_atomic_store(lctr + 32 + lane, st, RLX_WG); }
            if (!counted && __all(lane == 7 ? st >= 4u : st >= 2u)) { counted = true; arrive(myws, lctr, 0, lane); STAMPW(41); }
            if (__all(st >= 4u)) break;
            if (__all(st >= 2u)) __builtin_amdgcn_s_sleep(CFG_HSLEEP); else if (__all(st >= 1u)) __builtin_amdgcn_s_sleep(8); else __builtin_amdgcn_s_sleep(1);
        }
        return;
    }
    int link = wave; bool ded = false;
    { const int sl = (idx >> 3) * 4 + (idx & 3), h = (idx >> 2) & 1, r = (sl - 7) * 2 + h;
      if (sl < 7) { link = sl; idx = h * 7 + wave; ded = true; } else if (r < 7) { link = r; idx = 14 + wave; ded = true; } else idx = 21 + r - 7; }
    const unsigned p = (unsigned)(P.rank + 1 + link) & 7u;
    char* pws = P.ws[p];
    {
        const rsrc_t rx = mk_rsrc(P.x, M * KS * 2);
        const int sb = lane * 16, swz = sb ^ (((sb >> 9) & 1) << 5), c0 = idx * PJ;
        u32x4 in[PJ];
        const unsigned so = (unsigned)((c0 * 8 + (swz >> 6)) * KS) * 2u + (unsigned)(swz & 63);
#pragma unroll
        for (int j = 0; j < PJ; ++j) in[j] = __builtin_amdgcn_raw_buffer_load_b128(rx, so + (j >> 1) * (16 * KS * 2) + (j & 1) * 64, 0, 0);
        const rsrc_t rp = mk_rsrc(pws + STG + (size_t)(((rfl(ctrv) / NWG + 1u) & 1u) * NRANK + P.rank) * SLOT, XKK);
#pragma unroll
        for (int j = 0; j < PJ; ++j) {
            if (j >= CFG_PWIN) asm volatile("s_waitcnt vmcnt(%0)" :: "n"(CFG_PWIN - 1) : "memory");
            __builtin_amdgcn_raw_buffer_store_b128(in[j], rp, (unsigned)((c0 + j) * 1024 + sb), 0, AUX_SYS);
        }
    }
    const unsigned call = rfl(ctrv) / NWG + 1u, parity = call & 1u;
    const size_t soff = STG + (size_t)(parity * NRANK + p) * SLOT;
    char* const srcI = pws + soff; char* const dstI = myws + soff;
    char* const srcO = P.out[p]; char* const dstO = P.myout;
    const unsigned ycol = (unsigned)(RH + p * OWNH) * 256u;
    const rsrc_t rctl = mk_rsrc(myws, C_END);
    const unsigned tkoff = lane == 0 ? C_TK + (unsigned)link * 128u : C_END;
    const unsigned tbase = (call - 1u) * (NPT - NP0 - NPULL) - NP0 - 2 * NPULL;
    int gsig = 0;
    auto signal = [&](unsigned n) {
        while (gsig < NROUND * 8 && group_end(gsig) <= n) { arrive(myws, lctr, gsig, lane); ++gsig; }
    };
    auto decode = [&](unsigned n, unsigned (&off)[PU]) {
        if (n < NPI) { const unsigned o = in_piece(n, P.rank);
#pragma unroll
            for (int u = 0; u < PU; ++u) off[u] = o + u * 1024 + lane * 16; }
        else { const unsigned q = n - NPI;
#pragma unroll
            for (int u = 0; u < PU; ++u) { const unsigned ci = (q * PU + u) * 64 + lane; off[u] = (ci / YCH) * ROWB + ycol + (ci % YCH) * 16; } }
    };
    unsigned stage = 0;
    auto await = [&](unsigned need) {
        const unsigned long long t0 = __builtin_amdgcn_s_memrealtime();
        unsigned polls = 0;
        while ((stage = rfl(__hip_atomic_load(lctr + 32 + link, RLX_WG))) < need) {
            if ((++polls & 1023u) == 0u && expired(tmo, call, t0)) { stage = 5; break; }
            __builtin_amdgcn_s_sleep(1);
        }
        asm volatile("" ::: "memory");
    };
    unsigned n = NP0 + (unsigned)idx;
    unsigned off[PU];
    decode(n, off);
    rsrc_t rs = mk_rsrc(srcI, SLOT), rd = mk_rsrc(dstI, SLOT);
    STAMP(42);
    await(1);
    bool ok = stage != 5u;
#define LD16(dst, off, rs) asm volatile("buffer_load_dwordx4 %0, %1, %2, 0 offen sc0 sc1" : "=v"(dst) : "v"(off), "s"(rs) : "memory")
#define LANDED(u, cnt) asm volatile("s_waitcnt vmcnt(%1)" : "+v"(R[u]) : "n"(cnt) : "memory")
#define ST16(u) __builtin_amdgcn_raw_buffer_store_b128(ok ? R[u] : (u32x4){0x7fc07fc0u, 0x7fc07fc0u, 0x7fc07fc0u, 0x7fc07fc0u}, rd, off[u], 0, CFG_SIN)
    auto run = [&](auto wc) {
        constexpr int W = decltype(wc)::value;
        u32x4 R[4];
        unsigned tk = n + NPULL + tbase;
        STAMPW(48 + wave);
#pragma unroll
        for (int u = 0; u < W; ++u) LD16(R[u], off[u], rs);
        WAIT_V(0);
        if (lane == 0) __hip_atomic_fetch_add((unsigned*)(pws + C_G + (P.rank * 2 + 1) * 128), 1u, RLX_SYS);
        for (;;) {
            LANDED(0, 2 * W - 1);
            LD16(R[W], off[W], rs);
            ST16(0);
            asm volatile("" : "+v"(tk) :: "memory");
            const unsigned nn = rfl(tk) - tbase;
            tk = 1u;
            asm volatile("buffer_atomic_add %0, %1, %2, 0 offen sc0" : "+v"(tk) : "v"(nn < NPT ? tkoff : C_END), "s"(rctl) : "memory");
            unsigned offn[4];
            decode(nn, offn);
            const unsigned need = nn < 32 * PPB ? 1u : nn < 2 * gsz_of(0) ? 2u : nn < NPI ? 3u : nn < NPT ? 4u : 0u;
            if (stage < need) stage = rfl(__hip_atomic_load(lctr + 32 + link, RLX_WG));
            const rsrc_t rsn = mk_rsrc(nn < NPI ? srcI : srcO, nn < NPI ? SLOT : OUT_BYTES);
#pragma unroll
            for (int u = 1; u < 4; ++u) {
                if (u == 4 - W && stage < need) {
#pragma unroll
                    for (int v = u; v < 4; ++v) { LANDED(v, 0); ST16(v); }
                    WAIT_V(0);
                    signal(nn);
                    await(need);
                }
                LANDED(u, 2 * W - 1 + (u <= W ? 1 : 0));
                if (u + W < 4) LD16(R[u + W], off[u + W], rs); else LD16(R[u + W - 4], offn[u + W - 4], rsn);
                ST16(u);
            }
            if (nn >= NPT) break;
            signal(n);
            if (stage == 5u) ok = false;
            n = nn; rs = rsn;
            rd = mk_rsrc(n < NPI ? dstI : dstO, n < NPI ? SLOT : OUT_BYTES);
#pragma unroll
            for (int u = 0; u < 4; ++u) off[u] = offn[u];
        }
#pragma unroll
        for (int u = 0; u < 4; ++u) LANDED(u, 0);
        asm volatile("" : "+v"(tk) :: "memory");
    };
    if (ded ? wave < CFG_W3 : CFG_W3M) run(Int<3>{}); else run(Int<2>{});
    signal(NPT);
    STAMPW(56 + wave);
#undef LD16
#undef LANDED
#undef ST16
}

__device__ __forceinline__ unsigned cvt_pk_bf16(float lo, float hi) { unsigned r; asm volatile("v_cvt_pk_bf16_f32 %0, %1, %2" : "=v"(r) : "v"(lo), "v"(hi)); return r; }
__device__ __forceinline__ float gelu(float y) {
    const float k1 = -2.0f * 0.7978845608028654f * 1.4426950408889634f, k3 = k1 * 0.044715f;
    const float e = __builtin_amdgcn_exp2f(y * __builtin_fmaf(y * y, k3, k1));
    return y * __builtin_amdgcn_rcpf(1.0f + e);
}

__device__ __forceinline__ void gemm(const Params& P, char* myws, LAS unsigned char* lds, unsigned call, int xcd, int j) {
    const int tid = threadIdx.x, wid = (int)rfl(tid >> 6), lane = tid & 63, wr = wid >> 2, wc = wid & 3, fr = lane & 15, fq = lane >> 4;
    auto row_of = [&](int s) { return (xcd & 3) * 4 + ((NCT & 1) && s >= TPX - 2 ? (xcd >> 2) * 2 + (s & 1) : (s & 3)); };
    auto col_of = [&](int s) { return (NCT & 1) && s >= TPX - 2 ? NCT - 1 : 2 * (s >> 2) + (xcd >> 2); };
    int pm = row_of(j), c = col_of(j);
    unsigned* tmo = (unsigned*)(myws + C_TMO);
    const char* stg = myws + STG + (size_t)(call & 1u) * NRANK * SLOT;
    const char* baseA = stg + (size_t)pm * 2 * BLK;
    const unsigned arr = C_ARR + (unsigned)xcd * 128u;
    const rsrc_t rctl = mk_rsrc(myws, C_END);
    const unsigned voff = (unsigned)tid * 16u, ldsw = (unsigned)wid * 1024u;
    const unsigned atarget = call * NPULL;
    int aoff, boff;
    { const int ob = fr * 64 + fq * 16, sw = ob ^ (((ob >> 9) & 1) << 5); aoff = wr * 8192 + sw; boff = wc * 4096 + sw; }
    bool ok = true;
#define SA(b, h) (((b) * 2 + (h)) * BLK)
#define SB(b, h) ((4 + (b) * 2 + (h)) * BLK)
#define STAGE(bufoff, gbase) do { _Pragma("unroll") for (int _i = 0; _i < 2; ++_i) \
        __builtin_amdgcn_global_load_lds((const unsigned*)((const char*)(gbase) + (voff + _i * 8192u)), (LAS unsigned*)(lds + (bufoff) + ldsw + _i * 8192), 16, 0, AUX_DEV); } while (0)
#define LDA(dst, b, h) do { _Pragma("unroll") for (int m = 0; m < 4; ++m) _Pragma("unroll") for (int k = 0; k < 2; ++k) dst[m][k] = *(const LAS bf16x8*)(lds + SA(b, h) + aoff + m * 2048 + k * 1024); } while (0)
#define LDB(dst, b, h) do { _Pragma("unroll") for (int n = 0; n < 2; ++n) _Pragma("unroll") for (int k = 0; k < 2; ++k) dst[n][k] = *(const LAS bf16x8*)(lds + SB(b, h) + boff + n * 2048 + k * 1024); } while (0)
#define MMA(ai, bj, At, Bt) do { __builtin_amdgcn_s_setprio(1); _Pragma("unroll") for (int m = 0; m < 4; ++m) _Pragma("unroll") for (int n = 0; n < 2; ++n) _Pragma("unroll") for (int k = 0; k < 2; ++k) \
        acc[ai][bj][m][n] = __builtin_amdgcn_mfma_f32_16x16x32_bf16(Bt[n][k], At[m][k], acc[ai][bj][m][n], 0, 0, 0); __builtin_amdgcn_s_setprio(0); } while (0)
#define BAR __builtin_amdgcn_s_barrier()
#define SCHED __builtin_amdgcn_sched_barrier(0)
    f32x4 acc[2][2][4][2];
#pragma unroll
    for (int a = 0; a < 2; ++a)
#pragma unroll
        for (int b = 0; b < 2; ++b)
#pragma unroll
            for (int m = 0; m < 4; ++m)
#pragma unroll
                for (int n = 0; n < 2; ++n) acc[a][b][m][n] = (f32x4){0.f, 0.f, 0.f, 0.f};
    bf16x8 At[4][2], B0[2][2], B1[2][2];
    if (wid == 0) ok = wait_local((const unsigned*)(myws + arr), atarget, tmo, call);
    asm volatile("" ::: "memory"); BAR; asm volatile("" ::: "memory");
    STAMP(2); STAMPC(22);
    int h1 = 2 * c + 1 < NH ? 2 * c + 1 : 2 * c;
    const char* cB0 = stg + XBYTES + (size_t)panel_of(2 * c, P.rank) * BLK;
    const char* cB1 = stg + XBYTES + (size_t)panel_of(h1, P.rank) * BLK;
    STAGE(SB(0, 0), cB0); STAGE(SA(0, 0), baseA); STAGE(SB(0, 1), cB1); STAGE(SA(0, 1), baseA + BLK);
    if (wr == 1) BAR;
    WAIT_V(4); BAR;
    STAGE(SB(1, 0), cB0 + SLOT); STAGE(SA(1, 0), baseA + SLOT); STAGE(SB(1, 1), cB1 + SLOT);
    WAIT_V(6); BAR;
    unsigned pollv = 0;
    for (int round = 0;; ++round) {
        const int sn = (round + 1) * GPX + j;
        const bool has_next = sn < TPX;
        const int cn = col_of(sn), pmn = row_of(sn);
        const char* nA = has_next ? stg + (size_t)pmn * 2 * BLK : baseA;
        const int h1n = 2 * cn + 1 < NH ? 2 * cn + 1 : 2 * cn;
        const char* nB0 = has_next ? stg + XBYTES + (size_t)panel_of(2 * cn, P.rank) * BLK : cB0;
        const char* nB1 = has_next ? stg + XBYTES + (size_t)panel_of(h1n, P.rank) * BLK : cB1;
        for (int t = 0; t < 64; t += 2) {
            const bool last = t == 62;
            const int t2 = (t + 2) & 63;
            const char* a1 = baseA + (size_t)((unsigned)((t & 7) + 1) * SLOT + (unsigned)(t >> 3) * XKK);
            const char* a2 = (last ? nA : baseA) + (size_t)((unsigned)(t2 & 7) * SLOT + (unsigned)(t2 >> 3) * XKK);
            const unsigned kb = (unsigned)(t2 & 7) * SLOT + (unsigned)(t2 >> 3) * WKK;
            const char* b20 = (last ? nB0 : cB0) + kb; const char* b21 = (last ? nB1 : cB1) + kb;
            const unsigned gnext = (unsigned)(round * 8 + (t >> 3) + 1);
            if (wid == 0 && (t & 7) == 6 && (has_next || !last)) {
                if ((int)(rfl(pollv) - atarget) < 0) ok = wait_local((const unsigned*)(myws + arr + gnext * 1024), atarget, tmo, call) && ok;
            }
            LDB(B0, 0, 0); SCHED; LDA(At, 0, 0); STAGE(SA(1, 1), a1 + BLK);
            WAIT_L(8); BAR; WAIT_L(0); MMA(0, 0, At, B0); BAR; SCHED;
            LDB(B1, 0, 1); STAGE(SB(0, 0), b20);
            BAR; WAIT_L(0); MMA(0, 1, At, B1); BAR;
            LDA(At, 0, 1); STAGE(SA(0, 0), a2);
            BAR; WAIT_L(0); MMA(1, 0, At, B0); BAR; SCHED;
            STAGE(SB(0, 1), b21);
            WAIT_V(6); BAR; MMA(1, 1, At, B1); BAR;
            if (wid == 0 && (t & 7) == 4) asm volatile("buffer_load_dword %0, off, %1, %2 sc1" : "+v"(pollv) : "s"(rctl), "s"(arr + gnext * 1024) : "memory");
            LDB(B0, 1, 0); SCHED; LDA(At, 1, 0); STAGE(SA(0, 1), a2 + BLK);
            WAIT_L(8); BAR; WAIT_L(0); MMA(0, 0, At, B0); BAR; SCHED;
            LDB(B1, 1, 1); STAGE(SB(1, 0), b20 + SLOT);
            BAR; WAIT_L(0); MMA(0, 1, At, B1); BAR;
            LDA(At, 1, 1); STAGE(SA(1, 0), a2 + SLOT);
            BAR; WAIT_L(0); MMA(1, 0, At, B0); BAR; SCHED;
            STAGE(SB(1, 1), b21 + SLOT);
            asm volatile("s_waitcnt vmcnt(6)" : "+v"(pollv) :: "memory"); BAR; MMA(1, 1, At, B1); BAR;
        }
        STAMP(3 + 2 * round); STAMPC(23 + 2 * round);
        {
            const rsrc_t ro = mk_rsrc(P.myout, OUT_BYTES);
            const unsigned rowoff = (unsigned)(pm * 256 + wr * 64 + fr) * ROWB + (unsigned)(wc * 32 + fq * 8) * 2u;
#pragma unroll
            for (int bj = 0; bj < 2; ++bj) {
                const unsigned colb = (2 * c + bj < NH) ? (unsigned)panel_of(2 * c + bj, P.rank) * 256u : OUT_BYTES;
#pragma unroll
                for (int ai = 0; ai < 2; ++ai)
#pragma unroll
                    for (int m = 0; m < 4; ++m) {
                        const f32x4 v0 = acc[ai][bj][m][0], v1 = acc[ai][bj][m][1];
                        u32x4 o;
                        o.x = cvt_pk_bf16(gelu(v0[0]), gelu(v0[1])); o.y = cvt_pk_bf16(gelu(v0[2]), gelu(v0[3]));
                        o.z = cvt_pk_bf16(gelu(v1[0]), gelu(v1[1])); o.w = cvt_pk_bf16(gelu(v1[2]), gelu(v1[3]));
                        if (!ok) o = (u32x4){0x7fc07fc0u, 0x7fc07fc0u, 0x7fc07fc0u, 0x7fc07fc0u};
                        if (2 * c < OWNH) __builtin_amdgcn_raw_buffer_store_b128(o, ro, rowoff + colb + (unsigned)(ai * 128 + m * 16) * ROWB, 0, AUX_SYS);
                        else __builtin_amdgcn_raw_buffer_store_b128(o, ro, rowoff + colb + (unsigned)(ai * 128 + m * 16) * ROWB, 0, CFG_SO);
                    }
            }
            if (2 * c < OWNH) {
                WAIT_V(0);
                unsigned old = 0;
                if (lane == 0) old = __hip_atomic_fetch_add((LAS unsigned*)(lds + 8 * BLK) + 1, 1u, RLX_WG);
                if (rfl(old) == 7u) count_and_flag(P, myws, C_YD, 16 * ((OWNH + 1) / 2), 8 + P.rank, call, lane);
            }
        }
        STAMP(4 + 2 * round); STAMPC(24 + 2 * round);
        if (!has_next) break;
#pragma unroll
        for (int a = 0; a < 2; ++a)
#pragma unroll
            for (int b = 0; b < 2; ++b)
#pragma unroll
                for (int m = 0; m < 4; ++m)
#pragma unroll
                    for (int n = 0; n < 2; ++n) acc[a][b][m][n] = (f32x4){0.f, 0.f, 0.f, 0.f};
        c = cn; pm = pmn; baseA = nA; cB0 = nB0; cB1 = nB1;
    }
    WAIT_V(0);
    if (wr == 0) BAR;
#undef SA
#undef SB
#undef STAGE
#undef LDA
#undef LDB
#undef MMA
}

__global__ __launch_bounds__(512, 2) void fused_gemm_allreduce(Params P) {
    extern __shared__ __attribute__((aligned(16))) unsigned char shm[];
    char* myws = P.myws;
    const int b = blockIdx.x, xcd = b & 7, j = b >> 3;
    STAMP(0);
    const unsigned ctrv = __hip_atomic_load((unsigned*)(myws + C_CTR), RLX_AGENT);
    if (threadIdx.x < 64) ((LAS unsigned*)shm)[8 * BLK / 4 + threadIdx.x] = 0;
    WAIT_L(0); __builtin_amdgcn_s_barrier(); asm volatile("" ::: "memory");
    const int pi = (j - GPX) * 8 + xcd;
    if (j >= GPX && pi < NPULL) puller(P, myws, (LAS unsigned*)shm + 8 * BLK / 4, ctrv, pi);
    else {
        prepass(P, myws, (LAS unsigned char*)shm, ctrv, j < GPX ? b : GPX * 8 + pi - NPULL);
        STAMP(1);
        if (j < GPX) gemm(P, myws, (LAS unsigned char*)shm, rfl(ctrv) / NWG + 1u, xcd, j);
    }
    __syncthreads();
    STAMP(63);
    if (threadIdx.x == 0) __hip_atomic_fetch_add((unsigned*)(myws + C_CTR), 1u, RLX_AGENT);
}
}

extern "C" void kernel_launch(int rank, int nranks, ncclComm_t comm, hipStream_t stream,
                              void* const* d_in, const int* in_sizes, int n_in,
                              void* d_out, int out_size, const int* in_dtypes, int out_dtype,
                              void* const* d_out_all, void* const* d_ws_all, size_t ws_bytes) {
    if (nranks != NRANK || (unsigned)rank >= (unsigned)NRANK || n_in < 2 || in_sizes[0] != M * KS || in_sizes[1] != KS * N || out_size != M * N || ws_bytes < WS_NEED) {
        fprintf(stderr, "kernel_launch: unexpected problem geometry\n");
        return;
    }
    static bool attr_set = false;
    if (!attr_set) { HIP_CHECK(hipFuncSetAttribute((const void*)fused_gemm_allreduce, hipFuncAttributeMaxDynamicSharedMemorySize, LDS_BYTES)); attr_set = true; }
    Params P;
    P.x = (const u16*)d_in[0]; P.w = (const u16*)d_in[1]; P.rank = rank;
    for (int p = 0; p < NRANK; ++p) { P.ws[p] = (char*)d_ws_all[p]; P.out[p] = (char*)d_out_all[p]; }
    P.myws = P.ws[rank]; P.myout = (char*)d_out;
    hipLaunchKernelGGL(fused_gemm_allreduce, dim3(NWG), dim3(512), LDS_BYTES, stream, P);
    HIP_CHECK(hipGetLastError());
}
```

```cpp
#include <hip/hip_runtime.h>
#include <rccl/rccl.h>
#include <cstdio>

#define HIP_CHECK(expr) do { hipError_t _e = (expr); if (_e != hipSuccess) { \
    fprintf(stderr, "HIP error %s at %s:%d\n", hipGetErrorString(_e), __FILE__, __LINE__); \
    return; } } while (0)

namespace {
#define LAS __attribute__((address_space(3)))
typedef unsigned short u16;
typedef short bf16x8 __attribute__((ext_vector_type(8)));
typedef float f32x4 __attribute__((ext_vector_type(4)));
typedef unsigned u32x4 __attribute__((ext_vector_type(4)));
typedef __amdgpu_buffer_rsrc_t rsrc_t;

#define CFG_OWNH 6
#define CFG_GPX 29
#define CFG_PPX 3
#define CFG_SIN 17
#define CFG_SO 17
#define CFG_PJ 8
#define CFG_DELAY 350
#define CFG_SPAN 1200
#define CFG_HSLEEP 64
#define CFG_PWIN 3
#define CFG_W3 2
#define CFG_W3M 0
#define CFG_EW 0
#define CFG_DED 1
#define CFG_STEAL 1
#define CFG_PEEK 0
#define CFG_GSLEEP 64
#define CFG_PRO 120
#define CFG_MARG 6
#define CFG_TKS 128
#define CFG_FSLEEP 16
#define CFG_TLB 0

constexpr int NRANK = 8, M = 4096, N = 16384, KS = 512;
constexpr int NWG = 256;
constexpr int OWNH = CFG_OWNH;
constexpr int RH = 128 - NRANK * OWNH;
constexpr int NH = OWNH + RH;
constexpr int NCT = (NH + 1) / 2;
constexpr int GPX = CFG_GPX;
constexpr int TPX = 2 * NCT;
constexpr int NROUND = (TPX + GPX - 1) / GPX;
constexpr int NPULL = 8 * CFG_PPX;
constexpr int NPRE = NWG - NPULL;
constexpr int PU = 4;
constexpr unsigned BLK = 16384;
constexpr unsigned PPB = 16 / PU;
constexpr unsigned XKK = 32 * BLK, WKK = 128 * BLK;
constexpr unsigned XBYTES = 8 * XKK, SLOT = XBYTES + 8 * WKK;
constexpr size_t STG = 1u << 20;
constexpr size_t WS_NEED = STG + 2 * (size_t)NRANK * SLOT;
constexpr unsigned OUT_BYTES = (unsigned)M * N * 2;
constexpr unsigned ROWB = N * 2;
constexpr unsigned TKS = CFG_TKS;
constexpr unsigned C_CTR = 0, C_TMO = 128, C_YD = 256, C_S1 = 384, C_F = 512, C_G = 1024, C_R = 4096, C_ARR = 5120, C_TK = 0xc0000, C_END = C_TK + 16 * TKS, C_STAMP = 65536;
__host__ __device__ constexpr int pend_of(int r) { return r < 0 ? 0 : (((r + 1) * GPX + 3) / 4 * 4 < NH ? ((r + 1) * GPX + 3) / 4 * 4 : NH); }
__host__ __device__ constexpr int nh_of(int r) { return pend_of(r) - pend_of(r - 1); }
__host__ __device__ constexpr unsigned gsz_of(int r) { return ((r == 0 ? 32 : 0) + nh_of(r)) * PPB; }
__host__ __device__ constexpr unsigned rstart_of(int r) { return r == 0 ? 0u : rstart_of(r - 1) + 8 * gsz_of(r - 1); }
constexpr int R0 = nh_of(0) - OWNH;
constexpr int C0 = 32 + R0 + NRANK * OWNH;
constexpr int FK = 2;
constexpr int N0 = FK * C0, NIMG = 8 * (32 + 128);
constexpr int PJ = CFG_PJ;
constexpr int NPUSH = PJ * NPULL / 16;
constexpr unsigned long long DEADLINE = 150000000ull;
constexpr int LDS_BYTES = 8 * BLK + 256;
constexpr unsigned NP0 = NPUSH * PPB;
constexpr unsigned NPI = rstart_of(NROUND);
constexpr unsigned YCH = OWNH * 16;
constexpr unsigned NPO = M * YCH / (64 * PU);
constexpr unsigned NPT = NPI + NPO;
constexpr unsigned SEG = NPO / 7, LAST = NPT - SEG;
constexpr unsigned TB = NP0 + 2 * NPULL;
static_assert(N0 <= NPRE && NPRE + NIMG - N0 <= 8 * NPRE, "pre-pass list");
static_assert(NPUSH * 16 == PJ * NPULL && PJ % 2 == 0 && PJ > 0 && NPUSH <= 32, "push split");
static_assert(GPX / 4 * 4 >= OWNH + 1 && R0 > 0 && R0 < RH && GPX * 8 + NPULL <= NWG && NROUND <= 4 && nh_of(NROUND - 1) > 0, "config");
static_assert(C_ARR + (NROUND * 8 + 1) * 1024 <= C_STAMP && C_END <= STG, "ctrl");
static_assert(CFG_PRO <= SEG, "a wave leaves its list only behind the last mark");
static_assert(32 * PPB >= NP0 + NPULL, "every puller wave starts on x of K tile 0");

template <int V> struct Int { static constexpr int value = V; };
struct Params { const u16* x; const u16* w; char* myws; char* myout; int rank; char* ws[NRANK]; char* out[NRANK]; };

#define RLX_AGENT __ATOMIC_RELAXED, __HIP_MEMORY_SCOPE_AGENT
#define RLX_SYS __ATOMIC_RELAXED, __HIP_MEMORY_SCOPE_SYSTEM
#define AUX_SYS 17
#define AUX_DEV 16

__device__ __forceinline__ unsigned rfl(unsigned v) { return (unsigned)__builtin_amdgcn_readfirstlane((int)v); }
__device__ __forceinline__ rsrc_t mk_rsrc(const void* p, unsigned bytes) { return __builtin_amdgcn_make_buffer_rsrc((void*)p, 0, (int)bytes, 0x00020000); }
__device__ __forceinline__ int panel_of(int h, int rank) { return h < OWNH ? RH + rank * OWNH + h : h - OWNH; }

__device__ __forceinline__ unsigned peek(rsrc_t r, unsigned off) { return (unsigned)__builtin_amdgcn_raw_ptr_buffer_atomic_add_i32(0, r, (int)off, 0, AUX_DEV); }
__device__ __forceinline__ bool expired(unsigned* tmo, unsigned call, unsigned long long t0) {
    if (rfl(__hip_atomic_load(tmo, RLX_AGENT)) == call || __builtin_amdgcn_s_memrealtime() - t0 > DEADLINE) {
        __hip_atomic_store(tmo, call, RLX_AGENT);
        return true;
    }
    return false;
}
__device__ __forceinline__ bool wait_local(const unsigned* p, unsigned target, unsigned* tmo, unsigned call) {
    const unsigned long long t0 = __builtin_amdgcn_s_memrealtime();
    unsigned polls = 0;
    while ((int)(rfl(__hip_atomic_load(p, RLX_AGENT)) - target) < 0) {
        if ((++polls & 15u) == 0u && expired(tmo, call, t0)) return false;
        __builtin_amdgcn_s_sleep(CFG_GSLEEP);
    }
    asm volatile("" ::: "memory");
    return true;
}

__device__ __forceinline__ void count_and_flag(const Params& P, char* myws, unsigned cnt, unsigned total, unsigned word, unsigned call, int lane) {
    unsigned old = 0;
    if (lane == 0) old = __hip_atomic_fetch_add((unsigned*)(myws + cnt), 1u, RLX_AGENT);
    if (rfl(old) + 1u == call * total && lane < NRANK) __hip_atomic_store((unsigned*)(P.ws[lane] + C_F) + word, call, RLX_SYS);
}

#define WST(i, val) do {} while (0)
#define WDECL do {} while (0)
#define WDUMP do {} while (0)
#define STAMPC(i) do {} while (0)
#define STAMP(i) do {} while (0)
#define STAMPW(i) do {} while (0)

__device__ __forceinline__ void prepass(const Params& P, char* myws, LAS unsigned char* lds, unsigned ctrv, int u) {
    const int tid = threadIdx.x, lane = tid & 63, wave = (int)rfl(tid >> 6), half = wave & 1, pair = wave >> 1;
    const int kgl = lane >> 4, ng = lane & 15, sb = lane * 16, swz = sb ^ (((sb >> 9) & 1) << 5);
    const unsigned wlo = (unsigned)((half * 4 + kgl) * 8 * N + ng * 8) * 2u, xlo = (unsigned)((swz >> 6) * KS + half * 32) * 2u + (unsigned)(swz & 63);
    LAS unsigned char* lb = lds + pair * BLK + half * 1024;
    const unsigned ioff = (unsigned)((ng >> 2) * 4096 + (ng & 3) * 256 + ((kgl * 16) ^ (((ng >> 1) & 1) << 5)));
    LAS unsigned* cnt = (LAS unsigned*)(lds + 8 * BLK);
    char* const wsl = P.ws[lane & 7];
    WDECL; WST(0, 0); int wsi = 1;
    const unsigned long long tgo = __builtin_amdgcn_s_memrealtime() + CFG_DELAY;
    for (int i = pair * NPRE + u; i < NPRE + NIMG - N0; i += 4 * NPRE) {
        int kk, img;
        if (i < NPRE + (8 - FK) * C0) { const int v = i < NPRE ? i : i - NPRE + N0; if (i >= N0 && i < NPRE) continue;
            kk = v / C0; const int bi = v % C0; img = bi < 32 + R0 ? bi : bi + (RH - R0); }
        else { const int v = i - (NPRE + (8 - FK) * C0); kk = v / (RH - R0); img = 32 + R0 + v % (RH - R0); }
        if (i >= NPRE) while (__builtin_amdgcn_s_memrealtime() < tgo + (unsigned)((i - NPRE) * CFG_SPAN / (NIMG - N0))) __builtin_amdgcn_s_sleep(4);
        const bool isw = img >= 32;
        const unsigned so = isw ? wlo + (unsigned)(kk * 64 * N + (img - 32) * 128) * 2u : xlo + (unsigned)(img * 128 * KS + kk * 64) * 2u;
        const unsigned sstep = isw ? ROWB : 16 * KS * 2;
        const unsigned dbase = isw ? XBYTES + (unsigned)kk * WKK + (unsigned)(img - 32) * BLK : (unsigned)kk * XKK + (unsigned)img * BLK;
        const rsrc_t ri = mk_rsrc(isw ? P.w : P.x, isw ? KS * N * 2 : M * KS * 2);
        u32x4 in[8];
#pragma unroll
        for (int j = 0; j < 8; ++j) in[j] = __builtin_amdgcn_raw_buffer_load_b128(ri, so + j * sstep, 0, 0);
        if (isw) {
#pragma unroll
            for (int v8 = 0; v8 < 8; ++v8) {
                u32x4 o;
#pragma unroll
                for (int e = 0; e < 4; ++e) o[e] = __builtin_amdgcn_perm(in[2 * e + 1][v8 >> 1], in[2 * e][v8 >> 1], (v8 & 1) ? 0x07060302u : 0x05040100u);
                *(LAS u32x4*)(lb + ioff + (v8 >> 2) * 2048 + (v8 & 3) * 64) = o;
            }
            asm volatile("s_waitcnt lgkmcnt(0)" ::: "memory");
#pragma unroll
            for (int j = 0; j < 8; ++j) in[j] = *(LAS u32x4*)(lb + j * 2048 + lane * 16);
        }
        const unsigned parity = (rfl(ctrv) / NWG + 1u) & 1u;
        const rsrc_t rs = mk_rsrc(myws + STG + (size_t)(parity * NRANK + P.rank) * SLOT, SLOT);
#pragma unroll
        for (int j = 0; j < 8; ++j) __builtin_amdgcn_raw_buffer_store_b128(in[j], rs, dbase + (unsigned)(j * 2048 + half * 1024 + lane * 16), 0, AUX_SYS);
        if (i < NPRE) {
            asm volatile("s_waitcnt vmcnt(0)" ::: "memory");
            unsigned second = 0;
            if (lane == 0) second = __hip_atomic_fetch_add(cnt, 1u, __ATOMIC_RELAXED, __HIP_MEMORY_SCOPE_WORKGROUP);
            if (rfl(second) == 1u && i >= NPUSH && lane < NRANK && lane != P.rank && (img < 32 + RH || (img - 32 - RH) / OWNH == lane))
                __hip_atomic_fetch_add((unsigned*)(wsl + C_G + (P.rank * 3 + (i < 32 ? 0 : 1)) * 128), 1u, RLX_SYS);
            STAMPW(40);
        }
    }
    WDUMP;
    asm volatile("s_waitcnt vmcnt(0)" ::: "memory");
    __syncthreads();
    if (wave == 0) count_and_flag(P, myws, C_S1, NPRE, P.rank, rfl(ctrv) / NWG + 1u, lane);
}

__device__ __forceinline__ unsigned in_piece(unsigned n, int rank) {
    int r = 0;
#pragma unroll
    for (int i = 1; i < NROUND; ++i) if (n >= rstart_of(i)) r = i;
    unsigned kk = 0, pi = 0; int h0 = 0;
#pragma unroll
    for (int i = 0; i < NROUND; ++i) if (r == i) { const unsigned q = n - rstart_of(i); kk = q / gsz_of(i); pi = q % gsz_of(i); h0 = pend_of(i - 1); }
    if (r == 0 && pi < 32 * PPB) return kk * XKK + pi * (PU * 1024);
    if (r == 0) pi -= 32 * PPB;
    return XBYTES + kk * WKK + (unsigned)panel_of(h0 + (int)(pi / PPB), rank) * BLK + (pi % PPB) * (PU * 1024);
}
__device__ __forceinline__ unsigned group_end(int G) {
    unsigned e = 0;
#pragma unroll
    for (int i = 0; i < NROUND; ++i) if ((G >> 3) == i) e = rstart_of(i) + (unsigned)((G & 7) + 1) * gsz_of(i);
    return e;
}

#define WAIT_V(n) asm volatile("s_waitcnt vmcnt(" #n ")" ::: "memory")
#define WAIT_L(n) asm volatile("s_waitcnt lgkmcnt(" #n ")" ::: "memory")
#define RLX_WG __ATOMIC_RELAXED, __HIP_MEMORY_SCOPE_WORKGROUP
__device__ __forceinline__ void arrive(char* myws, LAS unsigned* lctr, int g, int lane) {
    unsigned old = 0;
    if (lane == 0) old = __hip_atomic_fetch_add(lctr + g, 1u, RLX_WG);
    if (rfl(old) == (g == 0 ? 7u : 6u)) {
        if (lane < 8) __hip_atomic_fetch_add((unsigned*)(myws + C_ARR + g * 1024 + lane * 128), 1u, RLX_AGENT);
        STAMPW(2 + g);
    }
}
__device__ __forceinline__ void puller(const Params& P, char* myws, LAS unsigned* lctr, unsigned ctrv, int idx) {
    const int tid = threadIdx.x, lane = tid & 63, wave = (int)rfl(tid >> 6);
    unsigned* tmo = (unsigned*)(myws + C_TMO);
    int link = wave; bool ded = false;
    { const int sl = (idx >> 3) * 4 + (idx & 3), h = (idx >> 2) & 1, r = (sl - 7) * 2 + h;
      if (sl < 7) { link = sl; idx = h * 7 + wave; ded = true; } else if (r < 7) { link = r; idx = 14 + wave; ded = true; } else idx = 21 + r - 7; }
    if (wave == 7) {
        const unsigned call = rfl(ctrv) / NWG + 1u, p = (unsigned)(P.rank + 1 + lane) & 7u;
        unsigned st = lane < 7 ? (ded && lane != link ? 5u : 0u) : lane == 7 ? 3u : 5u;
        bool counted = false;
        const rsrc_t rc = mk_rsrc(myws, C_END);
        const unsigned long long t0 = __builtin_amdgcn_s_memrealtime();
        for (unsigned polls = 1;; ++polls) {
            if (st < 5u) {
                const unsigned fw = st < 3u ? C_G + (p * 3 + st) * 128 : C_F + ((st - 3u) * 8 + p) * 4;
                const unsigned ft = call * (unsigned)(st == 0u ? 32 - NPUSH : st == 1u ? 32 + 2 * nh_of(0) : st == 2u ? NPULL : 1);
                if ((int)((CFG_PEEK ? peek(rc, fw) : __hip_atomic_load((const unsigned*)(myws + fw), RLX_SYS)) - ft) >= 0) { st = lane == 7 ? 5u : st + 1u; __hip_atomic_store(lctr + 32 + lane, st, RLX_WG); }
            }
            if ((polls & 15u) == 0u && expired(tmo, call, t0)) { st = 6u; if (lane < 7) __hip_atomic_store(lctr + 32 + lane, st, RLX_WG); }
            if (!counted && __all(lane == 7 ? st >= 5u : st >= 3u)) { counted = true; arrive(myws, lctr, 0, lane); STAMPW(41); }
            if (__all(st >= 5u)) break;
            if (__all(st >= 2u)) __builtin_amdgcn_s_sleep(CFG_HSLEEP); else if (__all(st >= 1u)) __builtin_amdgcn_s_sleep(8); else __builtin_amdgcn_s_sleep(1);
        }
        const int q = lane >> 3, l = lane & 7;
        const unsigned fo = q == 7 ? C_END : l == 7 ? C_TK + ((call & 1u) * 8 + (unsigned)q) * TKS : ded && q != link ? C_END : C_R + (unsigned)(q * 128 + l * 4);
        for (unsigned polls = 1;; ++polls) {
            unsigned v = peek(rc, fo);
            const unsigned long long have = __ballot(l < 7 && (int)(v - call * NPULL) >= 0);
            v += TB;
            if (__any(st == 6u) || ((polls & 15u) == 0u && expired(tmo, call, t0))) v = NPT;
            if (l == 7 && q < 7) { __hip_atomic_store(lctr + 48 + q, (unsigned)(have >> (8 * q)) & 127u, RLX_WG); __hip_atomic_fetch_max(lctr + 40 + q, v, RLX_WG); }
            if (__all(l < 7 || q == 7 || v >= NPT)) break;
            if (__any(l == 7 && q < 7 && v + CFG_PRO + 120 >= NPT)) __builtin_amdgcn_s_sleep(CFG_FSLEEP); else __builtin_amdgcn_s_sleep(127);
        }
        return;
    }
    WDECL; WST(0, 0);
    const unsigned p = (unsigned)(P.rank + 1 + link) & 7u;
    char* pws = P.ws[p];
    {
        const rsrc_t rx = mk_rsrc(P.x, M * KS * 2);
        const int sb = lane * 16, swz = sb ^ (((sb >> 9) & 1) << 5), c0 = idx * PJ;
        u32x4 in[PJ];
        const unsigned so = (unsigned)((c0 * 8 + (swz >> 6)) * KS) * 2u + (unsigned)(swz & 63);
#pragma unroll
        for (int j = 0; j < PJ; ++j) in[j] = __builtin_amdgcn_raw_buffer_load_b128(rx, so + (j >> 1) * (16 * KS * 2) + (j & 1) * 64, 0, 0);
        const rsrc_t rp = mk_rsrc(pws + STG + (size_t)(((rfl(ctrv) / NWG + 1u) & 1u) * NRANK + P.rank) * SLOT, XKK);
#pragma unroll
        for (int j = 0; j < PJ; ++j) {
            if (j >= CFG_PWIN) asm volatile("s_waitcnt vmcnt(%0)" :: "n"(CFG_PWIN - 1) : "memory");
            __builtin_amdgcn_raw_buffer_store_b128(in[j], rp, (unsigned)((c0 + j) * 1024 + sb), 0, AUX_SYS);
        }
    }
    const unsigned call = rfl(ctrv) / NWG + 1u, parity = call & 1u;
    const size_t soff = STG + (size_t)(parity * NRANK + p) * SLOT;
    char* const srcI = pws + soff; char* const dstI = myws + soff;
    char* const srcO = P.out[p]; char* const dstO = P.myout;
    const rsrc_t rctl = mk_rsrc(myws, C_END);
    const unsigned tkoff = lane == 0 ? C_TK + parity * 8 * TKS : C_END;
    int gsig = 0;
    auto signal = [&](unsigned n) {
        while (gsig < NROUND * 8 && group_end(gsig) <= n) { arrive(myws, lctr, gsig, lane); ++gsig; }
        while (gsig >= NROUND * 8 && gsig < NROUND * 8 + 6 && LAST - (unsigned)(NROUND * 8 + 5 - gsig) * SEG <= n) {
            const int vl = (link + 1 + gsig - NROUND * 8) % 7, r = (int)(p - 1u - (unsigned)vl) & 7;
            if (lane == 0) __hip_atomic_fetch_add((unsigned*)(P.ws[r] + C_R + ((P.rank - r - 1) & 7) * 128 + vl * 4), 1u, RLX_SYS);
            ++gsig;
        }
    };
    auto decode = [&](unsigned n, int l, unsigned (&off)[PU]) {
        if (n < NPI) { const unsigned o = in_piece(n, P.rank);
#pragma unroll
            for (int u = 0; u < PU; ++u) off[u] = o + u * 1024 + lane * 16; }
        else { unsigned q = n - NPI + (unsigned)l * SEG; if (q >= NPO) q -= NPO; if (n >= NPT) q = NPO;
            const unsigned ycol = (unsigned)(RH + ((P.rank + 1 + l) & 7) * OWNH) * 256u;
#pragma unroll
            for (int u = 0; u < PU; ++u) { const unsigned ci = (q * PU + u) * 64 + lane; off[u] = (ci / YCH) * ROWB + ycol + (ci % YCH) * 16; } }
    };
    auto pick = [&]() {
        unsigned k = lane < 7 ? __hip_atomic_load(lctr + 40 + lane, RLX_WG) : NPT;
        const unsigned m = rfl(__hip_atomic_load(lctr + 48 + link, RLX_WG));
        k = k < NPT ? k : NPT;
        int best = __all(k >= NPT) ? 8 : 7; unsigned bk = NPT, sum = 0;
#pragma unroll
        for (int l = 0; l < 7; ++l) { const unsigned kl = (unsigned)__builtin_amdgcn_readlane((int)k, l); sum += kl; if (((m >> l) & 1u) && kl >= LAST && kl < bk) { best = l; bk = kl; } }
        const unsigned ko = (unsigned)__builtin_amdgcn_readlane((int)k, link);
        return ko < NPT && (ko * 7u <= sum + 7u * CFG_MARG || ko <= bk) ? link : best;
    };
    unsigned stage = 0;
    auto await = [&](unsigned need) {
        const unsigned long long t0 = __builtin_amdgcn_s_memrealtime();
        unsigned polls = 0;
        while ((stage = rfl(__hip_atomic_load(lctr + 32 + link, RLX_WG))) < need) {
            if ((++polls & 1023u) == 0u && expired(tmo, call, t0)) { stage = 6; break; }
            __builtin_amdgcn_s_sleep(1);
        }
        asm volatile("" ::: "memory");
    };
    unsigned n = NP0 + (unsigned)idx;
    unsigned off[PU];
    decode(n, link, off);
    rsrc_t rs = mk_rsrc(srcI, SLOT), rd = mk_rsrc(dstI, SLOT);
    WST(2, 0);
    await(1);
    WST(3, 0);
    int wsi = 5;
    bool ok = stage != 6u;
#define LD16(dst, off, rs) asm volatile("buffer_load_dwordx4 %0, %1, %2, 0 offen sc0 sc1" : "=v"(dst) : "v"(off), "s"(rs) : "memory")
#define LANDED(u, cnt) asm volatile("s_waitcnt vmcnt(%1)" : "+v"(R[u]) : "n"(cnt) : "memory")
#define ST16(u) __builtin_amdgcn_raw_buffer_store_b128(ok ? R[u] : (u32x4){0x7fc07fc0u, 0x7fc07fc0u, 0x7fc07fc0u, 0x7fc07fc0u}, rd, off[u], 0, CFG_SIN)
    auto run = [&](auto wc) {
        constexpr int W = decltype(wc)::value;
        u32x4 R[4];
        unsigned tk = n + NPULL - TB, spins = 0;
        int cl = link;
        bool own = true;
        const unsigned long long t0 = __builtin_amdgcn_s_memrealtime();
#pragma unroll
        for (int u = 0; u < W; ++u) LD16(R[u], off[u], rs);
        WAIT_V(0);
        WST(4, n);
        if (lane == 0) __hip_atomic_fetch_add((unsigned*)(pws + C_G + (P.rank * 3 + 2) * 128), 1u, RLX_SYS);
        for (;;) {
            LANDED(0, 2 * W - 1);
            LD16(R[W], off[W], rs);
            ST16(0);
            asm volatile("" : "+v"(tk) :: "memory");
            const unsigned nn = cl < 7 ? rfl(tk) + TB : NPT;
            const int nl = cl;
            if (nn + (CFG_STEAL ? CFG_PRO : 0) >= NPT || cl != link) {
                if (cl < 7) { if (lane == 0 && nn + 1u >= NPT) __hip_atomic_store(lctr + 40 + cl, NPT, RLX_WG); }
                else __builtin_amdgcn_s_sleep(4);
                cl = pick();
                if (cl == 7 && (++spins & 1023u) == 0u && expired(tmo, call, t0)) cl = 8;
            }
            tk = 1u;
            asm volatile("buffer_atomic_add %0, %1, %2, 0 offen sc0" : "+v"(tk) : "v"(cl < 7 ? tkoff + (unsigned)cl * TKS : C_END), "s"(rctl) : "memory");
            unsigned offn[4];
            decode(nn, nl, offn);
            const unsigned need = nn < 32 * PPB ? 1u : nn < 2 * gsz_of(0) ? 2u : nn < NPI ? 4u : nn < NPT && nl == link ? 5u : 0u;
            if (stage < need) stage = rfl(__hip_atomic_load(lctr + 32 + link, RLX_WG));
            const rsrc_t rsn = mk_rsrc(nn < NPI ? srcI : srcO, nn < NPI ? SLOT : OUT_BYTES);
#pragma unroll
            for (int u = 1; u < 4; ++u) {
                if (u == 4 - W && stage < need) {
#pragma unroll
                    for (int v = u; v < 4; ++v) { LANDED(v, 0); ST16(v); }
                    WAIT_V(0);
                    signal(nn);
                    await(need);
                }
                LANDED(u, 2 * W - 1 + (u <= W ? 1 : 0));
                if (u + W < 4) LD16(R[u + W], off[u + W], rs); else LD16(R[u + W - 4], offn[u + W - 4], rsn);
                ST16(u);
            }
            if (n < NPT && (wsi < 40 || n + 12 * NPULL >= NPT || !own)) { WST(wsi, own ? n : n | 0x80000000u); if (wsi < 59) ++wsi; }
            if (nn >= NPT && cl == 8) break;
            signal(own ? n : NPT);
            own = nl == link;
            if (stage == 6u) ok = false;
            n = nn; rs = rsn;
            rd = mk_rsrc(n < NPI ? dstI : dstO, n < NPI ? SLOT : OUT_BYTES);
#pragma unroll
            for (int u = 0; u < 4; ++u) off[u] = offn[u];
        }
#pragma unroll
        for (int u = 0; u < 4; ++u) LANDED(u, 0);
        asm volatile("" : "+v"(tk) :: "memory");
    };
    if (ded ? wave < CFG_W3 : CFG_W3M) run(Int<3>{}); else run(Int<2>{});
    signal(NPT);
    STAMPW(56 + wave);
    WST(61, 0); WDUMP;
#undef LD16
#undef LANDED
#undef ST16
}

__device__ __forceinline__ unsigned cvt_pk_bf16(float lo, float hi) { unsigned r; asm volatile("v_cvt_pk_bf16_f32 %0, %1, %2" : "=v"(r) : "v"(lo), "v"(hi)); return r; }
__device__ __forceinline__ float gelu(float y) {
    const float k1 = -2.0f * 0.7978845608028654f * 1.4426950408889634f, k3 = k1 * 0.044715f;
    const float e = __builtin_amdgcn_exp2f(y * __builtin_fmaf(y * y, k3, k1));
    return y * __builtin_amdgcn_rcpf(1.0f + e);
}

__device__ __forceinline__ void gemm(const Params& P, char* myws, LAS unsigned char* lds, unsigned call, int xcd, int j) {
    const int tid = threadIdx.x, wid = (int)rfl(tid >> 6), lane = tid & 63, wr = wid >> 2, wc = wid & 3, fr = lane & 15, fq = lane >> 4;
    auto row_of = [&](int s) { return (xcd & 3) * 4 + ((NCT & 1) && s >= TPX - 2 ? (xcd >> 2) * 2 + (s & 1) : (s & 3)); };
    auto col_of = [&](int s) { return (NCT & 1) && s >= TPX - 2 ? NCT - 1 : 2 * (s >> 2) + (xcd >> 2); };
    int pm = row_of(j), c = col_of(j);
    unsigned* tmo = (unsigned*)(myws + C_TMO);
    const char* stg = myws + STG + (size_t)(call & 1u) * NRANK * SLOT;
    const char* baseA = stg + (size_t)pm * 2 * BLK;
    const unsigned arr = C_ARR + (unsigned)xcd * 128u;
    const rsrc_t rctl = mk_rsrc(myws, C_END);
    const unsigned voff = (unsigned)tid * 16u, ldsw = (unsigned)wid * 1024u;
    const unsigned atarget = call * NPULL;
    int aoff, boff;
    { const int ob = fr * 64 + fq * 16, sw = ob ^ (((ob >> 9) & 1) << 5); aoff = wr * 8192 + sw; boff = wc * 4096 + sw; }
    bool ok = true;
#define SA(b, h) (((b) * 2 + (h)) * BLK)
#define SB(b, h) ((4 + (b) * 2 + (h)) * BLK)
#define STAGE(bufoff, gbase) do { _Pragma("unroll") for (int _i = 0; _i < 2; ++_i) \
        __builtin_amdgcn_global_load_lds((const unsigned*)((const char*)(gbase) + (voff + _i * 8192u)), (LAS unsigned*)(lds + (bufoff) + ldsw + _i * 8192), 16, 0, AUX_DEV); } while (0)
#define LDA(dst, b, h) do { _Pragma("unroll") for (int m = 0; m < 4; ++m) _Pragma("unroll") for (int k = 0; k < 2; ++k) dst[m][k] = *(const LAS bf16x8*)(lds + SA(b, h) + aoff + m * 2048 + k * 1024); } while (0)
#define LDB(dst, b, h) do { _Pragma("unroll") for (int n = 0; n < 2; ++n) _Pragma("unroll") for (int k = 0; k < 2; ++k) dst[n][k] = *(const LAS bf16x8*)(lds + SB(b, h) + boff + n * 2048 + k * 1024); } while (0)
#define MMA(ai, bj, At, Bt) do { __builtin_amdgcn_s_setprio(1); _Pragma("unroll") for (int m = 0; m < 4; ++m) _Pragma("unroll") for (int n = 0; n < 2; ++n) _Pragma("unroll") for (int k = 0; k < 2; ++k) \
        acc[ai][bj][m][n] = __builtin_amdgcn_mfma_f32_16x16x32_bf16(Bt[n][k], At[m][k], acc[ai][bj][m][n], 0, 0, 0); __builtin_amdgcn_s_setprio(0); } while (0)
#define BAR __builtin_amdgcn_s_barrier()
#define SCHED __builtin_amdgcn_sched_barrier(0)
    f32x4 acc[2][2][4][2];
#pragma unroll
    for (int a = 0; a < 2; ++a)
#pragma unroll
        for (int b = 0; b < 2; ++b)
#pragma unroll
            for (int m = 0; m < 4; ++m)
#pragma unroll
                for (int n = 0; n < 2; ++n) acc[a][b][m][n] = (f32x4){0.f, 0.f, 0.f, 0.f};
    bf16x8 At[4][2], B0[2][2], B1[2][2];
    if (wid == 0) ok = wait_local((const unsigned*)(myws + arr), atarget, tmo, call);
    asm volatile("" ::: "memory"); BAR; asm volatile("" ::: "memory");
    STAMP(2); STAMPC(22);
    int h1 = 2 * c + 1 < NH ? 2 * c + 1 : 2 * c;
    const char* cB0 = stg + XBYTES + (size_t)panel_of(2 * c, P.rank) * BLK;
    const char* cB1 = stg + XBYTES + (size_t)panel_of(h1, P.rank) * BLK;
    STAGE(SB(0, 0), cB0); STAGE(SA(0, 0), baseA); STAGE(SB(0, 1), cB1); STAGE(SA(0, 1), baseA + BLK);
    if (wr == 1) BAR;
    WAIT_V(4); BAR;
    STAGE(SB(1, 0), cB0 + SLOT); STAGE(SA(1, 0), baseA + SLOT); STAGE(SB(1, 1), cB1 + SLOT);
    WAIT_V(6); BAR;
    unsigned pollv = 0;
    for (int round = 0;; ++round) {
        const int sn = (round + 1) * GPX + j;
        const bool has_next = sn < TPX;
        const int cn = col_of(sn), pmn = row_of(sn);
        const char* nA = has_next ? stg + (size_t)pmn * 2 * BLK : baseA;
        const int h1n = 2 * cn + 1 < NH ? 2 * cn + 1 : 2 * cn;
        const char* nB0 = has_next ? stg + XBYTES + (size_t)panel_of(2 * cn, P.rank) * BLK : cB0;
        const char* nB1 = has_next ? stg + XBYTES + (size_t)panel_of(h1n, P.rank) * BLK : cB1;
        for (int t = 0; t < 64; t += 2) {
            const bool last = t == 62;
            const int t2 = (t + 2) & 63;
            const char* a1 = baseA + (size_t)((unsigned)((t & 7) + 1) * SLOT + (unsigned)(t >> 3) * XKK);
            const char* a2 = (last ? nA : baseA) + (size_t)((unsigned)(t2 & 7) * SLOT + (unsigned)(t2 >> 3) * XKK);
            const unsigned kb = (unsigned)(t2 & 7) * SLOT + (unsigned)(t2 >> 3) * WKK;
            const char* b20 = (last ? nB0 : cB0) + kb; const char* b21 = (last ? nB1 : cB1) + kb;
            const unsigned gnext = (unsigned)(round * 8 + (t >> 3) + 1);
            if (wid == 0 && (t & 7) == 6 && (has_next || !last)) {
                if ((int)(rfl(pollv) - atarget) < 0) ok = wait_local((const unsigned*)(myws + arr + gnext * 1024), atarget, tmo, call) && ok;
            }
            LDB(B0, 0, 0); SCHED; LDA(At, 0, 0); STAGE(SA(1, 1), a1 + BLK);
            WAIT_L(8); BAR; WAIT_L(0); MMA(0, 0, At, B0); BAR; SCHED;
            LDB(B1, 0, 1); STAGE(SB(0, 0), b20);
            BAR; WAIT_L(0); MMA(0, 1, At, B1); BAR;
            LDA(At, 0, 1); STAGE(SA(0, 0), a2);
            BAR; WAIT_L(0); MMA(1, 0, At, B0); BAR; SCHED;
            STAGE(SB(0, 1), b21);
            WAIT_V(6); BAR; MMA(1, 1, At, B1); BAR;
            if (wid == 0 && (t & 7) == 4) asm volatile("buffer_load_dword %0, off, %1, %2 sc1" : "+v"(pollv) : "s"(rctl), "s"(arr + gnext * 1024) : "memory");
            LDB(B0, 1, 0); SCHED; LDA(At, 1, 0); STAGE(SA(0, 1), a2 + BLK);
            WAIT_L(8); BAR; WAIT_L(0); MMA(0, 0, At, B0); BAR; SCHED;
            LDB(B1, 1, 1); STAGE(SB(1, 0), b20 + SLOT);
            BAR; WAIT_L(0); MMA(0, 1, At, B1); BAR;
            LDA(At, 1, 1); STAGE(SA(1, 0), a2 + SLOT);
            BAR; WAIT_L(0); MMA(1, 0, At, B0); BAR; SCHED;
            STAGE(SB(1, 1), b21 + SLOT);
            asm volatile("s_waitcnt vmcnt(6)" : "+v"(pollv) :: "memory"); BAR; MMA(1, 1, At, B1); BAR;
        }
        STAMP(3 + 2 * round); STAMPC(23 + 2 * round);
        {
            const rsrc_t ro = mk_rsrc(P.myout, OUT_BYTES);
            const unsigned rowoff = (unsigned)(pm * 256 + wr * 64 + fr) * ROWB + (unsigned)(wc * 32 + fq * 8) * 2u;
#pragma unroll
            for (int bj = 0; bj < 2; ++bj) {
                const unsigned colb = (2 * c + bj < NH) ? (unsigned)panel_of(2 * c + bj, P.rank) * 256u : OUT_BYTES;
#pragma unroll
                for (int ai = 0; ai < 2; ++ai)
#pragma unroll
                    for (int m = 0; m < 4; ++m) {
                        const f32x4 v0 = acc[ai][bj][m][0], v1 = acc[ai][bj][m][1];
                        u32x4 o;
                        o.x = cvt_pk_bf16(gelu(v0[0]), gelu(v0[1])); o.y = cvt_pk_bf16(gelu(v0[2]), gelu(v0[3]));
                        o.z = cvt_pk_bf16(gelu(v1[0]), gelu(v1[1])); o.w = cvt_pk_bf16(gelu(v1[2]), gelu(v1[3]));
                        if (!ok) o = (u32x4){0x7fc07fc0u, 0x7fc07fc0u, 0x7fc07fc0u, 0x7fc07fc0u};
                        if (2 * c < OWNH) __builtin_amdgcn_raw_buffer_store_b128(o, ro, rowoff + colb + (unsigned)(ai * 128 + m * 16) * ROWB, 0, AUX_SYS);
                        else __builtin_amdgcn_raw_buffer_store_b128(o, ro, rowoff + colb + (unsigned)(ai * 128 + m * 16) * ROWB, 0, CFG_SO);
                    }
            }
            if (2 * c < OWNH) {
                WAIT_V(0);
                unsigned old = 0;
                if (lane == 0) old = __hip_atomic_fetch_add((LAS unsigned*)(lds + 8 * BLK) + 1, 1u, RLX_WG);
                if (rfl(old) == 7u) count_and_flag(P, myws, C_YD, 16 * ((OWNH + 1) / 2), 8 + P.rank, call, lane);
            }
        }
        STAMP(4 + 2 * round); STAMPC(24 + 2 * round);
        if (!has_next) break;
#pragma unroll
        for (int a = 0; a < 2; ++a)
#pragma unroll
            for (int b = 0; b < 2; ++b)
#pragma unroll
                for (int m = 0; m < 4; ++m)
#pragma unroll
                    for (int n = 0; n < 2; ++n) acc[a][b][m][n] = (f32x4){0.f, 0.f, 0.f, 0.f};
        c = cn; pm = pmn; baseA = nA; cB0 = nB0; cB1 = nB1;
    }
    WAIT_V(0);
    if (wr == 0) BAR;
#undef SA
#undef SB
#undef STAGE
#undef LDA
#undef LDB
#undef MMA
}

__global__ __launch_bounds__(512, 2) void fused_gemm_allreduce(Params P) {
    extern __shared__ __attribute__((aligned(16))) unsigned char shm[];
    char* myws = P.myws;
    const int b = blockIdx.x, xcd = b & 7, j = b >> 3;
    STAMP(0);
    const unsigned ctrv = __hip_atomic_load((unsigned*)(myws + C_CTR), RLX_AGENT);
    if (threadIdx.x < 64) ((LAS unsigned*)shm)[8 * BLK / 4 + threadIdx.x] = 0;
    WAIT_L(0); __builtin_amdgcn_s_barrier(); asm volatile("" ::: "memory");
    const int pi = (j - GPX) * 8 + xcd;
    if (j >= GPX && pi < NPULL) puller(P, myws, (LAS unsigned*)shm + 8 * BLK / 4, ctrv, pi);
    else {
        prepass(P, myws, (LAS unsigned char*)shm, ctrv, j < GPX ? b : GPX * 8 + pi - NPULL);
        STAMP(1);
        if (j < GPX) gemm(P, myws, (LAS unsigned char*)shm, rfl(ctrv) / NWG + 1u, xcd, j);
    }
    __syncthreads();
    STAMP(63);
    if (b == 0 && threadIdx.x < 8) __hip_atomic_store((unsigned*)(myws + C_TK + (((ctrv / NWG) & 1u) * 8 + threadIdx.x) * TKS), 0u, RLX_AGENT);
    if (threadIdx.x == 0) __hip_atomic_fetch_add((unsigned*)(myws + C_CTR), 1u, RLX_AGENT);
}
}

extern "C" void kernel_launch(int rank, int nranks, ncclComm_t comm, hipStream_t stream,
                              void* const* d_in, const int* in_sizes, int n_in,
                              void* d_out, int out_size, const int* in_dtypes, int out_dtype,
                              void* const* d_out_all, void* const* d_ws_all, size_t ws_bytes) {
    if (nranks != NRANK || (unsigned)rank >= (unsigned)NRANK || n_in < 2 || in_sizes[0] != M * KS || in_sizes[1] != KS * N || out_size != M * N || ws_bytes < WS_NEED) {
        fprintf(stderr, "kernel_launch: unexpected problem geometry\n");
        return;
    }
    static bool attr_set = false;
    if (!attr_set) { HIP_CHECK(hipFuncSetAttribute((const void*)fused_gemm_allreduce, hipFuncAttributeMaxDynamicSharedMemorySize, LDS_BYTES)); attr_set = true; }
    Params P;
    P.x = (const u16*)d_in[0]; P.w = (const u16*)d_in[1]; P.rank = rank;
    for (int p = 0; p < NRANK; ++p) { P.ws[p] = (char*)d_ws_all[p]; P.out[p] = (char*)d_out_all[p]; }
    P.myws = P.ws[rank]; P.myout = (char*)d_out;
    hipLaunchKernelGGL(fused_gemm_allreduce, dim3(NWG), dim3(512), LDS_BYTES, stream, P);
    HIP_CHECK(hipGetLastError());
}
```

```cpp
#include <hip/hip_runtime.h>
#include <rccl/rccl.h>
#include <cstdio>

#define HIP_CHECK(expr) do { hipError_t _e = (expr); if (_e != hipSuccess) { \
    fprintf(stderr, "HIP error %s at %s:%d\n", hipGetErrorString(_e), __FILE__, __LINE__); \
    return; } } while (0)

namespace {
#define LAS __attribute__((address_space(3)))
typedef unsigned short u16;
typedef short bf16x8 __attribute__((ext_vector_type(8)));
typedef float f32x4 __attribute__((ext_vector_type(4)));
typedef unsigned u32x4 __attribute__((ext_vector_type(4)));
typedef __amdgpu_buffer_rsrc_t rsrc_t;

#define CFG_OWNH 5
#define CFG_MB 13
#define CFG_FK 3
#define CFG_GPX 29
#define CFG_PPX 3
#define CFG_SIN 17
#define CFG_SO 17
#define CFG_PJ 8
#define CFG_DELAY 350
#define CFG_SPAN 1200
#define CFG_HSLEEP 64
#define CFG_PWIN 3
#define CFG_W3 2
#define CFG_W3M 0
#define CFG_EW 0
#define CFG_DED 1
#define CFG_STEAL 1
#define CFG_PEEK 0
#define CFG_GSLEEP 127
#define CFG_PRO 120
#define CFG_MARG 6
#define CFG_CPOL 16
#define CFG_TKS 128
#define CFG_GAUX 16
#define CFG_MORD 3
#define CFG_WL 0
#define CFG_MSLEEP 32
#define CFG_FSLEEP 16

constexpr int NRANK = 8, M = 4096, N = 16384, KS = 512;
constexpr int NWG = 256;
constexpr int OWNH = CFG_OWNH;
constexpr int RH = 128 - NRANK * OWNH;
constexpr int NH = OWNH + RH;
constexpr int GPX = CFG_GPX;
constexpr int MB = CFG_MB;
constexpr int NROUND = 4;
constexpr int T0 = NH - MB - 2 * GPX;
constexpr int ZT = 2 * T0, ZB = 4 * GPX - ZT;
constexpr int NPULL = 8 * CFG_PPX;
constexpr int NPRE = NWG - NPULL;
constexpr int PU = 4;
constexpr unsigned BLK = 16384;
constexpr unsigned PPB = 16 / PU;
constexpr unsigned XKK = 32 * BLK, WKK = 128 * BLK;
constexpr unsigned XBYTES = 8 * XKK, SLOT = XBYTES + 8 * WKK;
constexpr size_t STG = 1u << 20;
constexpr size_t WS_NEED = STG + 2 * (size_t)NRANK * SLOT;
constexpr unsigned OUT_BYTES = (unsigned)M * N * 2;
constexpr unsigned ROWB = N * 2;
constexpr unsigned TKS = CFG_TKS;
constexpr unsigned C_CTR = 0, C_TMO = 128, C_YD = 256, C_S1 = 384, C_F = 512, C_G = 1024, C_R = 4096, C_ARR = 5120, C_TK = 0xc0000, C_END = C_TK + 16 * TKS, C_STAMP = 65536;
__host__ __device__ constexpr int imax(int a, int b) { return a > b ? a : b; }
__host__ __device__ constexpr int pend_of(int r) {
    return r < 0 ? 0 : r == 0 ? MB : r == NROUND - 1 ? NH : imax((ZB + (r - 1) * 2 * GPX + 3) / 4 * 2, MB + (ZT + (r - 1) * 2 * GPX + 3) / 4 * 2); }
__host__ __device__ constexpr int nh_of(int r) { return pend_of(r) - pend_of(r - 1); }
__host__ __device__ constexpr unsigned gsz_of(int r) { return ((r < 2 ? 16 : 0) + nh_of(r)) * PPB; }
__host__ __device__ constexpr unsigned rstart_of(int r) { return 8 * PPB * ((r < 2 ? 16 * r : 32) + pend_of(r - 1)); }
constexpr int R0 = MB - OWNH;
constexpr int C0 = 16 + R0 + NRANK * OWNH;
constexpr int FK = CFG_FK;
constexpr int PM = MB + (nh_of(1) + 1) / 2;
constexpr int NMARK = 8 + 24 + 8 + 8;
constexpr int NYD = 16 * OWNH + 8 * ((OWNH + 1) / 2);
constexpr int N0 = FK * C0, NIMG = 8 * (32 + 128);
constexpr int PJ = CFG_PJ;
constexpr int NPUSH = PJ * NPULL / 16;
constexpr unsigned long long DEADLINE = 150000000ull;
constexpr int LDS_BYTES = 8 * BLK + 512;
constexpr unsigned NP0 = NPUSH * PPB;
constexpr unsigned NPI = rstart_of(NROUND);
constexpr unsigned YCH = OWNH * 16;
constexpr unsigned NPO = M * YCH / (64 * PU);
constexpr unsigned NPT = NPI + NPO;
constexpr unsigned SEG = NPO / 7, LAST = NPT - SEG;
constexpr unsigned TB = NP0 + 2 * NPULL;
static_assert(N0 <= NPRE && NPRE + NIMG - N0 <= 8 * NPRE, "pre-pass list");
static_assert(NPUSH * 16 == PJ * NPULL && PJ % 2 == 0 && PJ > 0 && NPUSH <= 16, "push split");
static_assert(R0 >= 0 && R0 < RH && GPX * 8 + NPULL <= NWG && 2 * MB <= GPX && (NH - MB) % 2 == 0 && T0 > 0 && T0 <= GPX && (NH + 1) / 2 * 4 <= ZB + 4 * GPX, "tiling");
static_assert(ZB >= 4 * ((OWNH + 1) / 2) && nh_of(1) > 0 && nh_of(2) > 0 && nh_of(3) > 0, "rounds");
static_assert(C_ARR + (NMARK + 1) * 1024 <= C_STAMP && C_END <= STG, "ctrl");
static_assert(CFG_PRO <= SEG, "a wave leaves its list only behind the last mark");
static_assert(gsz_of(0) >= NP0 + NPULL, "every puller wave starts on K tile 0");

template <int V> struct Int { static constexpr int value = V; };
struct Params { const u16* x; const u16* w; int rank; char* ws[NRANK]; char* out[NRANK]; };

#define RLX_AGENT __ATOMIC_RELAXED, __HIP_MEMORY_SCOPE_AGENT
#define RLX_SYS __ATOMIC_RELAXED, __HIP_MEMORY_SCOPE_SYSTEM
#define AUX_SYS 17
#define AUX_DEV 16

__device__ __forceinline__ unsigned rfl(unsigned v) { return (unsigned)__builtin_amdgcn_readfirstlane((int)v); }
__device__ __forceinline__ rsrc_t mk_rsrc(const void* p, unsigned bytes) { return __builtin_amdgcn_make_buffer_rsrc((void*)p, 0, (int)bytes, 0x00020000); }
__device__ __forceinline__ int panel_of(int h, int rank) { return h < OWNH ? RH + rank * OWNH + h : h - OWNH; }

__device__ __forceinline__ unsigned peek(rsrc_t r, unsigned off) { return (unsigned)__builtin_amdgcn_raw_ptr_buffer_atomic_add_i32(0, r, (int)off, 0, AUX_DEV); }
__device__ __forceinline__ unsigned ctr_load(char* myws) { return __builtin_amdgcn_raw_buffer_load_b32(mk_rsrc(myws, 128), C_CTR, 0, CFG_CPOL); }
__device__ __forceinline__ unsigned call_of(unsigned ctrv) { return rfl(ctrv) / NWG + 1u; }
__device__ __forceinline__ bool expired(unsigned* tmo, unsigned call, unsigned long long t0) {
    if (rfl(__hip_atomic_load(tmo, RLX_AGENT)) == call || __builtin_amdgcn_s_memrealtime() - t0 > DEADLINE) {
        __hip_atomic_store(tmo, call, RLX_AGENT);
        return true;
    }
    return false;
}
template <int SLEEP> __device__ __forceinline__ bool wait_local(const unsigned* p, unsigned target, unsigned* tmo, unsigned call) {
    const unsigned long long t0 = __builtin_amdgcn_s_memrealtime();
    unsigned polls = 0;
    while ((int)(rfl(__hip_atomic_load(p, RLX_AGENT)) - target) < 0) {
        if ((++polls & 15u) == 0u && expired(tmo, call, t0)) return false;
        __builtin_amdgcn_s_sleep(SLEEP);
    }
    asm volatile("" ::: "memory");
    return true;
}

__device__ __forceinline__ bool count_and_flag(const Params& P, char* myws, unsigned cnt, unsigned total, unsigned word, unsigned call, int lane) {
    unsigned old = 0;
    if (lane == 0) old = __hip_atomic_fetch_add((unsigned*)(myws + cnt), 1u, RLX_AGENT);
    const bool last = rfl(old) + 1u == call * total;
    if (last && lane < NRANK) __hip_atomic_store((unsigned*)(P.ws[lane] + C_F) + word, call, RLX_SYS);
    return last;
}

#define WST(i, val) do {} while (0)
#define WDECL do {} while (0)
#define WDUMP do {} while (0)
#define STAMPC(i) do {} while (0)
#define STAMP(i) do {} while (0)
#define STAMPW(i) do {} while (0)

__device__ __forceinline__ unsigned prepass(const Params& P, char* myws, LAS unsigned char* lds, int u) {
    const int tid = threadIdx.x, lane = tid & 63, wave = (int)rfl(tid >> 6), half = wave & 1, pair = wave >> 1;
    const int kgl = lane >> 4, ng = lane & 15, sb = lane * 16, swz = sb ^ (((sb >> 9) & 1) << 5);
    const unsigned wlo = (unsigned)((half * 4 + kgl) * 8 * N + ng * 8) * 2u, xlo = (unsigned)((swz >> 6) * KS + half * 32) * 2u + (unsigned)(swz & 63);
    LAS unsigned char* lb = lds + pair * BLK + half * 1024;
    const unsigned ioff = (unsigned)((ng >> 2) * 4096 + (ng & 3) * 256 + ((kgl * 16) ^ (((ng >> 1) & 1) << 5)));
    LAS unsigned* cnt = (LAS unsigned*)(lds + 8 * BLK);
    char* const wsl = P.ws[lane & 7];
    WDECL; WST(0, 0); int wsi = 1;
    const unsigned long long tgo = __builtin_amdgcn_s_memrealtime() + CFG_DELAY;
    for (int i = pair * NPRE + u; i < NPRE + NIMG - N0; i += 4 * NPRE) {
        int kk, img;
        if (i < NPRE + (8 - FK) * C0) { const int v = i < NPRE ? i : i - NPRE + N0; if (i >= N0 && i < NPRE) continue;
            kk = v / C0; const int bi = v % C0; img = bi < 16 ? bi : bi < 16 + R0 ? bi + 16 : bi + 16 + (RH - R0); }
        else { const int v = i - (NPRE + (8 - FK) * C0); kk = v / (160 - C0); const int ci = v % (160 - C0); img = ci < 16 ? 16 + ci : 16 + R0 + ci; }
        if (i >= NPRE) while (__builtin_amdgcn_s_memrealtime() < tgo + (unsigned)((i - NPRE) * CFG_SPAN / (NIMG - N0))) __builtin_amdgcn_s_sleep(4);
        const bool isw = img >= 32;
        const unsigned so = isw ? wlo + (unsigned)(kk * 64 * N + (img - 32) * 128) * 2u : xlo + (unsigned)(img * 128 * KS + kk * 64) * 2u;
        const unsigned sstep = isw ? ROWB : 16 * KS * 2;
        const unsigned dbase = isw ? XBYTES + (unsigned)kk * WKK + (unsigned)(img - 32) * BLK : (unsigned)kk * XKK + (unsigned)img * BLK;
        const rsrc_t ri = mk_rsrc(isw ? P.w : P.x, isw ? KS * N * 2 : M * KS * 2);
        u32x4 in[8];
        const unsigned ctrv = ctr_load(myws);
#pragma unroll
        for (int j = 0; j < 8; ++j) in[j] = __builtin_amdgcn_raw_buffer_load_b128(ri, so + j * sstep, 0, 0);
        if (isw) {
#pragma unroll
            for (int v8 = 0; v8 < 8; ++v8) {
                u32x4 o;
#pragma unroll
                for (int e = 0; e < 4; ++e) o[e] = __builtin_amdgcn_perm(in[2 * e + 1][v8 >> 1], in[2 * e][v8 >> 1], (v8 & 1) ? 0x07060302u : 0x05040100u);
                *(LAS u32x4*)(lb + ioff + (v8 >> 2) * 2048 + (v8 & 3) * 64) = o;
            }
            asm volatile("s_waitcnt lgkmcnt(0)" ::: "memory");
#pragma unroll
            for (int j = 0; j < 8; ++j) in[j] = *(LAS u32x4*)(lb + j * 2048 + lane * 16);
        }
        const unsigned parity = call_of(ctrv) & 1u;
        const rsrc_t rs = mk_rsrc(myws + STG + (size_t)(parity * NRANK + P.rank) * SLOT, SLOT);
#pragma unroll
        for (int j = 0; j < 8; ++j) __builtin_amdgcn_raw_buffer_store_b128(in[j], rs, dbase + (unsigned)(j * 2048 + half * 1024 + lane * 16), 0, AUX_SYS);
        if (i < NPRE) {
            asm volatile("s_waitcnt vmcnt(0)" ::: "memory");
            unsigned second = 0;
            if (lane == 0) second = __hip_atomic_fetch_add(cnt, 1u, __ATOMIC_RELAXED, __HIP_MEMORY_SCOPE_WORKGROUP);
            if (rfl(second) == 1u && i >= NPUSH && lane < 7 && (img < 32 + RH || (img - 32 - RH) / OWNH == ((P.rank + 1 + lane) & 7)))
                __hip_atomic_fetch_add((unsigned*)(wsl + C_G + (P.rank * 3 + (i < C0 ? 0 : 1)) * 128), 1u, RLX_SYS);
        }
    }
    WDUMP;
    const unsigned call = call_of(ctr_load(myws));
    asm volatile("s_waitcnt vmcnt(0)" :: "s"(call) : "memory");
    __syncthreads();
    if (wave == 0 && count_and_flag(P, myws, C_S1, NPRE, P.rank, call, lane) && lane < 8)
        __hip_atomic_store((unsigned*)(myws + C_ARR + NMARK * 1024 + lane * 128), call * NPULL, RLX_AGENT);
    if (u >= GPX * 8 && tid == 64) __hip_atomic_fetch_add((unsigned*)(myws + C_CTR), 1u, RLX_AGENT);
    if (u == 0 && tid >= 504) __hip_atomic_store((unsigned*)(myws + C_TK + (((call + 1u) & 1u) * 8 + (tid & 7)) * TKS), 0u, RLX_AGENT);
    return call;
}

__device__ __forceinline__ unsigned in_piece(unsigned n, int rank) {
    int r = 0;
#pragma unroll
    for (int i = 1; i < NROUND; ++i) if (n >= rstart_of(i)) r = i;
    unsigned kk = 0, pi = 0; int h0 = 0;
#pragma unroll
    for (int i = 0; i < NROUND; ++i) if (r == i) { const unsigned q = n - rstart_of(i); kk = q / gsz_of(i); pi = q % gsz_of(i); h0 = pend_of(i - 1); }
    if (r < 2 && pi < 16 * PPB) return kk * XKK + (unsigned)r * (16 * BLK) + pi * (PU * 1024);
    if (r < 2) pi -= 16 * PPB;
    return XBYTES + kk * WKK + (unsigned)panel_of(h0 + (int)(pi / PPB), rank) * BLK + (pi % PPB) * (PU * 1024);
}
__device__ __forceinline__ unsigned mark_end(int G) {
    if (G < 8) return (unsigned)(G + 1) * gsz_of(0);
    if (G < 32) { const unsigned kk = (unsigned)(G - 8) / 3u, sub = (unsigned)(G - 8) % 3u;
        return rstart_of(1) + kk * gsz_of(1) + (sub == 0 ? 16 * PPB : sub == 1 ? (16 + PM - MB) * PPB : gsz_of(1)); }
    return G < 40 ? rstart_of(2) + (unsigned)(G - 31) * gsz_of(2) : rstart_of(3) + (unsigned)(G - 39) * gsz_of(3);
}

#define WAIT_V(n) asm volatile("s_waitcnt vmcnt(" #n ")" ::: "memory")
#define WAIT_L(n) asm volatile("s_waitcnt lgkmcnt(" #n ")" ::: "memory")
#define RLX_WG __ATOMIC_RELAXED, __HIP_MEMORY_SCOPE_WORKGROUP
__device__ __forceinline__ void arrive(char* myws, LAS unsigned* lctr, int g, int lane) {
    unsigned old = 0;
    if (lane == 0) old = __hip_atomic_fetch_add(lctr + 64 + g, 1u, RLX_WG);
    if (rfl(old) == (g == 0 ? 7u : 6u)) {
        if (lane < 8 || (lane == 8 && g == 32)) __hip_atomic_fetch_add((unsigned*)(myws + (lane < 8 ? C_ARR + g * 1024 + lane * 128 : C_CTR)), 1u, RLX_AGENT);
        STAMPW(2 + g);
    }
}
__device__ __forceinline__ void puller(const Params& P, char* myws, LAS unsigned* lctr, int idx) {
    const int tid = threadIdx.x, lane = tid & 63, wave = (int)rfl(tid >> 6);
    unsigned* tmo = (unsigned*)(myws + C_TMO);
    int link = wave; bool ded = false;
    { const int sl = (idx >> 3) * 4 + (idx & 3), h = (idx >> 2) & 1, r = (sl - 7) * 2 + h;
      if (sl < 7) { link = sl; idx = h * 7 + wave; ded = true; } else if (r < 7) { link = r; idx = 14 + wave; ded = true; } else idx = 21 + r - 7; }
    if (wave == 7) {
        const unsigned call = call_of(ctr_load(myws)), p = (unsigned)(P.rank + 1 + lane) & 7u;
        unsigned st = lane < 7 && (!ded || lane == link) ? 0u : 5u;
        bool counted = false;
        const rsrc_t rc = mk_rsrc(myws, C_END);
        const unsigned long long t0 = __builtin_amdgcn_s_memrealtime();
        for (unsigned polls = 1;; ++polls) {
            if (st < 5u) {
                const unsigned fw = st < 3u ? C_G + (p * 3 + st) * 128 : C_F + ((st - 3u) * 8 + p) * 4;
                const unsigned ft = call * (unsigned)(st == 0u ? 16 + MB - NPUSH : st == 1u ? (FK - 1) * (16 + MB) : st == 2u ? NPULL : 1);
                if ((int)((CFG_PEEK ? peek(rc, fw) : __hip_atomic_load((const unsigned*)(myws + fw), RLX_SYS)) - ft) >= 0) { ++st; __hip_atomic_store(lctr + 32 + lane, st, RLX_WG); }
            }
            if ((polls & 15u) == 0u && expired(tmo, call, t0)) { st = 6u; if (lane < 7) __hip_atomic_store(lctr + 32 + lane, st, RLX_WG); }
            if (!counted && __all(st >= 3u)) { counted = true; arrive(myws, lctr, 0, lane); }
            if (__all(st >= 5u)) break;
            if (__all(st >= 2u)) __builtin_amdgcn_s_sleep(CFG_HSLEEP); else if (__all(st >= 1u)) __builtin_amdgcn_s_sleep(8); else __builtin_amdgcn_s_sleep(1);
        }
        const int q = lane >> 3, l = lane & 7;
        const unsigned fo = q == 7 ? C_END : l == 7 ? C_TK + ((call & 1u) * 8 + (unsigned)q) * TKS : ded && q != link ? C_END : C_R + (unsigned)(q * 128 + l * 4);
        for (unsigned polls = 1;; ++polls) {
            unsigned v = peek(rc, fo);
            const unsigned long long have = __ballot(l < 7 && (int)(v - call * NPULL) >= 0);
            v += TB;
            if (__any(st == 6u) || ((polls & 15u) == 0u && expired(tmo, call, t0))) v = NPT;
            if (l == 7 && q < 7) { __hip_atomic_store(lctr + 48 + q, (unsigned)(have >> (8 * q)) & 127u, RLX_WG); __hip_atomic_fetch_max(lctr + 40 + q, v, RLX_WG); }
            if (__all(l < 7 || q == 7 || v >= NPT)) break;
            if (__any(l == 7 && q < 7 && v + CFG_PRO + 120 >= NPT)) __builtin_amdgcn_s_sleep(CFG_FSLEEP); else __builtin_amdgcn_s_sleep(127);
        }
        return;
    }
    WDECL; WST(0, 0);
    const unsigned p = (unsigned)(P.rank + 1 + link) & 7u;
    char* const pws = P.ws[link];
    unsigned ctrv;
    {
        const rsrc_t rx = mk_rsrc(P.x, M * KS * 2);
        const int sb = lane * 16, swz = sb ^ (((sb >> 9) & 1) << 5), c0 = idx * PJ;
        u32x4 in[PJ];
        const unsigned so = (unsigned)((c0 * 8 + (swz >> 6)) * KS) * 2u + (unsigned)(swz & 63);
        asm volatile("buffer_load_dword %0, off, %1, 0 sc1" : "=v"(ctrv) : "s"(mk_rsrc(myws + C_CTR, 4)) : "memory");
#pragma unroll
        for (int j = 0; j < PJ; ++j) asm volatile("buffer_load_dwordx4 %0, %1, %2, %3 offen offset:%4" : "=v"(in[j]) : "v"(so), "s"(rx), "s"((j >> 1) * (16 * KS * 2)), "n"((j & 1) * 64) : "memory");
        asm volatile("s_waitcnt vmcnt(%1)" : "+v"(ctrv) : "n"(PJ) : "memory");
        const rsrc_t rp = mk_rsrc(pws + STG + (size_t)((call_of(ctrv) & 1u) * NRANK + P.rank) * SLOT, XKK);
        WST(60, link);
#pragma unroll
        for (int j = 0; j < PJ; ++j) {
            asm volatile("s_waitcnt vmcnt(%1)" : "+v"(in[j]) : "n"(j < CFG_PWIN ? PJ - 1 : CFG_PWIN - 1) : "memory");
            if (j == 0) WST(1, link);
            __builtin_amdgcn_raw_buffer_store_b128(in[j], rp, (unsigned)((c0 + j) * 1024 + sb), 0, AUX_SYS);
        }
    }
    const unsigned call = call_of(ctrv), parity = call & 1u;
    const size_t soff = STG + (size_t)(parity * NRANK + p) * SLOT;
    char* const srcI = pws + soff; char* const dstI = myws + soff;
    char* const srcO = P.out[link]; char* const dstO = P.out[7];
    const rsrc_t rctl = mk_rsrc(myws, C_END);
    const unsigned tkoff = lane == 0 ? C_TK + parity * 8 * TKS : C_END;
    int gsig = 0;
    auto signal = [&](unsigned n) {
        while (gsig < NMARK && mark_end(gsig) <= n) { arrive(myws, lctr, gsig, lane); ++gsig; }
        while (gsig >= NMARK && gsig < NMARK + 6 && LAST - (unsigned)(NMARK + 5 - gsig) * SEG <= n) {
            const int vl = (link + 1 + gsig - NMARK) % 7, rl = (link - vl - 1) & 7;
            if (lane == 0) __hip_atomic_fetch_add((unsigned*)(P.ws[rl] + C_R + (6 - rl) * 128 + vl * 4), 1u, RLX_SYS);
            ++gsig;
        }
    };
    auto decode = [&](unsigned n, int l, unsigned (&off)[PU]) {
        if (n < NPI) { const unsigned o = in_piece(n, P.rank);
#pragma unroll
            for (int u = 0; u < PU; ++u) off[u] = o + u * 1024 + lane * 16; }
        else { unsigned q = n - NPI + (unsigned)l * SEG; if (q >= NPO) q -= NPO; if (n >= NPT) q = NPO;
            const unsigned ycol = (unsigned)(RH + ((P.rank + 1 + l) & 7) * OWNH) * 256u;
#pragma unroll
            for (int u = 0; u < PU; ++u) { const unsigned ci = (q * PU + u) * 64 + lane; off[u] = (ci / YCH) * ROWB + ycol + (ci % YCH) * 16; } }
    };
    auto pick = [&]() {
        unsigned k = lane < 7 ? __hip_atomic_load(lctr + 40 + lane, RLX_WG) : NPT;
        const unsigned m = rfl(__hip_atomic_load(lctr + 48 + link, RLX_WG));
        k = k < NPT ? k : NPT;
        int best = __all(k >= NPT) ? 8 : 7; unsigned bk = NPT, sum = 0;
#pragma unroll
        for (int l = 0; l < 7; ++l) { const unsigned kl = (unsigned)__builtin_amdgcn_readlane((int)k, l); sum += kl; if (((m >> l) & 1u) && kl >= LAST && kl < bk) { best = l; bk = kl; } }
        const unsigned ko = (unsigned)__builtin_amdgcn_readlane((int)k, link);
        return ko < NPT && (ko * 7u <= sum + 7u * CFG_MARG || ko <= bk) ? link : best;
    };
    unsigned stage = 0;
    auto await = [&](unsigned need) {
        const unsigned long long t0 = __builtin_amdgcn_s_memrealtime();
        unsigned polls = 0;
        while ((stage = rfl(__hip_atomic_load(lctr + 32 + link, RLX_WG))) < need) {
            if ((++polls & 1023u) == 0u && expired(tmo, call, t0)) { stage = 6; break; }
            __builtin_amdgcn_s_sleep(1);
        }
        asm volatile("" ::: "memory");
    };
    unsigned n = NP0 + (unsigned)idx;
    unsigned off[PU];
    decode(n, link, off);
    rsrc_t rs = mk_rsrc(srcI, SLOT), rd = mk_rsrc(dstI, SLOT);
    WST(2, 0);
    await(1);
    WST(3, 0);
    int wsi = 5;
    bool ok = stage != 6u;
#define LD16(dst, off, rs) asm volatile("buffer_load_dwordx4 %0, %1, %2, 0 offen sc0 sc1" : "=v"(dst) : "v"(off), "s"(rs) : "memory")
#define LANDED(u, cnt) asm volatile("s_waitcnt vmcnt(%1)" : "+v"(R[u]) : "n"(cnt) : "memory")
#define ST16(u) __builtin_amdgcn_raw_buffer_store_b128(ok ? R[u] : (u32x4){0x7fc07fc0u, 0x7fc07fc0u, 0x7fc07fc0u, 0x7fc07fc0u}, rd, off[u], 0, CFG_SIN)
    auto run = [&](auto wc) {
        constexpr int W = decltype(wc)::value;
        u32x4 R[4];
        unsigned tk = n + NPULL - TB, spins = 0;
        int cl = link;
        bool own = true;
        const unsigned long long t0 = __builtin_amdgcn_s_memrealtime();
#pragma unroll
        for (int u = 0; u < W; ++u) LD16(R[u], off[u], rs);
        WAIT_V(0);
        WST(4, n);
        if (lane == 0) __hip_atomic_fetch_add((unsigned*)(pws + C_G + (P.rank * 3 + 2) * 128), 1u, RLX_SYS);
        for (;;) {
            LANDED(0, 2 * W - 1);
            LD16(R[W], off[W], rs);
            ST16(0);
            asm volatile("" : "+v"(tk) :: "memory");
            const unsigned nn = cl < 7 ? rfl(tk) + TB : NPT;
            const int nl = cl;
            if (nn + (CFG_STEAL ? CFG_PRO : 0) >= NPT || cl != link) {
                if (cl < 7) { if (lane == 0 && nn + 1u >= NPT) __hip_atomic_store(lctr + 40 + cl, NPT, RLX_WG); }
                else __builtin_amdgcn_s_sleep(4);
                cl = pick();
                if (cl == 7 && (++spins & 1023u) == 0u && expired(tmo, call, t0)) cl = 8;
            }
            tk = 1u;
            asm volatile("buffer_atomic_add %0, %1, %2, 0 offen sc0" : "+v"(tk) : "v"(cl < 7 ? tkoff + (unsigned)cl * TKS : C_END), "s"(rctl) : "memory");
            unsigned offn[4];
            decode(nn, nl, offn);
            const unsigned need = nn < gsz_of(0) ? 1u : nn < FK * gsz_of(0) ? 2u : nn < NPI ? 4u : nn < NPT && nl == link ? 5u : 0u;
            if (stage < need) stage = rfl(__hip_atomic_load(lctr + 32 + link, RLX_WG));
            const rsrc_t rsn = mk_rsrc(nn < NPI ? srcI : srcO, nn < NPI ? SLOT : OUT_BYTES);
#pragma unroll
            for (int u = 1; u < 4; ++u) {
                if (u == 4 - W && stage < need) {
#pragma unroll
                    for (int v = u; v < 4; ++v) { LANDED(v, 0); ST16(v); }
                    WAIT_V(0);
                    signal(nn);
                    await(need);
                }
                LANDED(u, 2 * W - 1 + (u <= W ? 1 : 0));
                if (u + W < 4) LD16(R[u + W], off[u + W], rs); else LD16(R[u + W - 4], offn[u + W - 4], rsn);
                ST16(u);
            }
            if (n < NPT && (wsi < 40 || n + 12 * NPULL >= NPT || !own)) { WST(wsi, own ? n : n | 0x80000000u); if (wsi < 59) ++wsi; }
            if (nn >= NPT && cl == 8) break;
            signal(own ? n : NPT);
            own = nl == link;
            if (stage == 6u) ok = false;
            n = nn; rs = rsn;
            rd = mk_rsrc(n < NPI ? dstI : dstO, n < NPI ? SLOT : OUT_BYTES);
#pragma unroll
            for (int u = 0; u < 4; ++u) off[u] = offn[u];
        }
#pragma unroll
        for (int u = 0; u < 4; ++u) LANDED(u, 0);
        asm volatile("" : "+v"(tk) :: "memory");
    };
    if (ded ? wave < CFG_W3 : CFG_W3M) run(Int<3>{}); else run(Int<2>{});
    signal(NPT);
    WST(61, 0); WDUMP;
#undef LD16
#undef LANDED
#undef ST16
}

__device__ __forceinline__ unsigned cvt_pk_bf16(float lo, float hi) { unsigned r; asm volatile("v_cvt_pk_bf16_f32 %0, %1, %2" : "=v"(r) : "v"(lo), "v"(hi)); return r; }
__device__ __forceinline__ float gelu(float y) {
    const float k1 = -2.0f * 0.7978845608028654f * 1.4426950408889634f, k3 = k1 * 0.044715f;
    const float e = __builtin_amdgcn_exp2f(y * __builtin_fmaf(y * y, k3, k1));
    return y * __builtin_amdgcn_rcpf(1.0f + e);
}

__device__ __forceinline__ void gemm(const Params& P, char* myws, LAS unsigned char* lds, unsigned call, int xcd, int j) {
    const int tid = threadIdx.x, wid = (int)rfl(tid >> 6), lane = tid & 63, wr = wid >> 2, wc = wid & 3, fr = lane & 15, fq = lane >> 4;
    auto tile_of = [&](int round) {
        int q = xcd & 3, z; const int par = xcd >> 2;
        if (round > 1) z = (q >= 2 ? ZB : ZT) + ((round - 2) * 2 + par) * GPX + j;
        else if (q >= 2) z = par * GPX + j;
        else if (j < T0) z = par * T0 + j;
        else { q += 2; z = 2 * GPX + par * (GPX - T0) + (j - T0); }
        return q * 4 + (z & 3) + (((q >= 2 ? 0 : MB) + (z >> 2) * 2) << 4);
    };
    unsigned tabA, tabB;
    { const int u = lane - 8, kk = lane < 8 ? lane : u / 7, sl = lane < 8 ? P.rank : (P.rank + 1 + u - 7 * kk) & 7;
      tabA = (unsigned)sl * SLOT + (unsigned)kk * XKK; tabB = (unsigned)sl * SLOT + XBYTES + (unsigned)kk * WKK; }
#define KA(t) (unsigned)__builtin_amdgcn_readlane((int)tabA, (t))
#define KB(t) (unsigned)__builtin_amdgcn_readlane((int)tabB, (t))
    int pm = tile_of(1) & 15, h0 = tile_of(1) >> 4;
    unsigned* tmo = (unsigned*)(myws + C_TMO);
    const rsrc_t rs = mk_rsrc(myws + STG + (size_t)(call & 1u) * NRANK * SLOT, NRANK * SLOT);
    const unsigned arr = C_ARR + (unsigned)xcd * 128u;
    const rsrc_t rctl = mk_rsrc(myws, C_END);
    const unsigned voff = (unsigned)(wid * 2048 + lane * 16);
    LAS unsigned char* const ldw = lds + wid * 2048;
    const unsigned atarget = call * NPULL;
    int aoff, boff;
    { const int ob = fr * 64 + fq * 16, sw = ob ^ (((ob >> 9) & 1) << 5); aoff = wr * 8192 + sw; boff = wc * 4096 + sw + 4 * BLK; }
    asm volatile("" : "+v"(boff));
    bool ok = true;
#define SA(b, h) (((b) * 2 + (h)) * BLK)
#define SB(b, h) ((4 + (b) * 2 + (h)) * BLK)
#define STAGE(bufoff, so) do { __builtin_amdgcn_raw_ptr_buffer_load_lds(rs, (LAS void*)(ldw + (bufoff)), 16, (int)voff, (int)(so), 0, CFG_GAUX); \
        __builtin_amdgcn_raw_ptr_buffer_load_lds(rs, (LAS void*)(ldw + (bufoff)), 16, (int)voff, (int)(so), 1024, CFG_GAUX); } while (0)
#define LDA(dst, off) do { _Pragma("unroll") for (int m = 0; m < 4; ++m) _Pragma("unroll") for (int k = 0; k < 2; ++k) dst[m][k] = *(const LAS bf16x8*)(lds + (off) + aoff + m * 2048 + k * 1024); } while (0)
#define LDB(dst, off) do { _Pragma("unroll") for (int n = 0; n < 2; ++n) _Pragma("unroll") for (int k = 0; k < 2; ++k) dst[n][k] = *(const LAS bf16x8*)(lds + ((off) - 4 * (int)BLK) + boff + n * 2048 + k * 1024); } while (0)
#define MMA(C, At, Bt, FILL) do { __builtin_amdgcn_s_setprio(1); _Pragma("unroll") for (int _q = 0; _q < 16; ++_q) { \
        const int m = CFG_MORD == 5 ? _q >> 2 : CFG_MORD == 6 ? ((_q >> 1) & 3) ^ (_q >> 3) * 3 : (_q >> 1) & 3; \
        const int n = CFG_MORD == 5 ? (_q >> 1) & 1 : CFG_MORD == 1 ? _q & 1 : _q >> 3; \
        const int k = CFG_MORD == 1 ? _q >> 3 : CFG_MORD == 7 ? (_q ^ (_q >> 1)) & 1 : _q & 1; \
        C[m][n] = __builtin_amdgcn_mfma_f32_16x16x32_bf16(Bt[n][k], At[m][k], C[m][n], 0, 0, 0); SCHED; FILL; SCHED; } __builtin_amdgcn_s_setprio(0); } while (0)
#define BAR __builtin_amdgcn_s_barrier()
#define SCHED __builtin_amdgcn_sched_barrier(0)
#define PIN(x) asm volatile("" : "+s"(x))
    f32x4 acc[2][2][4][2];
#pragma unroll
    for (int a = 0; a < 2; ++a)
#pragma unroll
        for (int b = 0; b < 2; ++b)
#pragma unroll
            for (int m = 0; m < 4; ++m)
#pragma unroll
                for (int n = 0; n < 2; ++n) acc[a][b][m][n] = (f32x4){0.f, 0.f, 0.f, 0.f};
    bf16x8 At[4][2], B0[2][2], B1[2][2];
    if (wid == 0) ok = wait_local<CFG_GSLEEP>((const unsigned*)(myws + arr + NMARK * 1024), atarget, tmo, call);
    asm volatile("" ::: "memory"); BAR; asm volatile("" ::: "memory");
    STAMP(2); STAMPC(22);
    const rsrc_t ro = mk_rsrc(P.out[7], OUT_BYTES);
    auto put = [&](f32x4 (&a)[4][2], int rowp, int h) {
        const unsigned o0 = (unsigned)(rowp * 128 + wr * 64 + fr) * ROWB + (unsigned)(wc * 32 + fq * 8) * 2u + (h < NH ? (unsigned)panel_of(h, P.rank) * 256u : OUT_BYTES);
#pragma unroll
        for (int m = 0; m < 4; ++m) {
            const f32x4 v0 = a[m][0], v1 = a[m][1];
            u32x4 o;
            o.x = cvt_pk_bf16(gelu(v0[0]), gelu(v0[1])); o.y = cvt_pk_bf16(gelu(v0[2]), gelu(v0[3]));
            o.z = cvt_pk_bf16(gelu(v1[0]), gelu(v1[1])); o.w = cvt_pk_bf16(gelu(v1[2]), gelu(v1[3]));
            if (!ok) o = (u32x4){0x7fc07fc0u, 0x7fc07fc0u, 0x7fc07fc0u, 0x7fc07fc0u};
            __builtin_amdgcn_raw_buffer_store_b128(o, ro, o0 + (unsigned)(m * 16) * ROWB, 0, AUX_SYS);
            a[m][0] = a[m][1] = (f32x4){0.f, 0.f, 0.f, 0.f};
        }
    };
    auto publish = [&]() {
        WAIT_V(0);
        unsigned old = 0;
        if (lane == 0) old = __hip_atomic_fetch_add((LAS unsigned*)(lds + 8 * BLK) + 1, 1u, RLX_WG);
        if ((rfl(old) & 7u) == 7u) count_and_flag(P, myws, C_YD, NYD, 8 + P.rank, call, lane);
    };
    unsigned pA = (unsigned)pm * 2 * BLK, pB0 = (unsigned)panel_of(h0, P.rank) * BLK, pB1 = (unsigned)panel_of(h0 + 1 < NH ? h0 + 1 : h0, P.rank) * BLK;
    unsigned pollv = 0, sa = 0, sa1 = 0, sb0 = 0, sb1 = 0, gw = 0, gstep = 0;
    int tn = 0;
    auto gate = [&](int s) {
        if (__builtin_expect(s + 3 >= tn, 0)) {
            if (s + 3 == tn) { if (wid == 0) asm volatile("s_nop 4\n\tbuffer_load_dword %0, off, %1, %2 sc1" : "+v"(pollv) : "s"(rctl), "s"(gw) : "memory"); }
            else {
                if (wid == 0 && (int)(rfl(pollv) - atarget) < 0) ok = wait_local<CFG_GSLEEP>((const unsigned*)(myws + gw), atarget, tmo, call) && ok;
                tn = tn < 57 ? tn + 7 : 99; gw += gstep;
            }
        }
    };
#define WAIT_LX do {} while (0)
#define WAIT_P asm volatile("s_waitcnt vmcnt(6)" : "+v"(pollv) :: "memory")
    for (int round = 0;; ++round) {
        const int tnext = tile_of(round + 1), pmn = tnext & 15, h0n = tnext >> 4;
        const bool has_next = round < NROUND - 1 && h0n < NH;
        const unsigned nA = has_next ? (unsigned)pmn * 2 * BLK : pA;
        const unsigned nB0 = has_next ? (unsigned)panel_of(h0n, P.rank) * BLK : pB0;
        const unsigned nB1 = has_next ? (unsigned)panel_of(h0n + 1 < NH ? h0n + 1 : h0n, P.rank) * BLK : pB1;
        const int tb = round == 1 ? 8 : 0, tl = round == 0 ? 6 : 62;
        tn = round == 0 ? 99 : 8;
        gstep = round == 1 ? 3072u : 1024u;
        gw = arr + (round == 1 ? (h0 + 1 < MB ? 8u : h0 + 1 < PM ? 9u : 10u) : (unsigned)round * 8u + 16u) * 1024u;
        if (round < 2) {
            if (round == 1) {
                if (wid == 0) ok = wait_local<CFG_MSLEEP>((const unsigned*)(myws + gw), atarget, tmo, call) && ok;
                asm volatile("" ::: "memory"); BAR; asm volatile("" ::: "memory");
                tn = 15; gw += gstep;
            }
            STAGE(SB(0, 0), pB0 + KB(tb)); STAGE(SA(0, 0), pA + KA(tb)); STAGE(SB(0, 1), pB1 + KB(tb)); STAGE(SA(0, 1), pA + KA(tb) + BLK);
            if (wr == 1) BAR;
            WAIT_V(4); BAR;
            STAGE(SB(1, 0), pB0 + KB(tb + 1)); STAGE(SA(1, 0), pA + KA(tb + 1)); STAGE(SB(1, 1), pB1 + KB(tb + 1));
            sa1 = pA + KA(tb + 1) + BLK;
            WAIT_V(6); BAR;
            STAMP(10);
            LDB(B0, SB(0, 0));
        }
        for (int t = tb;; t += 2) {
            int t2 = t + 2;
            LDA(At, SA(0, 0)); STAGE(SA(1, 1), sa1);
            BAR; WAIT_LX; SCHED;
            MMA(acc[0][0], At, B0,
                if (_q == 0) { PIN(t2); if (t == tl) { t2 = 0; pA = nA; } PIN(t2); PIN(pA); }
                if (_q == 1) { if (t2 == 0) { pB0 = nB0; pB1 = nB1; } PIN(pB0); PIN(pB1); }
                if (_q == 2) { sb1 = KB(t2); sa = KA(t2); PIN(sb1); PIN(sa); }
                if (_q == 4) { sb0 = sb1 + pB0; sb1 += pB1; PIN(sb0); PIN(sb1); }
                if (_q == 5) { sa += pA; sa1 = sa + BLK; PIN(sa); PIN(sa1); }
                if (_q == 8) gate(t));
            BAR; SCHED;
            LDB(B1, SB(0, 1)); STAGE(SB(0, 0), sb0);
            BAR; WAIT_LX; SCHED; MMA(acc[0][1], At, B1, ); BAR; SCHED;
            LDA(At, SA(0, 1)); STAGE(SA(0, 0), sa);
            WAIT_V(10); BAR; WAIT_LX; SCHED; MMA(acc[1][0], At, B0, ); BAR; SCHED;
            LDB(B0, SB(1, 0)); STAGE(SB(0, 1), sb1);
            WAIT_P; BAR; SCHED; MMA(acc[1][1], At, B1, ); WAIT_L(0); BAR; SCHED;
            LDA(At, SA(1, 0)); STAGE(SA(0, 1), sa1);
            BAR; WAIT_LX; SCHED;
            MMA(acc[0][0], At, B0,
                if (_q == 0) { PIN(t2); ++t2; PIN(t2); }
                if (_q == 2) { sb1 = KB(t2); sa = KA(t2); PIN(sb1); PIN(sa); }
                if (_q == 4) { sb0 = sb1 + pB0; sb1 += pB1; PIN(sb0); PIN(sb1); }
                if (_q == 5) { sa += pA; sa1 = sa + BLK; PIN(sa); PIN(sa1); }
                if (_q == 8) gate(t + 1));
            BAR; SCHED;
            LDB(B1, SB(1, 1)); STAGE(SB(1, 0), sb0);
            BAR; WAIT_LX; SCHED; MMA(acc[0][1], At, B1, ); BAR; SCHED;
            LDA(At, SA(1, 1)); STAGE(SA(1, 0), sa);
            WAIT_V(10); BAR; WAIT_LX; SCHED; MMA(acc[1][0], At, B0, ); BAR; SCHED;
            LDB(B0, SB(0, 0)); STAGE(SB(1, 1), sb1);
            WAIT_P; BAR; SCHED; MMA(acc[1][1], At, B1, ); WAIT_L(0); BAR; SCHED;
            if (t2 == 1) break;
        }
        if (round == 0) {
            WAIT_V(0);
            if (wr == 0) BAR;
            STAMP(11);
            if (j < 2 * MB) {
                const unsigned mA = (unsigned)(xcd * 2 + (j & 1)) * BLK, mB = (unsigned)panel_of(j >> 1, P.rank) * BLK;
                f32x4 mc[4][2];
#pragma unroll
                for (int m = 0; m < 4; ++m) mc[m][0] = mc[m][1] = (f32x4){0.f, 0.f, 0.f, 0.f};
#define MSTAGE(t) do { const int _k = (t) < 64 ? (t) : 63; STAGE(((t) & 3) * 2 * BLK, mA + KA(_k)); STAGE((((t) & 3) * 2 + 1) * BLK, mB + KB(_k)); } while (0)
                MSTAGE(0); MSTAGE(1); MSTAGE(2);
                tn = 8; gw = arr;
                WAIT_V(8); BAR;
                if (wr == 1) BAR;
                for (int t = 0; t < 64; ++t) {
                    const int lb = (t & 3) * 2 * BLK;
                    LDB(B0, lb + BLK); LDA(At, lb);
                    MSTAGE(t + 3);
                    WAIT_V(8); WAIT_L(0); BAR; SCHED;
                    MMA(mc, At, B0,
                        if (_q == 8 && t + 4 == tn) {
                            if (wid == 0) ok = (tn == 57 ? wait_local<CFG_MSLEEP>((const unsigned*)(myws + gw), atarget, tmo, call) : wait_local<CFG_GSLEEP>((const unsigned*)(myws + gw), atarget, tmo, call)) && ok;
                            tn = tn < 57 ? tn + 7 : 0; gw += 1024u;
                        });
                    BAR; SCHED;
                }
#undef MSTAGE
                WAIT_V(0);
                if (wr == 0) BAR;
                STAMP(9);
                put(mc, xcd * 2 + (j & 1), j >> 1);
                if ((j >> 1) < OWNH) publish();
            }
            continue;
        }
        STAMP(1 + 2 * round); STAMPC(21 + 2 * round);
        put(acc[0][0], pm * 2, h0); put(acc[1][0], pm * 2 + 1, h0); put(acc[0][1], pm * 2, h0 + 1); put(acc[1][1], pm * 2 + 1, h0 + 1);
        if (h0 < OWNH) publish();
        if (round == 1 && tid == 64) __hip_atomic_fetch_add((unsigned*)(myws + C_CTR), 1u, RLX_AGENT);
        STAMP(2 + 2 * round); STAMPC(22 + 2 * round);
        if (!has_next) break;
        h0 = h0n; pm = pmn;
    }
    WAIT_V(0);
    if (wr == 0) BAR;
#undef KA
#undef KB
#undef SA
#undef SB
#undef STAGE
#undef LDA
#undef LDB
#undef MMA
#undef PIN
#undef WAIT_P
}

__global__ __launch_bounds__(512, 2) void fused_gemm_allreduce(Params P) {
    extern __shared__ __attribute__((aligned(16))) unsigned char shm[];
    char* myws = P.ws[7];
    const int b = blockIdx.x, xcd = b & 7, j = b >> 3;
    STAMP(0);
    if (threadIdx.x < 128) ((LAS unsigned*)shm)[8 * BLK / 4 + threadIdx.x] = 0;
    asm volatile("" :: "s"(P.x), "s"(P.w), "s"(P.rank), "s"(myws), "s"(P.out[7]));
    WAIT_L(0); __builtin_amdgcn_s_barrier(); asm volatile("" ::: "memory");
    const int pi = (j - GPX) * 8 + xcd;
    if (j >= GPX && pi < NPULL) puller(P, myws, (LAS unsigned*)shm + 8 * BLK / 4, pi);
    else {
        const unsigned call = prepass(P, myws, (LAS unsigned char*)shm, j < GPX ? b : GPX * 8 + pi - NPULL);
        STAMP(1);
        if (j < GPX) gemm(P, myws, (LAS unsigned char*)shm, call, xcd, j);
    }
}
}

extern "C" void kernel_launch(int rank, int nranks, ncclComm_t comm, hipStream_t stream,
                              void* const* d_in, const int* in_sizes, int n_in,
                              void* d_out, int out_size, const int* in_dtypes, int out_dtype,
                              void* const* d_out_all, void* const* d_ws_all, size_t ws_bytes) {
    if (nranks != NRANK || (unsigned)rank >= (unsigned)NRANK || n_in < 2 || in_sizes[0] != M * KS || in_sizes[1] != KS * N || out_size != M * N || ws_bytes < WS_NEED) {
        fprintf(stderr, "kernel_launch: unexpected problem geometry\n");
        return;
    }
    static bool attr_set = false;
    if (!attr_set) { HIP_CHECK(hipFuncSetAttribute((const void*)fused_gemm_allreduce, hipFuncAttributeMaxDynamicSharedMemorySize, LDS_BYTES)); attr_set = true; }
    Params P;
    P.x = (const u16*)d_in[0]; P.w = (const u16*)d_in[1]; P.rank = rank;
    for (int l = 0; l < NRANK; ++l) { P.ws[l] = (char*)d_ws_all[(rank + 1 + l) % NRANK]; P.out[l] = (char*)d_out_all[(rank + 1 + l) % NRANK]; }
    P.out[7] = (char*)d_out;
    hipLaunchKernelGGL(fused_gemm_allreduce, dim3(NWG), dim3(512), LDS_BYTES, stream, P);
    HIP_CHECK(hipGetLastError());
}
```

```cpp
#include <hip/hip_runtime.h>
#include <rccl/rccl.h>
#include <cstdio>

#define HIP_CHECK(expr) do { hipError_t _e = (expr); if (_e != hipSuccess) { \
    fprintf(stderr, "HIP error %s at %s:%d\n", hipGetErrorString(_e), __FILE__, __LINE__); \
    return; } } while (0)

namespace {
#define LAS __attribute__((address_space(3)))
typedef unsigned short u16;
typedef short bf16x8 __attribute__((ext_vector_type(8)));
typedef float f32x2 __attribute__((ext_vector_type(2)));
typedef float f32x4 __attribute__((ext_vector_type(4)));
typedef unsigned u32x4 __attribute__((ext_vector_type(4)));
typedef __amdgpu_buffer_rsrc_t rsrc_t;

#define CFG_OWNH 5
#define CFG_XH 1
#define CFG_MB (13 + CFG_XH)
#define CFG_MM (CFG_MB % 2 ? 0 : 2)
#define CFG_FK 3
#define CFG_GPX 29
#define CFG_PPX 3
#define CFG_SO 0
#define CFG_PJ 4
#define CFG_DELAY 280
#define CFG_SPAN 3000
#define CFG_HSLEEP 64
#define CFG_LSLEEP 16
#define CFG_PWIN 3
#define CFG_W3 0
#define CFG_YST 1
#define CFG_PF 1
#define CFG_ZF 2
#define CFG_PO 0
#define CFG_WTR 3
#define CFG_EWL 0
#define CFG_W3M 0
#define CFG_EW 0
#define CFG_ES 4
#define CFG_DED 1
#define CFG_GSLEEP 127
#define CFG_CPOL 16
#define CFG_TKS 128
#define CFG_GAUX 16
#define CFG_MORD 6
#define CFG_NOPRIO 1
#define CFG_MSLEEP 32
#define CFG_SPEC 1
#define CFG_EZ (96 + 24 * CFG_XH)
#define CFG_TX 264
#define CFG_NPC 2
#define CFG_GT 1
#define CFG_GTN 9
#define CFG_KD 80
#define CFG_ESM 3
#define CFG_SLOT (24u << 20)
#define CFG_STG (5u << 19)
#define CFG_ROT 1

constexpr int NRANK = 8, M = 4096, N = 16384, KS = 512;
constexpr int NWG = 256;
constexpr int OWNH = CFG_OWNH;
constexpr int RH = 128 - NRANK * OWNH;
constexpr int XH = CFG_XH;
constexpr int NH = OWNH + XH + RH;
constexpr int GPX = CFG_GPX;
constexpr int MB = CFG_MB;
constexpr int NROUND = 4;
constexpr int T0 = NH - MB - 2 * GPX;
constexpr int ZT = 2 * T0, ZB = 4 * GPX - ZT;
constexpr int NPULL = 8 * CFG_PPX;
constexpr int NPRE = NWG - NPULL;
constexpr int PU = 4;
constexpr unsigned BLK = 16384;
constexpr unsigned PPB = 16 / PU, PIECE = PU * 1024;
constexpr unsigned XKK = 32 * BLK, WKK = (128 + XH) * BLK;
constexpr unsigned XBYTES = 9 * XKK, SLOT = CFG_SLOT;
constexpr size_t STG = CFG_STG;
constexpr size_t YST = STG + (size_t)NRANK * SLOT;
static_assert(XBYTES + 9 * WKK <= SLOT, "slot");
constexpr unsigned YBYTES = M * OWNH * 256;
constexpr size_t WS_NEED = YST + YBYTES;
constexpr unsigned OUT_BYTES = (unsigned)M * N * 2;
constexpr unsigned ROWB = N * 2;
constexpr unsigned TKS = CFG_TKS;
constexpr unsigned C_CTR = 0xd0000, CTS = 256, C_S0 = 0, C_TMO = 128, C_YD = 256, C_S1 = 384, C_F = 512, C_G = 1024, C_R = 4096, C_ARR = 5120, C_TK = 0xc0000, C_END = C_TK + 32 * TKS, C_STAMP = 65536;
__host__ __device__ constexpr int imax(int a, int b) { return a > b ? a : b; }
__host__ __device__ constexpr int pend_of(int r) {
    return r < 0 ? 0 : r == 0 ? MB : r == NROUND - 1 ? NH : imax((ZB + (r - 1) * 2 * GPX + 3) / 4 * 2, MB + (ZT + (r - 1) * 2 * GPX + 3) / 4 * 2); }
__host__ __device__ constexpr int nh_of(int r) { return pend_of(r) - pend_of(r - 1); }
__host__ __device__ constexpr unsigned gsz_of(int r) { return ((r < 2 ? 16 : 0) + nh_of(r)) * PPB; }
__host__ __device__ constexpr unsigned rstart_of(int r) { return 8 * PPB * ((r < 2 ? 16 * r : 32) + pend_of(r - 1)); }
constexpr int R0 = MB - OWNH - XH;
constexpr int C0 = 16 + R0 + NRANK * OWNH + XH;
constexpr int FK = CFG_FK;
constexpr int PM = (MB + (nh_of(1) + 1) / 2) & ~((MB + 1) & 1);
constexpr int NMARK = 8 + 24 + 8 + 8;
constexpr int NYD = 16 * OWNH + 8 * ((OWNH + 1) / 2);
constexpr int N0 = FK * C0, NIMG = 8 * (160 + XH);
constexpr int PJ = CFG_PJ;
constexpr int NPUSH = PJ * NPULL / 16;
constexpr unsigned long long DEADLINE = 150000000ull;
constexpr int LDS_BYTES = 8 * BLK + 512;
constexpr unsigned NP0 = NPUSH * PPB;
constexpr unsigned NPI = rstart_of(NROUND);
constexpr unsigned NPO = M / 16 * OWNH;
constexpr unsigned DH = OWNH - XH, NPD = M / 16 * DH;
constexpr unsigned NPT = NPI + NPO;
constexpr unsigned EZ = CFG_EZ, REST = NPO - 8 * EZ, NPC = CFG_NPC;
constexpr unsigned NREL = NPI + 7 * EZ, NOWN = NPT - EZ, NEND = NOWN + (7 - XH) * EZ;
constexpr unsigned TB = NP0 + 2 * NPULL;
static_assert(N0 <= NPRE && NPRE + NIMG - N0 <= 8 * NPRE, "pre-pass list");
static_assert(NPUSH * 16 == PJ * NPULL && PJ % 2 == 0 && PJ > 0 && NPUSH <= 16 && CFG_PWIN <= PJ, "push split");
static_assert(R0 >= 0 && R0 < RH && GPX * 8 + NPULL <= NWG && 2 * MB <= GPX && (NH - MB) % 2 == 0 && T0 > 0 && T0 <= GPX && (NH + 1) / 2 * 4 <= ZB + 4 * GPX, "tiling");
static_assert(ZB >= 4 * ((OWNH + 1) / 2) && nh_of(1) > 0 && nh_of(2) > 0 && nh_of(3) > 0, "rounds");
static_assert(C_ARR + (NMARK + 1) * 1024 <= C_STAMP && C_END <= C_CTR && C_CTR + NWG * CTS <= STG, "ctrl");
static_assert((XH == 0 || XH == 1) && 8 * EZ + (1 - XH) * NPULL <= NPD && NPULL <= EZ && CFG_TX <= (7 - XH) * EZ && EZ <= NPO - NPD + (1 - XH) * EZ, "zones");
static_assert(gsz_of(0) >= NP0 + NPULL, "every puller wave starts on K tile 0");

constexpr unsigned S_HI = 1, S_K0 = 2, S_FK = 3, S_PUSH = 4, S_R0 = 5, S_PRE = 6, S_Y = 7, S_POOL = 8, S_TMO = 9;

template <int V> struct Int { static constexpr int value = V; };
struct Params { const u16* x; const u16* w; int rank; char* ws[NRANK]; char* out[NRANK]; };

#define RLX_AGENT __ATOMIC_RELAXED, __HIP_MEMORY_SCOPE_AGENT
#define RLX_SYS __ATOMIC_RELAXED, __HIP_MEMORY_SCOPE_SYSTEM
#define AUX_SYS 17
#define AUX_DEV 16

__device__ __forceinline__ unsigned rfl(unsigned v) { return (unsigned)__builtin_amdgcn_readfirstlane((int)v); }
__device__ __forceinline__ rsrc_t mk_rsrc(const void* p, unsigned bytes) { return __builtin_amdgcn_make_buffer_rsrc((void*)p, 0, (int)bytes, 0x00020000); }
__device__ __forceinline__ int panel_of(int h, int rank) { return h < OWNH + XH ? RH + rank * OWNH + h : h - OWNH - XH; }
__device__ __forceinline__ int col_of(int img) { return XH && img == 128 ? RH : img; }

__device__ __forceinline__ char* ctr_of(char* myws) { return myws + C_CTR + blockIdx.x * CTS; }
__device__ __forceinline__ unsigned ctr_load(char* myws) { return __builtin_amdgcn_raw_buffer_load_b32(mk_rsrc(ctr_of(myws), 4), 0, 0, CFG_CPOL); }
__device__ __forceinline__ unsigned call_of(unsigned ctrv) { return rfl(ctrv) + 1u; }
__device__ __forceinline__ bool expired(unsigned* tmo, unsigned call, unsigned long long t0) {
    if (rfl(__hip_atomic_load(tmo, RLX_AGENT)) == call || __builtin_amdgcn_s_memrealtime() - t0 > DEADLINE) {
        __hip_atomic_store(tmo, call, RLX_AGENT);
        return true;
    }
    return false;
}
template <int SLEEP> __device__ __forceinline__ bool wait_local(const unsigned* p, unsigned target, unsigned* tmo, unsigned call) {
    const unsigned long long t0 = __builtin_amdgcn_s_memrealtime();
    unsigned polls = 0;
    while ((int)(rfl(__hip_atomic_load(p, RLX_AGENT)) - target) < 0) {
        if ((++polls & 15u) == 0u && expired(tmo, call, t0)) return false;
        __builtin_amdgcn_s_sleep(SLEEP);
    }
    asm volatile("" ::: "memory");
    return true;
}

__device__ __forceinline__ bool count_and_flag(const Params& P, char* myws, unsigned cnt, unsigned total, unsigned word, unsigned call, int lane) {
    unsigned old = 0;
    if (lane == 0) old = __hip_atomic_fetch_add((unsigned*)(myws + cnt), 1u, RLX_AGENT);
    const bool last = rfl(old) + 1u == call * total;
    if (last && lane < NRANK) __hip_atomic_store((unsigned*)(P.ws[lane] + C_F) + word, call, RLX_SYS);
    return last;
}

#define WST(i, val) do {} while (0)
#define WDECL do {} while (0)
#define CNT_RETRY do {} while (0)
#define WNEXT(c, val) do {} while (0)
#define WDUMP do {} while (0)
#define STAMPC(i) do {} while (0)
#define STAMP(i) do {} while (0)
#define STAMPW(i) do {} while (0)

__device__ __forceinline__ void touch(const Params& P, int lane) {
    unsigned t[7], at = lane > 2 ? (unsigned)YST : lane == 2 ? 1u << 20 : (unsigned)STG - 4096u + (lane == 0 ? (unsigned)P.rank * SLOT : 0u), mask = lane == 1 ? ~0u : 0u;
    asm volatile("" : "+v"(at), "+v"(mask));
#pragma unroll
    for (int l = 0; l < 7; ++l) t[l] = __builtin_amdgcn_raw_buffer_load_b32(mk_rsrc(P.ws[l], (unsigned)YST), at + (mask & (unsigned)((P.rank + 1 + l) & 7) * SLOT), 0, AUX_SYS);
#pragma unroll
    for (int l = 0; l < 7; ++l) asm volatile("" :: "v"(t[l]));
}
__device__ __forceinline__ unsigned prepass(const Params& P, char* myws, LAS unsigned char* lds, int u) {
    const unsigned ctrv = ctr_load(myws);
    const int tid = threadIdx.x, lane = tid & 63, wave = (int)rfl(tid >> 6), half = wave & 1, pair = wave >> 1;
    const int kgl = lane >> 4, ng = lane & 15, sb = lane * 16, swz = sb ^ (((sb >> 9) & 1) << 5);
    const unsigned wlo = (unsigned)((half * 4 + kgl) * 8 * N + ng * 8) * 2u, xlo = (unsigned)((swz >> 6) * KS + half * 32) * 2u + (unsigned)(swz & 63);
    LAS unsigned char* lb = lds + pair * BLK + half * 1024;
    const unsigned ioff = (unsigned)((ng >> 2) * 4096 + (ng & 3) * 256 + ((kgl * 16) ^ (((ng >> 1) & 1) << 5)));
    LAS unsigned* cnt = (LAS unsigned*)(lds + 8 * BLK);
    char* const wsl = P.ws[lane & 7];
    WDECL; WST(0, 0); int wsi = 1;
    const unsigned long long tgo = __builtin_amdgcn_s_memrealtime() + CFG_DELAY;
    unsigned zbase = SLOT;
    for (int i = pair * NPRE + u; i < NPRE + NIMG - N0; i += 4 * NPRE) {
        int kk, img;
        if (i < NPRE + (8 - FK) * C0) { const int v = i < NPRE ? i : i - NPRE + N0; if (i >= N0 && i < NPRE) continue;
            kk = v / C0; const int bi = v % C0; img = bi < 16 ? bi : bi < 16 + R0 ? bi + 16 : bi + 16 + (RH - R0); }
        else { const int v = i - (NPRE + (8 - FK) * C0); kk = v / (160 + XH - C0); const int ci = v % (160 + XH - C0); img = ci < 16 ? 16 + ci : 16 + R0 + ci; }
        if (i >= NPRE) while (__builtin_amdgcn_s_memrealtime() < tgo + (unsigned)((i - NPRE) * CFG_SPAN / (NIMG - N0))) __builtin_amdgcn_s_sleep(4);
        if (CFG_KD && (i < NPUSH || i >= C0) && i < NPRE) while (__builtin_amdgcn_s_memrealtime() < tgo - CFG_DELAY + CFG_KD) __builtin_amdgcn_s_sleep(1);
        const bool isw = img >= 32;
        const unsigned so = isw ? wlo + (unsigned)(kk * 64 * N + col_of(img - 32) * 128) * 2u : xlo + (unsigned)(img * 128 * KS + kk * 64) * 2u;
        const unsigned sstep = isw ? ROWB : 16 * KS * 2;
        unsigned dbase = isw ? XBYTES + (unsigned)(kk + 1) * WKK + (unsigned)(img - 32) * BLK : (unsigned)(kk + 1) * XKK + (unsigned)img * BLK;
        const rsrc_t ri = mk_rsrc(isw ? P.w : P.x, isw ? KS * N * 2 : M * KS * 2);
        u32x4 in[8];
#pragma unroll
        for (int j = 0; j < 8; ++j) in[j] = __builtin_amdgcn_raw_buffer_load_b128(ri, so + j * sstep, 0, 0);
        if (isw) {
#pragma unroll
            for (int v8 = 0; v8 < 8; ++v8) {
                u32x4 o;
#pragma unroll
                for (int e = 0; e < 4; ++e) o[e] = __builtin_amdgcn_perm(in[2 * e + 1][v8 >> 1], in[2 * e][v8 >> 1], (v8 & 1) ? 0x07060302u : 0x05040100u);
                *(LAS u32x4*)(lb + ioff + (v8 >> 2) * 2048 + (v8 & 3) * 64) = o;
            }
            asm volatile("s_waitcnt lgkmcnt(0)" ::: "memory");
#pragma unroll
            for (int j = 0; j < 8; ++j) in[j] = *(LAS u32x4*)(lb + j * 2048 + lane * 16);
        }
        const unsigned parity = call_of(ctrv) & 1u, alt = kk ? 0u : isw ? WKK : XKK;
        if (i >= NPUSH && i < C0) zbase = dbase - parity * alt;
        dbase -= (parity ^ 1u) * alt;
        const rsrc_t rs = mk_rsrc(myws + STG + (size_t)P.rank * SLOT, SLOT);
#pragma unroll
        for (int j = 0; j < 8; ++j) __builtin_amdgcn_raw_buffer_store_b128(in[j], rs, dbase + (unsigned)(j * 2048 + half * 1024 + lane * 16), 0, AUX_SYS);
        if (i < NPRE) {
            asm volatile("s_waitcnt vmcnt(0)" ::: "memory");
            unsigned second = 0;
            if (lane == 0) second = __hip_atomic_fetch_add(cnt, 1u, __ATOMIC_RELAXED, __HIP_MEMORY_SCOPE_WORKGROUP);
            if (rfl(second) == 1u && i >= NPUSH && lane < 7 && (img < 32 + RH || (unsigned)(img - 32 - RH - ((P.rank + 1 + lane) & 7) * OWNH) < OWNH + XH))
                __hip_atomic_fetch_add((unsigned*)(wsl + C_G + (P.rank * 3 + (i < C0 ? 0 : 1)) * 128), 1u, RLX_SYS);
        } else if (i < NPRE + (8 - FK) * C0) {
            asm volatile("s_waitcnt vmcnt(0)" ::: "memory");
            count_and_flag(P, myws, C_S0, 2 * (8 - FK) * C0, P.rank, call_of(ctrv), lane);
        }
    }
    WDUMP;
    const unsigned call = call_of(ctrv);
    if (zbase < SLOT) {
        const rsrc_t rz = mk_rsrc(myws + STG + (size_t)P.rank * SLOT, SLOT);
#pragma unroll
        for (int j = 0; j < 8; ++j) __builtin_amdgcn_raw_buffer_store_b128((u32x4){0u, 0u, 0u, 0u}, rz, zbase + (unsigned)(j * 2048 + half * 1024 + lane * 16), 0, AUX_SYS);
    }
    asm volatile("s_waitcnt vmcnt(0)" :: "s"(call) : "memory");
    __syncthreads();
    if (wave == 0 && count_and_flag(P, myws, C_S1, NPRE, 8 + P.rank, call, lane) && lane < 8)
        __hip_atomic_store((unsigned*)(myws + C_ARR + NMARK * 1024 + lane * 128), call * NPULL, RLX_AGENT);
    if (tid == 64) __hip_atomic_store((unsigned*)ctr_of(myws), call, RLX_AGENT);
    if (u == 0 && tid >= 496) __hip_atomic_store((unsigned*)(myws + C_TK + (((call + 1u) & 1u) * 16 + (tid & 15)) * TKS), 0u, RLX_AGENT);
    return call;
}

__device__ __forceinline__ void run_of(int i, int rank, int link, unsigned& first, unsigned& base) {
    unsigned at, need = S_PRE;
    if (i < 24) { const unsigned kk = (unsigned)i / 3u, s = (unsigned)i % 3u;
        first = kk * gsz_of(0) + (s == 0 ? 0 : s == 1 ? 16 * PPB : (16 + OWNH + XH) * PPB);
        at = s == 0 ? (kk + 1) * XKK : XBYTES + (kk + 1) * WKK + (s == 1 ? (unsigned)panel_of(0, rank) * BLK : 0u);
        need = kk == 0 ? S_K0 : kk < FK ? S_FK : S_R0; }
    else if (i < 40) { const unsigned kk = (unsigned)(i - 24) >> 1;
        first = rstart_of(1) + kk * gsz_of(1) + (i & 1) * 16 * PPB;
        at = i & 1 ? XBYTES + (kk + 1) * WKK + R0 * BLK : (kk + 1) * XKK + 16 * BLK; }
    else if (i < 56) { const unsigned kk = (unsigned)i & 7u;
        first = i < 48 ? rstart_of(2) + kk * gsz_of(2) : rstart_of(3) + kk * gsz_of(3);
        at = XBYTES + (kk + 1) * WKK + ((i < 48 ? pend_of(1) : pend_of(2)) - OWNH - XH) * BLK; }
    else if (i < 59) {
        const int f = CFG_ZF * link % 7, k1 = 7 - f, k2 = (14 - rank - f) % 7, k = i == 56 ? 0 : (i == 57) == (k1 < k2) ? k1 : k2;
        first = NPI + (unsigned)k * EZ;
        at = (unsigned)((rank + 1 + (f + k) % 7) & 7) * (EZ * PIECE);
        need = S_Y; }
    else { const unsigned rot = (unsigned)link * (CFG_ROT * REST / 7);
        const bool shrt = XH && link == 0;
        first = i == 59 ? NREL : i == 60 ? (shrt ? NPI + NPD - EZ : NOWN - rot) : i == 61 ? NOWN : i == 62 ? NEND : ~0u;
        at = (i == 59 ? 8 * EZ + rot : i == 60 ? (shrt ? (unsigned)rank : 8u) * EZ : 0u) * PIECE;
        need = i < 61 ? S_Y : i == 61 ? S_POOL : 0; }
    base = at - first * PIECE + need;
}
__device__ __forceinline__ unsigned mark_end(int G) {
    if (G < 8) return (unsigned)(G + 1) * gsz_of(0);
    if (G < 32) { const unsigned kk = (unsigned)(G - 8) / 3u, sub = (unsigned)(G - 8) % 3u;
        return rstart_of(1) + kk * gsz_of(1) + (sub == 0 ? 16 * PPB : sub == 1 ? (16 + PM - MB) * PPB : gsz_of(1)); }
    return G < 40 ? rstart_of(2) + (unsigned)(G - 31) * gsz_of(2) : rstart_of(3) + (unsigned)(G - 39) * gsz_of(3);
}

#define WAIT_V(n) asm volatile("s_waitcnt vmcnt(" #n ")" ::: "memory")
#define WAIT_L(n) asm volatile("s_waitcnt lgkmcnt(" #n ")" ::: "memory")
#define RLX_WG __ATOMIC_RELAXED, __HIP_MEMORY_SCOPE_WORKGROUP
__device__ __forceinline__ void arrive(char* myws, LAS unsigned* lctr, int g, int lane) {
    unsigned old = 0;
    if (lane == 0) old = __hip_atomic_fetch_add(lctr + 64 + g, 1u, RLX_WG);
    if (rfl(old) == (g == 0 ? 7u : 6u)) {
        if (lane < 8 || (lane == 8 && g == 32)) __hip_atomic_fetch_add((unsigned*)(myws + (lane < 8 ? C_ARR + g * 1024 + lane * 128 : C_CTR + blockIdx.x * CTS)), 1u, RLX_AGENT);
        STAMPW(2 + g);
    }
}
__device__ __forceinline__ void puller(const Params& P, char* myws, LAS unsigned* lctr, int idx) {
    const int tid = threadIdx.x, lane = tid & 63, wave = (int)rfl(tid >> 6);
    unsigned* tmo = (unsigned*)(myws + C_TMO);
    int link = wave; bool ded = false;
    { const int sl = (idx >> 3) * 4 + (idx & 3), h = (idx >> 2) & 1, r = (sl - 7) * 2 + h;
      if (sl < 7) { link = sl; idx = h * 7 + wave; ded = true; } else if (r < 7) { link = r; idx = 14 + wave; ded = true; } else idx = 21 + r - 7; }
    if (wave == 7) {
        const unsigned ctrv = ctr_load(myws);
        if (ded && idx == 21 && lane == 0) __hip_atomic_fetch_add((unsigned*)(P.ws[link] + C_G + P.rank * 384 + 64), 1u, RLX_SYS);
        const unsigned call = call_of(ctrv), p = (unsigned)(P.rank + 1 + lane) & 7u;
        unsigned st = lane < 7 && (!ded || lane == link) ? 0u : S_POOL;
        bool counted = false, told = false;
        const unsigned long long t0 = __builtin_amdgcn_s_memrealtime();
        for (unsigned polls = 1;; ++polls) {
            if (st < S_POOL) {
                const unsigned fw = st == 0u ? C_G + p * 384 + 64 : st < S_PUSH ? C_G + (p * 3 + st - 1u) * 128 : st < S_Y ? C_F + ((st - S_PUSH) * 8 + p) * 4 : C_R + lane * 128;
                const unsigned ft = call * (unsigned)(st == S_HI ? 16 + MB - NPUSH : st == S_K0 ? (FK - 1) * (16 + MB) : st == S_FK || st == S_Y ? NPULL : 1);
                if ((int)(__hip_atomic_load((const unsigned*)(myws + fw), RLX_SYS) - ft) >= 0) { ++st; __hip_atomic_store(lctr + 32 + lane, st, RLX_WG); }
            }
            if ((polls & 15u) == 0u && expired(tmo, call, t0)) { st = S_TMO; if (lane < 7) __hip_atomic_store(lctr + 32 + lane, st, RLX_WG); }
            if (!counted && __all(st >= S_PUSH)) { counted = true; arrive(myws, lctr, 0, lane); }
            if (!told && rfl(__hip_atomic_load(lctr + 64 + NMARK, RLX_WG)) == 7u) {
                told = true;
                if (lane < 7) __hip_atomic_fetch_add((unsigned*)(P.ws[lane] + C_R + (6 - lane) * 128), 1u, RLX_SYS);
            }
            if (__all(st >= S_POOL) && (told || __builtin_amdgcn_s_memrealtime() - t0 > DEADLINE + DEADLINE / 4)) break;
            if (CFG_LSLEEP && rfl(__hip_atomic_load(lctr + 64 + NMARK - 1, RLX_WG)) != 0u) { __builtin_amdgcn_s_sleep(CFG_LSLEEP); continue; }
            if (__all(st >= S_R0)) __builtin_amdgcn_s_sleep(127);
            if (__all(st >= S_FK)) __builtin_amdgcn_s_sleep(CFG_HSLEEP); else if (__all(st >= (CFG_SPEC ? S_HI : S_K0))) __builtin_amdgcn_s_sleep(8); else __builtin_amdgcn_s_sleep(1);
        }
        return;
    }
    WDECL; WST(0, 0);
    const unsigned p = (unsigned)(P.rank + 1 + link) & 7u;
    char* const pws = P.ws[link];
    unsigned ctrv, rfirst, rbase;
    {
        const rsrc_t rx = mk_rsrc(P.x, M * KS * 2);
        const int sb = lane * 16, swz = sb ^ (((sb >> 9) & 1) << 5), c0 = idx * PJ;
        u32x4 in[PJ];
        const unsigned so = (unsigned)((c0 * 8 + (swz >> 6)) * KS) * 2u + (unsigned)(swz & 63);
        asm volatile("buffer_load_dword %0, off, %1, 0 sc1" : "=v"(ctrv) : "s"(mk_rsrc(ctr_of(myws), 4)) : "memory");
#pragma unroll
        for (int j = 0; j < PJ; ++j) asm volatile("buffer_load_dwordx4 %0, %1, %2, %3 offen offset:%4" : "=v"(in[j]) : "v"(so), "s"(rx), "s"((j >> 1) * (16 * KS * 2)), "n"((j & 1) * 64) : "memory");
        asm volatile("s_waitcnt vmcnt(%1)" : "+v"(ctrv) : "n"(PJ) : "memory");
        const rsrc_t rp = mk_rsrc(pws + STG + (size_t)P.rank * SLOT + (call_of(ctrv) & 1u) * XKK, XKK);
        WST(60, link);
#pragma unroll
        for (int j = 0; j < PJ; ++j) {
            asm volatile("s_waitcnt vmcnt(%1)" : "+v"(in[j]) : "n"(j < CFG_PWIN ? PJ - 1 : CFG_PWIN - 1) : "memory");
            if (j == 0) WST(1, link);
            __builtin_amdgcn_raw_buffer_store_b128(in[j], rp, (unsigned)((c0 + j) * 1024 + sb), 0, AUX_SYS);
            if (j == CFG_PWIN - 1) {
                int ln = lane;
#pragma unroll
                for (int k = j + 1; k < PJ; ++k) asm volatile("s_waitcnt vmcnt(%1)" : "+v"(in[k]) : "n"(CFG_PWIN) : "memory");
                asm volatile("" : "+v"(ln) :: "memory");
                run_of(ln, P.rank, link, rfirst, rbase);
                asm volatile("" : "+v"(rfirst), "+v"(rbase) :: "memory");
            }
        }
    }
    const unsigned call = call_of(ctrv), parity = call & 1u;
    { const unsigned long long mx = parity ? 0ull : 1ull | 1ull << 24, mw = parity ? 0ull : 6ull | 1ull << 25 | 1ull << 40 | 1ull << 48;
      rbase -= (unsigned)(mx >> lane & 1u) * XKK + (unsigned)(mw >> lane & 1u) * WKK; }
    const size_t soff = STG + (size_t)p * SLOT;
    char* const srcI = pws + soff; char* const dstI = myws + soff;
    char* const srcO = P.out[link]; char* const dstO = P.out[7];
    const rsrc_t rctl = mk_rsrc(myws, C_END);
    const unsigned tkoff = lane == 0 ? C_TK + parity * 16 * TKS : C_END;
    int gsig = 0;
    unsigned mend = mark_end(0);
    auto signal = [&](unsigned n) {
        while (mend <= n) {
            if (gsig < NMARK) arrive(myws, lctr, gsig, lane);
            else if (lane == 0) __hip_atomic_fetch_add(lctr + 64 + NMARK, 1u, RLX_WG);
            gsig = (int)rfl((unsigned)gsig + 1u);
            mend = gsig < NMARK ? mark_end(gsig) : gsig == NMARK ? NREL : ~0u;
        }
    };
    auto look = [&](unsigned n) { return (unsigned)__builtin_amdgcn_readlane((int)rbase, __builtin_popcountll(__ballot(rfirst <= n)) - 1) + n * PIECE; };
    auto place = [&](unsigned q, unsigned l) {
        return (q < NPD ? q / DH * (16 * ROWB - DH * 256) + (q + XH) * 256u : (q - NPD) * (16 * ROWB)) + (RH + ((unsigned)(P.rank + 1 + l) & 7u) * OWNH) * 256u; };
    auto pooled = [&](unsigned c) { return CFG_PO ? place(c / (7 - XH) + (unsigned)P.rank * EZ, c % (7 - XH) + XH) : place(c % EZ + (unsigned)P.rank * EZ, c / EZ + XH); };
    unsigned stage = 0;
    auto await = [&](unsigned need) {
        const unsigned long long t0 = __builtin_amdgcn_s_memrealtime();
        unsigned polls = 0;
        while ((stage = rfl(__hip_atomic_load(lctr + 32 + link, RLX_WG))) < need) {
            if ((++polls & 1023u) == 0u && expired(tmo, call, t0)) { stage = S_TMO; break; }
            __builtin_amdgcn_s_sleep(1);
        }
        asm volatile("" ::: "memory");
    };
    unsigned n = NP0 + (unsigned)idx;
    const unsigned vrow = (unsigned)(lane >> 4) * ROWB + (unsigned)(lane & 15) * 16u;
    rsrc_t rs = mk_rsrc(srcI, SLOT), rd = mk_rsrc(dstI, SLOT);
    unsigned vs = lane * 16, vd = vs, so = look(n) & ~15u, od = so, ss = 1024, ds = 1024;
    WST(2, 0);
    await(CFG_SPEC ? S_HI : S_K0);
    WST(3, 0);
    int wsi = 5;
#define RC "memory", "v200", "v201", "v202", "v203", "v204", "v205", "v206", "v207", "v208", "v209", "v210", "v211", "v212", "v213", "v214", "v215", \
        "v216", "v217", "v218", "v219", "v220", "v221", "v222", "v223", "v224", "v225", "v226", "v227", "v228", "v229", "v230", "v231"
#define LD16X(pre, u, v, r, o) asm volatile(pre "buffer_load_dwordx4 v[%0:%1], %2, %3, %4 offen sc0 sc1" :: "n"(200 + 4 * (u)), "n"(203 + 4 * (u)), "v"(v), "s"(r), "s"(o) : RC)
#define LD16(u, v, r, o) LD16X("", u, v, r, o)
#define LDC(u) LD16(u, vs, rs, so + (u) * ss)
#define LDR(u) LD16X("s_nop 4\n\t", u, vs, rs, so + (u) * 1024)
#define LANDED(cnt) asm volatile("s_waitcnt vmcnt(%0)" :: "n"(cnt) : RC)
#define NAN16(u) asm volatile("v_mov_b32 v[%0], %1\n\tv_mov_b32 v[%0+1], %1\n\tv_mov_b32 v[%0+2], %1\n\tv_mov_b32 v[%0+3], %1" :: "n"(200 + 4 * (u)), "v"(0x7fc07fc0u) : RC)
#define ST16(u, v, r, o) asm volatile("buffer_store_dwordx4 v[%0:%1], %2, %3, %4 offen sc0 sc1\n\ts_nop 1" :: "n"(200 + 4 * (u)), "n"(203 + 4 * (u)), "v"(v), "s"(r), "s"(o) : RC)
#define ZERO(u) ({ unsigned _m; asm volatile("v_min3_u32 %0, v[%1], v[%1+1], v[%1+2]\n\tv_min_u32 %0, %0, v[%1+3]" : "=v"(_m) : "n"(200 + 4 * (u)) : RC); __any(_m == 0u); })
#define AGAIN(ld) { stage = rfl(__hip_atomic_load(lctr + 32 + link, RLX_WG)); \
        if ((++spins & 255u) == 0u && expired(tmo, call, t0)) stage = S_TMO; \
        ld; LANDED(0); CNT_RETRY; \
        if (stage >= S_K0) { if (stage == S_TMO) odd |= 2u; break; } }
#define STC(u) do { if (__builtin_expect(odd != 0u, 0)) { if (CFG_SPEC && (odd & 1u)) while (ZERO(u)) AGAIN(LDR(u)) if (odd & 2u) NAN16(u); } ST16(u, vd, rd, od + (u) * ds); } while (0)
#define DRAW asm volatile("v_mov_b32 v199, 1\n\tbuffer_atomic_add v199, %0, %1, 0 offen sc0" :: "v"(tko), "s"(rctl) : RC, "v199")
#define TICKET ({ unsigned _t; asm volatile("v_readfirstlane_b32 %0, v199\n\ts_nop 4" : "=s"(_t) :: RC, "v199"); _t; })
    const unsigned turn = (XH && link == 0 ? NPI + NPD : NOWN) - NPULL;
    unsigned tb = TB, tko = tkoff + (unsigned)link * TKS, nn, odn;
    rsrc_t rsn = rs, rdn = rd;
    auto run = [&](auto wc) {
        constexpr int W = decltype(wc)::value;
        unsigned odd = stage == S_TMO ? 3u : 1u, spins = 0, tm = 1;
        unsigned vsn = vs, vdn = vd, son, ssn = ss, dsn = ds;
        asm volatile("v_mov_b32 v199, %0" :: "s"(n + NPULL - TB) : RC, "v199");
        const unsigned long long t0 = __builtin_amdgcn_s_memrealtime();
        LDR(0); LDR(1);
        if (W == 3) { asm volatile("buffer_store_dword %0, %0, %1, 0 offen" :: "v"(C_END), "s"(rctl) : "memory"); LDR(2); }
        LANDED(2 * W - 3);
        WST(4, n);
        asm volatile("buffer_atomic_add %0, %1, %2, 0 offen sc1" :: "v"(1u), "v"(lane == 0 ? 0u : 4u), "s"(mk_rsrc(pws + C_G + (P.rank * 3 + 2) * 128, 4)) : "memory");
        if (__builtin_expect(ZERO(0), 0)) do AGAIN(LDR(0); LDR(1); if (W == 3) LDR(2)) while (ZERO(0) || ZERO(1) || ZERO(W - 1));
        for (;;) {
            LANDED(2 * W - 1);
            LDC(W);
            STC(0);
            nn = TICKET * tm + tb;
            son = look(nn);
            const unsigned need = son & 15u;
            odn = son -= need;
            if (nn >= NPI) {
                const unsigned q = son / PIECE;
                odn = need == S_Y ? place(q, (unsigned)link) : need ? pooled(q) : OUT_BYTES;
                rdn = mk_rsrc(dstO, OUT_BYTES); vdn = vrow; dsn = 4 * ROWB;
                if (CFG_YST && need == S_Y) rsn = mk_rsrc(pws + YST, YBYTES);
                else { rsn = mk_rsrc(srcO, OUT_BYTES); vsn = vrow; ssn = 4 * ROWB; son = odn; }
                if (nn >= turn) { const unsigned pc = (unsigned)(idx + link) % NPC; tb = NOWN + pc; tm = NPC; tko = tkoff + (8 + pc) * TKS; }
            }
            DRAW;
            bool hold = false;
            unsigned oddn = odd & 2u;
            if (__builtin_expect(stage < need, 0)) {
                stage = rfl(__hip_atomic_load(lctr + 32 + link, RLX_WG));
                if (CFG_SPEC && need == S_K0) { if (stage < S_K0) oddn |= 1u; } else hold = stage < need;
            }
            if (nn + CFG_TX >= NEND && !hold) break;
#pragma unroll
            for (int u = 1; u < 4; ++u) {
                if (u == 4 - W && __builtin_expect(hold, 0)) {
                    LANDED(0);
#pragma unroll
                    for (int v = u; v < 4; ++v) STC(v);
                    LANDED(0);
                    signal(nn);
                    await(need);
                }
                LANDED(2 * W - 1 + (u <= W ? 1 : 0));
                if (u + W < 4) LDC(u + W); else LD16(u + W - 4, vsn, rsn, son + (u + W - 4) * ssn);
                STC(u);
            }
            WNEXT(wsi < 40 || n + 8 * NPULL >= NOWN, n);
            signal(n);
            odd = stage == S_TMO ? 2u : oddn;
            n = nn; rs = rsn; rd = rdn; vs = vsn; vd = vdn; so = son; od = odn; ss = ssn; ds = dsn;
        }
        WST(60, nn);
        if (W == 2) LDC(3);
#pragma unroll
        for (int u = 0; u < 4; ++u) LD16(4 + u, vrow, rsn, odn + u * 4 * ROWB);
#pragma unroll
        for (int u = 1; u < 4; ++u) { LANDED(u <= W ? 9 - W + u - 1 : 4 + W); if (__builtin_expect(odd & 2u, 0)) NAN16(u); ST16(u, vd, rd, od + u * ds); }
        WNEXT(true, n);
        if (W == 3) LANDED(7);
    };
    if (ded ? wave < CFG_W3 : CFG_W3M) run(Int<3>{}); else run(Int<2>{});
    auto more = [&](auto hc) {
        constexpr int H = decltype(hc)::value;
        const unsigned o = odn, nx = TICKET * NPC + tb;
        if (nx < NEND) {
            CNT_RETRY;
            DRAW;
            odn = pooled(nx - NOWN);
#pragma unroll
            for (int u = 0; u < 4; ++u) LD16(H + u, vrow, rsn, odn + u * 4 * ROWB);
            LANDED(5);
        } else LANDED(0);
        WNEXT(nn < NEND, nn);
        nn = nx;
#pragma unroll
        for (int u = 0; u < 4; ++u) { if (__builtin_expect(stage == S_TMO, 0)) NAN16(4 - H + u); ST16(4 - H + u, vrow, rdn, o + u * 4 * ROWB); }
        LANDED(8);
        return nx < NEND;
    };
    while (more(Int<0>{}) && more(Int<4>{})) {}
    if (mend != ~0u) { LANDED(0); signal(NEND); }
    WST(61, nretry); WDUMP;
#undef STC
#undef LDC
#undef LDR
#undef LD16X
#undef NAN16
#undef DRAW
#undef TICKET
#undef RC
#undef AGAIN
#undef ZERO
#undef LD16
#undef LANDED
#undef ST16
}

__device__ __forceinline__ unsigned cvt_pk_bf16(float lo, float hi) {
    typedef __bf16 bf16x2 __attribute__((ext_vector_type(2)));
    return __builtin_bit_cast(unsigned, __builtin_convertvector((f32x2){lo, hi}, bf16x2));
}
constexpr float GELU_K1 = -2.0f * 0.7978845608028654f * 1.4426950408889634f;
__device__ __forceinline__ u32x4 gelu8(f32x4 v0, f32x4 v1, float k1) {
    f32x2 y[4] = {{v0[0], v0[1]}, {v0[2], v0[3]}, {v1[0], v1[1]}, {v1[2], v1[3]}}, e[4];
#pragma unroll
    for (int i = 0; i < 4; ++i) e[i] = y[i] * __builtin_elementwise_fma(y[i] * y[i], (f32x2)(GELU_K1 * 0.044715f), (f32x2)(k1));
#pragma unroll
    for (int i = 0; i < 4; ++i) e[i] = (f32x2){__builtin_amdgcn_exp2f(e[i][0]), __builtin_amdgcn_exp2f(e[i][1])};
#pragma unroll
    for (int i = 0; i < 4; ++i) e[i] += 1.0f;
#pragma unroll
    for (int i = 0; i < 4; ++i) e[i] = (f32x2){__builtin_amdgcn_rcpf(e[i][0]), __builtin_amdgcn_rcpf(e[i][1])};
#pragma unroll
    for (int i = 0; i < 4; ++i) e[i] *= y[i];
    return (u32x4){cvt_pk_bf16(e[0][0], e[0][1]), cvt_pk_bf16(e[1][0], e[1][1]), cvt_pk_bf16(e[2][0], e[2][1]), cvt_pk_bf16(e[3][0], e[3][1])};
}

__device__ __forceinline__ void gemm(const Params& P, char* myws, LAS unsigned char* lds, unsigned call, int xcd, int j) {
    const int tid = threadIdx.x, wid = (int)rfl(tid >> 6), lane = tid & 63, wr = wid >> 2, wc = wid & 3, fr = lane & 15, fq = lane >> 4;
    auto tile_of = [&](int round) {
        int q = xcd & 3, z; const int par = xcd >> 2;
        if (round > 1) z = (q >= 2 ? ZB : ZT) + ((round - 2) * 2 + par) * GPX + j;
        else if (q >= 2) z = par * GPX + j;
        else if (j < T0) z = par * T0 + j;
        else { q += 2; z = 2 * GPX + par * (GPX - T0) + (j - T0); }
        return q * 4 + (z & 3) + (((q >= 2 ? 0 : MB) + (z >> 2) * 2) << 4);
    };
    unsigned tabA, tabB;
    { const int u = lane - 8, kk = lane < 8 ? lane : u / 7, sl = lane < 8 ? P.rank : (P.rank + 1 + u - 7 * kk) & 7;
      const unsigned kx = kk ? (unsigned)kk + 1u : call & 1u;
      tabA = (unsigned)sl * SLOT + kx * XKK; tabB = (unsigned)sl * SLOT + XBYTES + kx * WKK; }
#define KA(t) (unsigned)__builtin_amdgcn_readlane((int)tabA, (t))
#define KB(t) (unsigned)__builtin_amdgcn_readlane((int)tabB, (t))
    int pm = tile_of(1) & 15, h0 = tile_of(1) >> 4;
    unsigned* tmo = (unsigned*)(myws + C_TMO);
    const rsrc_t rs = mk_rsrc(myws + STG, NRANK * SLOT);
    const unsigned arr = C_ARR + (unsigned)xcd * 128u;
    const rsrc_t rctl = mk_rsrc(myws, C_END);
    const unsigned voff = (unsigned)(wid * 2048 + lane * 16);
    LAS unsigned char* const ldw = lds + wid * 2048;
    const unsigned atarget = call * NPULL;
    int aoff, boff;
    { const int ob = fr * 64 + fq * 16, sw = ob ^ (((ob >> 9) & 1) << 5); aoff = wr * 8192 + sw; boff = wc * 4096 + sw + 4 * BLK; }
    asm volatile("" : "+v"(boff));
    bool ok = true;
#define SA(b, h) (((b) * 2 + (h)) * BLK)
#define SB(b, h) ((4 + (b) * 2 + (h)) * BLK)
#define STAGE(bufoff, so) do { __builtin_amdgcn_raw_ptr_buffer_load_lds(rs, (LAS void*)(ldw + (bufoff)), 16, (int)voff, (int)(so), 0, CFG_GAUX); \
        __builtin_amdgcn_raw_ptr_buffer_load_lds(rs, (LAS void*)(ldw + (bufoff)), 16, (int)voff, (int)(so), 1024, CFG_GAUX); } while (0)
#define LDA(dst, off) do { _Pragma("unroll") for (int m = 0; m < 4; ++m) _Pragma("unroll") for (int k = 0; k < 2; ++k) dst[m][k] = *(const LAS bf16x8*)(lds + (off) + aoff + m * 2048 + k * 1024); } while (0)
#define LDB(dst, off) do { _Pragma("unroll") for (int n = 0; n < 2; ++n) _Pragma("unroll") for (int k = 0; k < 2; ++k) dst[n][k] = *(const LAS bf16x8*)(lds + ((off) - 4 * (int)BLK) + boff + n * 2048 + k * 1024); } while (0)
#define MM16(C, At, Bt, FILL) do { _Pragma("unroll") for (int _q = 0; _q < 16; ++_q) { \
        const int m = CFG_MORD == 6 ? ((_q >> 1) & 3) ^ (_q >> 3) * 3 : (_q >> 1) & 3, n = _q >> 3, k = _q & 1; \
        C[m][n] = __builtin_amdgcn_mfma_f32_16x16x32_bf16(Bt[n][k], At[m][k], C[m][n], 0, 0, 0); SCHED; FILL; SCHED; } } while (0)
#define TURN do { PRIO(1); __builtin_amdgcn_s_waitcnt(0xc07f); BAR; SCHED; } while (0)
#define DONE do { BAR; PRIO(0); SCHED; } while (0)
#define MMA(C, At, Bt, FILL) do { TURN; MM16(C, At, Bt, FILL); DONE; } while (0)
#define PRIO(n) do {} while (0)
#define BAR __builtin_amdgcn_s_barrier()
#define SCHED __builtin_amdgcn_sched_barrier(0)
#define PIN(x) asm volatile("" : "+s"(x))
    f32x4 acc[2][2][4][2];
#pragma unroll
    for (int a = 0; a < 2; ++a)
#pragma unroll
        for (int b = 0; b < 2; ++b)
#pragma unroll
            for (int m = 0; m < 4; ++m)
#pragma unroll
                for (int n = 0; n < 2; ++n) acc[a][b][m][n] = (f32x4){0.f, 0.f, 0.f, 0.f};
    bf16x8 At[4][2], B0[2][2], B1[2][2];
    if (wid == 0) ok = wait_local<CFG_GSLEEP>((const unsigned*)(myws + arr + NMARK * 1024), atarget, tmo, call);
    asm volatile("" ::: "memory"); BAR; asm volatile("" ::: "memory");
    STAMP(2); STAMPC(22);
    const rsrc_t ro = mk_rsrc(P.out[7], OUT_BYTES);
    bool slow = CFG_ESM & 1;
    auto put = [&](f32x4 (&a)[4][2], int rowp, int h, bool last) {
        const unsigned o0 = (unsigned)(rowp * 128 + wr * 64 + fr) * ROWB + (unsigned)(wc * 32 + fq * 8) * 2u + (h < NH ? (unsigned)col_of(panel_of(h, P.rank)) * 256u : OUT_BYTES);
#pragma unroll
        for (int m = 0; m < 4; ++m) {
            const u32x4 o = gelu8(a[m][0], a[m][1], ok ? GELU_K1 : __builtin_nanf(""));
            if (slow) __builtin_amdgcn_s_sleep(CFG_ES);
            if (last || h < OWNH) __builtin_amdgcn_raw_buffer_store_b128(o, ro, o0 + (unsigned)(m * 16) * ROWB, 0, AUX_SYS);
            else __builtin_amdgcn_raw_buffer_store_b128(o, ro, o0 + (unsigned)(m * 16) * ROWB, 0, CFG_SO);
            if (h < OWNH) __builtin_amdgcn_raw_buffer_store_b128(o, mk_rsrc(myws + YST, YBYTES), (unsigned)(fr * 256 + wc * 64 + fq * 16) + (h < XH ? NPD + (unsigned)(rowp * 8 + wr * 4 + m) : (unsigned)(rowp * 8 + wr * 4 + m) * DH + h - XH) * PIECE, 0, AUX_SYS);
            a[m][0] = a[m][1] = (f32x4){0.f, 0.f, 0.f, 0.f};
        }
    };
    auto publish = [&]() {
        WAIT_V(0);
        unsigned old = 0;
        if (lane == 0) old = __hip_atomic_fetch_add((LAS unsigned*)(lds + 8 * BLK) + 1, 1u, RLX_WG);
        if ((rfl(old) & 7u) == 7u) count_and_flag(P, myws, C_YD, NYD, 16 + P.rank, call, lane);
    };
    unsigned pA = (unsigned)pm * 2 * BLK, pB0 = (unsigned)panel_of(h0, P.rank) * BLK, pB1 = (unsigned)panel_of(h0 + 1 < NH ? h0 + 1 : h0, P.rank) * BLK;
    unsigned pollv = 0, sa = 0, sa1 = 0, sb0 = 0, sb1 = 0, gw = 0, gstep = 0;
    int tn = 0;
    auto gate = [&](int s) {
        if (__builtin_expect(s + 3 >= tn, 0)) {
            if (s + 3 == tn) { if (wid == 0) asm volatile("s_nop 4\n\tbuffer_load_dword %0, off, %1, %2 sc1" : "+v"(pollv) : "s"(rctl), "s"(gw) : "memory"); }
            else {
                if (wid == 0 && (int)(rfl(pollv) - atarget) < 0) ok = wait_local<CFG_GSLEEP>((const unsigned*)(myws + gw), atarget, tmo, call) && ok;
                tn = tn < 57 ? tn + 7 : 99; gw += gstep;
            }
        }
    };
#define WAIT_P asm volatile("s_waitcnt vmcnt(%1)" : "+v"(pollv) : "n"(8) : "memory")
    for (int round = 0;; ++round) {
        const int tnext = tile_of(round + 1), pmn = tnext & 15, h0n = tnext >> 4;
        const bool has_next = round < NROUND - 1 && h0n < NH;
        const unsigned nA = has_next ? (unsigned)pmn * 2 * BLK : pA;
        const unsigned nB0 = has_next ? (unsigned)panel_of(h0n, P.rank) * BLK : pB0;
        const unsigned nB1 = has_next ? (unsigned)panel_of(h0n + 1 < NH ? h0n + 1 : h0n, P.rank) * BLK : pB1;
        const int tb = round == 1 ? 8 : 0, tl = round == 0 ? 6 : 62;
        tn = round == 0 ? 99 : 8;
        gstep = round == 1 ? 3072u : 1024u;
        gw = arr + (round == 1 ? (h0 + 1 < MB ? 8u : h0 + 1 < PM ? 9u : 10u) : (unsigned)round * 8u + 16u) * 1024u;
        if (round < 2) {
            if (round == 1) {
                if (wid == 0) ok = wait_local<CFG_MSLEEP>((const unsigned*)(myws + gw), atarget, tmo, call) && ok;
                asm volatile("" ::: "memory"); BAR; asm volatile("" ::: "memory");
                tn = 15; gw += gstep;
            }
            STAGE(SB(0, 0), pB0 + KB(tb)); STAGE(SB(0, 1), pB1 + KB(tb)); STAGE(SA(0, 0), pA + KA(tb)); STAGE(SA(0, 1), pA + KA(tb) + BLK);
            STAGE(SB(1, 0), pB0 + KB(tb + 1)); STAGE(SB(1, 1), pB1 + KB(tb + 1)); STAGE(SA(1, 0), pA + KA(tb + 1));
            sa1 = pA + KA(tb + 1) + BLK;
            WAIT_V(8); BAR;
            if (wr == 1) BAR;
            STAMP(10);
        }
        for (int t = tb;; t += 2) {
            int t2 = t + 2;
            LDB(B0, SB(0, 0)); LDA(At, SA(0, 0)); LDB(B1, SB(0, 1)); STAGE(SA(1, 1), sa1);
            WAIT_P; TURN;
            MM16(acc[0][0], At, B0,
                if (_q == 0) { PIN(t2); if (t == tl) { t2 = 0; pA = nA; } PIN(t2); PIN(pA); }
                if (_q == 1) { if (t2 == 0) { pB0 = nB0; pB1 = nB1; } PIN(pB0); PIN(pB1); }
                if (_q == 2) { sb1 = KB(t2); sa = KA(t2); PIN(sb1); PIN(sa); }
                if (_q == 4) { sb0 = sb1 + pB0; sb1 += pB1; PIN(sb0); PIN(sb1); }
                if (_q == 5) { sa += pA; sa1 = sa + BLK; PIN(sa); PIN(sa1); });
            MM16(acc[0][1], At, B1, if (_q == 8) gate(t));
            DONE;
            LDA(At, SA(0, 1)); STAGE(SB(0, 0), sb0); STAGE(SB(0, 1), sb1); STAGE(SA(0, 0), sa);
            WAIT_V(8); TURN; MM16(acc[1][1], At, B1, ); MM16(acc[1][0], At, B0, ); DONE;
            LDB(B0, SB(1, 0)); LDA(At, SA(1, 0)); LDB(B1, SB(1, 1)); STAGE(SA(0, 1), sa1);
            WAIT_P; TURN;
            MM16(acc[0][0], At, B0,
                if (_q == 0) { PIN(t2); ++t2; PIN(t2); }
                if (_q == 2) { sb1 = KB(t2); sa = KA(t2); PIN(sb1); PIN(sa); }
                if (_q == 4) { sb0 = sb1 + pB0; sb1 += pB1; PIN(sb0); PIN(sb1); }
                if (_q == 5) { sa += pA; sa1 = sa + BLK; PIN(sa); PIN(sa1); });
            MM16(acc[0][1], At, B1, if (_q == 8) gate(t + 1));
            DONE;
            LDA(At, SA(1, 1)); STAGE(SB(1, 0), sb0); STAGE(SB(1, 1), sb1); STAGE(SA(1, 0), sa);
            WAIT_V(8); TURN; MM16(acc[1][1], At, B1, ); MM16(acc[1][0], At, B0, ); DONE;
            if (t2 == 1) break;
        }
        if (round == 0) {
            WAIT_V(0);
            if (wr == 0) BAR;
            STAMP(11);
            if (j < 2 * MB) {
                const int mr = CFG_MM ? (xcd & 3) * 4 + (j & 3) : xcd * 2 + (j & 1), mh = CFG_MM == 1 ? (xcd >> 2) * (MB / 2) + (j >> 2) : CFG_MM ? (j >> 2) * 2 + (xcd >> 2) : j >> 1;
                const unsigned mA = (unsigned)mr * BLK, mB = (unsigned)panel_of(mh, P.rank) * BLK;
                f32x4 mc[4][2];
#pragma unroll
                for (int m = 0; m < 4; ++m) mc[m][0] = mc[m][1] = (f32x4){0.f, 0.f, 0.f, 0.f};
#define MSTAGE(t) do { const int _k = (t) < 64 ? (t) : 63; STAGE(((t) & 3) * 2 * BLK, mA + KA(_k)); STAGE((((t) & 3) * 2 + 1) * BLK, mB + KB(_k)); } while (0)
                MSTAGE(0); MSTAGE(1); MSTAGE(2);
                tn = 8; gw = arr;
                WAIT_V(8); BAR;
                if (wr == 1) BAR;
                for (int t = 0; t < 64; ++t) {
                    const int lb = (t & 3) * 2 * BLK;
                    LDB(B0, lb + BLK); LDA(At, lb);
                    MSTAGE(t + 3);
                    WAIT_V(8);
                    MMA(mc, At, B0,
                        if (_q == 8 && t + 4 == tn) {
                            if (wid == 0) ok = (tn == 57 ? wait_local<CFG_MSLEEP>((const unsigned*)(myws + gw), atarget, tmo, call) : wait_local<CFG_GSLEEP>((const unsigned*)(myws + gw), atarget, tmo, call)) && ok;
                            tn = tn < 57 ? tn + 7 : 0; gw += 1024u;
                        });
                }
#undef MSTAGE
                WAIT_V(0);
                if (wr == 0) BAR;
                STAMP(9);
                put(mc, mr, mh, false);
                if (mh < OWNH) publish();
            }
            continue;
        }
        STAMP(1 + 2 * round); STAMPC(21 + 2 * round);
        slow = CFG_ESM >> round & 1;
        unsigned pf;
        pf = __builtin_amdgcn_raw_buffer_load_b32(mk_rsrc(myws + YST, YBYTES), lane * 128, round == 2 ? ((unsigned)(wid * GPX + j) * 8u + (unsigned)xcd) * 8192u : YBYTES, AUX_SYS);
        put(acc[0][0], pm * 2, h0, round >= CFG_WTR);
        asm volatile("" :: "v"(pf));
        put(acc[1][0], pm * 2 + 1, h0, round >= CFG_WTR); put(acc[0][1], pm * 2, h0 + 1, round >= CFG_WTR); put(acc[1][1], pm * 2 + 1, h0 + 1, round >= CFG_WTR);
        if (h0 < OWNH) publish();
        STAMP(2 + 2 * round); STAMPC(22 + 2 * round);
        if (!has_next) break;
        h0 = h0n; pm = pmn;
    }
    WAIT_V(0);
    if (wr == 0) BAR;
#undef KA
#undef KB
#undef SA
#undef SB
#undef STAGE
#undef LDA
#undef LDB
#undef MMA
#undef PIN
#undef WAIT_P
}

__global__ __launch_bounds__(512, 2) void fused_gemm_allreduce(Params P) {
    extern __shared__ __attribute__((aligned(16))) unsigned char shm[];
    char* myws = P.ws[7];
    const int b = blockIdx.x, xcd = b & 7, j = b >> 3;
    STAMP(0);
    if (threadIdx.x < 128) ((LAS unsigned*)shm)[8 * BLK / 4 + threadIdx.x] = 0;
    asm volatile("" :: "s"(P.x), "s"(P.w), "s"(P.rank), "s"(myws), "s"(P.out[7]));
    WAIT_L(0); __builtin_amdgcn_s_barrier(); asm volatile("" ::: "memory");
    if (__builtin_expect(j == CFG_GTN, 0)) {
        asm volatile("" ::: "memory");
        if (threadIdx.x >= 448) touch(P, threadIdx.x & 63);
    }
    const int pi = (j - GPX) * 8 + xcd;
    if (j >= GPX && pi < NPULL) puller(P, myws, (LAS unsigned*)shm + 8 * BLK / 4, pi);
    else {
        const unsigned call = prepass(P, myws, (LAS unsigned char*)shm, j < GPX ? b : GPX * 8 + pi - NPULL);
        STAMP(1);
        if (j < GPX) gemm(P, myws, (LAS unsigned char*)shm, call, xcd, j);
    }
}
}

extern "C" void kernel_launch(int rank, int nranks, ncclComm_t comm, hipStream_t stream,
                              void* const* d_in, const int* in_sizes, int n_in,
                              void* d_out, int out_size, const int* in_dtypes, int out_dtype,
                              void* const* d_out_all, void* const* d_ws_all, size_t ws_bytes) {
    if (nranks != NRANK || (unsigned)rank >= (unsigned)NRANK || n_in < 2 || in_sizes[0] != M * KS || in_sizes[1] != KS * N || out_size != M * N || ws_bytes < WS_NEED) {
        fprintf(stderr, "kernel_launch: unexpected problem geometry\n");
        return;
    }
    static bool attr_set = false;
    if (!attr_set) { HIP_CHECK(hipFuncSetAttribute((const void*)fused_gemm_allreduce, hipFuncAttributeMaxDynamicSharedMemorySize, LDS_BYTES)); attr_set = true; }
    Params P;
    P.x = (const u16*)d_in[0]; P.w = (const u16*)d_in[1]; P.rank = rank;
    for (int l = 0; l < NRANK; ++l) { P.ws[l] = (char*)d_ws_all[(rank + 1 + l) % NRANK]; P.out[l] = (char*)d_out_all[(rank + 1 + l) % NRANK]; }
    P.out[7] = (char*)d_out;
    hipLaunchKernelGGL(fused_gemm_allreduce, dim3(NWG), dim3(512), LDS_BYTES, stream, P);
    HIP_CHECK(hipGetLastError());
}
```
